```python
import numpy as np

kernel_name = "gpu_gemm_ar_m4096_k4096_n4096_e5m2e5m2_dequant"


ABSMAX_RTOL = 0.05

COMPUTE_DTYPE = "float32"
EPILOGUE = "dequant"
IN_QUANT = None
OUT_QUANT = None
IN_FP8 = "e5m2e5m2"
IN_DTYPES = {"x": "e5m2", "w_mat": "e5m2",
             "scale_x": "float32", "scale_w": "float32"}
WORLD_SIZE = 8

def _snap_fp8(x, mant_bits, max_abs, min_normal):
    a = np.abs(x.astype(np.float64))
    _m, _e = np.frexp(a)
    q = np.where(a >= min_normal, np.ldexp(1.0, _e - 1 - mant_bits),
                 min_normal * 2.0 ** -mant_bits)
    snapped = np.minimum(np.round(a / q) * q, max_abs)
    return (np.sign(x) * snapped).astype(np.float32)

def setup_inputs():

    x = _snap_fp8(np.random.randn(4096, 4096).astype(np.float32), 2, 57344.0, 2.0 ** -14)
    w_mat = _snap_fp8(np.random.randn(4096, 4096).astype(np.float32), 2, 57344.0, 2.0 ** -14)
    scale_x = (0.02 * (1.0 + np.random.rand(1))).astype(np.float32)
    scale_w = (0.02 * (1.0 + np.random.rand(1))).astype(np.float32)
    return {"x": x, "w_mat": w_mat, "scale_x": scale_x, "scale_w": scale_w}

def reference(x, w_mat, scale_x, scale_w):
    acc = (x @ w_mat).astype(np.float32)
    y = (acc * (scale_x * scale_w)).astype(np.float32)
    return y
```

```cpp
#include <hip/hip_runtime.h>
#include <rccl/rccl.h>
#include <cstdio>
#include <cstdint>

#define HIP_CHECK(expr) do { hipError_t _e = (expr); if (_e != hipSuccess) { \
    fprintf(stderr, "HIP error %s at %s:%d\n", hipGetErrorString(_e), __FILE__, __LINE__); \
    return; } } while (0)

namespace {

#define LAS __attribute__((address_space(3)))
typedef unsigned u32x4 __attribute__((ext_vector_type(4)));
typedef int i32x2 __attribute__((ext_vector_type(2)));
typedef int i32x4 __attribute__((ext_vector_type(4)));
typedef int i32x8 __attribute__((ext_vector_type(8)));
typedef float f32x4 __attribute__((ext_vector_type(4)));

constexpr int NR = 8;
constexpr int MN = 4096;
constexpr int KS = 512;
constexpr int NCOMM = 16;
constexpr int NGEMM = 240;
constexpr int NWG = NCOMM + NGEMM;
constexpr unsigned ENT_B = 1u << 20;
constexpr unsigned OP_B = 1u << 19;
constexpr unsigned SLOT_B = 32u << 20;
constexpr size_t DATA_OFF = 1u << 20;
constexpr size_t WS_NEED = DATA_OFF + 2ull * SLOT_B;
constexpr unsigned CTL_DONE = 0x0000, CTL_TMO = 0x0080, CTL_RFLAG = 0x1000, CTL_LFLAG = 0x5000;
constexpr unsigned long long DEADLINE_TICKS = 100000000ull;

constexpr int A_SLOT = 18 * 2048, B_SLOT = 16 * 2048;
constexpr int LDS_A = 0, LDS_B = 2 * A_SLOT;
constexpr int LDS_TBL = LDS_B + 2 * B_SLOT;
constexpr int LDS_SNAP = LDS_TBL + 352 * 4;
constexpr int LDS_ORD = LDS_SNAP + 512;
constexpr int LDS_CTL = LDS_ORD + 128;
constexpr int LDS_BYTES = LDS_CTL + 256;

struct Params {
  const uint8_t* x; const uint8_t* w; const float* sx; const float* sw; float* out;
  uint8_t* ws[NR];
  int rank;
  unsigned pubmask;
  unsigned sched[8];
  uint8_t inv[NR][32];
};

#define WAIT_V(n) asm volatile("s_waitcnt vmcnt(" #n ")" ::: "memory")
#define WAIT_L(n) asm volatile("s_waitcnt lgkmcnt(" #n ")" ::: "memory")
#define BAR do { asm volatile("" ::: "memory"); __builtin_amdgcn_s_barrier(); asm volatile("" ::: "memory"); } while (0)
#define SCHED __builtin_amdgcn_sched_barrier(0)

__device__ __forceinline__ unsigned unit_src_off(int type, int kt, int f, int h, int lane) {
  return type ? (unsigned)((kt * 128 + h * 64 + lane) * MN + f * 16)
              : (unsigned)((f * 16 + (lane & 15)) * KS + kt * 128 + h * 64 + (lane >> 4) * 16);
}
__device__ __forceinline__ u32x4 xpose(u32x4 raw, LAS unsigned char* scr, int lane) {
  *(LAS u32x4*)(scr + lane * 16) = raw;
  const int a = ((lane >> 4) * 16 + ((lane & 15) >> 1)) * 16 + 8 * (lane & 1);
  asm volatile("" ::: "memory");
  i32x2 lo = __builtin_amdgcn_ds_read_tr8_b64_v2i32((LAS i32x2*)(scr + a));
  i32x2 hi = __builtin_amdgcn_ds_read_tr8_b64_v2i32((LAS i32x2*)(scr + a + 128));
  asm volatile("" ::: "memory");
  u32x4 r; r.x = lo.x; r.y = lo.y; r.z = hi.x; r.w = hi.y; return r;
}

__device__ __forceinline__ void comm_role(const Params& p, LAS unsigned char* lds, int wg, int wv, int lane) {
  LAS unsigned* cnt = (LAS unsigned*)(lds + LDS_CTL);
  if (threadIdx.x < 32) cnt[threadIdx.x] = 0u;
  const __amdgpu_buffer_rsrc_t rx = __builtin_amdgcn_make_buffer_rsrc((void*)p.x, 0, MN * KS, 0x00020000);
  const __amdgpu_buffer_rsrc_t rw = __builtin_amdgcn_make_buffer_rsrc((void*)p.w, 0, MN * KS, 0x00020000);
  LAS unsigned char* scr = lds + wv * 1024;
  const int h = wv & 1, fsub = wv >> 1;
  auto code_of = [&](int r) -> int { return (int)((p.sched[(r >> 2) & 7] >> ((r & 3) * 8)) & 31u); };
  auto issue_load = [&](int r) -> u32x4 {
    const int c = code_of(r < 32 ? r : 31), type = (c >> 2) & 1;
    const unsigned off = unit_src_off(type, c >> 3, ((c & 3) * 16 + wg) * 4 + fsub, h, lane);
    u32x4 v;
    if (type) asm volatile("buffer_load_dwordx4 %0, %1, %2, 0 offen" : "=v"(v) : "v"(off), "s"(rw) : "memory");
    else      asm volatile("buffer_load_dwordx4 %0, %1, %2, 0 offen" : "=v"(v) : "v"(off), "s"(rx) : "memory");
    return v;
  };
  u32x4 raw = issue_load(0);
  const unsigned call = __builtin_amdgcn_readfirstlane(
      __hip_atomic_load((const unsigned*)(p.ws[p.rank] + CTL_DONE), __ATOMIC_RELAXED, __HIP_MEMORY_SCOPE_AGENT)) / NWG + 1u;
  asm volatile("s_waitcnt vmcnt(0)" : "+v"(raw) :: "memory");
  __amdgpu_buffer_rsrc_t rs[7];
#pragma unroll
  for (int j = 0; j < 7; ++j) {
    const int peer = (p.rank + 1 + (j + wg + wv) % 7) & (NR - 1);
    rs[j] = __builtin_amdgcn_make_buffer_rsrc(p.ws[peer] + DATA_OFF + (size_t)(call & 1u) * SLOT_B + (size_t)p.rank * 4 * ENT_B,
                                              0, 4 * ENT_B, 0x00020000);
  }
  unsigned* myflag = (unsigned*)(p.ws[(p.rank + 1 + lane % 7) & (NR - 1)] + CTL_RFLAG) + p.rank * 512 + wg;
  __syncthreads();
  u32x4 cur = ((code_of(0) >> 2) & 1) ? xpose(raw, scr, lane) : raw;
  raw = issue_load(1);
  for (int r = 0; r < 32; ++r) {
    const int c = code_of(r);
    const unsigned so = (unsigned)(c >> 3) * ENT_B + (unsigned)((c >> 2) & 1) * OP_B +
                        (unsigned)((((c & 3) * 16 + wg) * 4 + fsub) * 2 + h) * 1024u + lane * 16;
#pragma unroll
    for (int j = 0; j < 7; ++j)
      asm volatile("buffer_store_dwordx4 %0, %1, %2, 0 offen sc0 sc1\n\ts_nop 1" :: "v"(cur), "v"(so), "s"(rs[j]) : "memory");
    asm volatile("s_waitcnt vmcnt(7)" : "+v"(raw) :: "memory");
    u32x4 nxt = ((code_of(r + 1 < 32 ? r + 1 : 31) >> 2) & 1) ? xpose(raw, scr, lane) : raw;
    asm volatile("" : "+v"(nxt));
    raw = issue_load(r + 2);
    WAIT_V(1);
    if ((p.pubmask >> r) & 1u) {
      unsigned old = 0;
      if (lane == 0) old = __hip_atomic_fetch_add(cnt + r, 1u, __ATOMIC_RELAXED, __HIP_MEMORY_SCOPE_WORKGROUP);
      old = __builtin_amdgcn_readfirstlane(old);
      if (old == 7u && lane < 7) __hip_atomic_store(myflag + r * 16, call, __ATOMIC_RELAXED, __HIP_MEMORY_SCOPE_SYSTEM);
    }
    cur = nxt;
  }
  asm volatile("s_waitcnt vmcnt(0)" : "+v"(raw) :: "memory");
}

#define DMA2(m0v, soff) asm volatile("s_mov_b32 m0, %0\n\ts_nop 0\n\t" \
    "buffer_load_dwordx4 %1, %2, %3 offen sc1 lds\n\t" \
    "buffer_load_dwordx4 %1, %2, %3 offen offset:1024 sc1 lds" \
    :: "s"(m0v), "v"(voff), "s"(rdata), "s"(soff) : "memory")
#define DMA3(m0v, soff) asm volatile("s_mov_b32 m0, %0\n\ts_nop 0\n\t" \
    "buffer_load_dwordx4 %1, %2, %3 offen sc1 lds\n\t" \
    "buffer_load_dwordx4 %1, %2, %3 offen offset:1024 sc1 lds\n\t" \
    "s_mov_b32 exec_hi, 0\n\t" \
    "buffer_load_dwordx4 %1, %2, %3 offen offset:2048 sc1 lds\n\t" \
    "s_mov_b32 exec_hi, -1" \
    :: "s"(m0v), "v"(voff), "s"(rdata), "s"(soff) : "memory")

__device__ __forceinline__ int ready_from(unsigned long long okmask, int base) {
  const unsigned long long nb = ~okmask & 0x7fffffffffffffffull;
  const int e = nb ? (int)((unsigned)__builtin_ctzll(nb) * 57u >> 9) : 7;
  const int r = base + e;
  return r < 32 ? r : 32;
}

__device__ __forceinline__ void gemm_role(const Params& p, LAS unsigned char* lds, int g, int wv, int lane) {
  const int tcol = 2 * (g & 7) + ((g >> 3) & 1), trow = g >> 4;
  uint8_t* const ctl = p.ws[p.rank];
  const unsigned call = __builtin_amdgcn_readfirstlane(
      __hip_atomic_load((const unsigned*)(ctl + CTL_DONE), __ATOMIC_RELAXED, __HIP_MEMORY_SCOPE_AGENT)) / NWG + 1u;
  const __amdgpu_buffer_rsrc_t rdata =
      __builtin_amdgcn_make_buffer_rsrc(ctl + DATA_OFF + (size_t)(call & 1u) * SLOT_B, 0, SLOT_B, 0x00020000);
  {
    const __amdgpu_buffer_rsrc_t rx = __builtin_amdgcn_make_buffer_rsrc((void*)p.x, 0, MN * KS, 0x00020000);
    const __amdgpu_buffer_rsrc_t rw = __builtin_amdgcn_make_buffer_rsrc((void*)p.w, 0, MN * KS, 0x00020000);
    const int h = wv & 1, fsub = wv >> 1;
    u32x4 raw[3];
#pragma unroll
    for (int i = 0; i < 3; ++i) {
      const int gid = g + i * NGEMM, type = (gid >> 8) & 1, kt = (gid >> 6) & 3, f = (gid & 63) * 4 + fsub;
      if (gid < 512) {
        const unsigned off = unit_src_off(type, kt, f, h, lane);
        raw[i] = type ? __builtin_amdgcn_raw_buffer_load_b128(rw, off, 0, 0) : __builtin_amdgcn_raw_buffer_load_b128(rx, off, 0, 0);
      }
    }
#pragma unroll
    for (int i = 0; i < 3; ++i) {
      const int gid = g + i * NGEMM, type = (gid >> 8) & 1, kt = (gid >> 6) & 3, f = (gid & 63) * 4 + fsub;
      if (gid < 512) {
        const u32x4 v = type ? xpose(raw[i], lds + (wv * 3 + i) * 1024, lane) : raw[i];
        __builtin_amdgcn_raw_buffer_store_b128(v, rdata, (unsigned)(p.rank * 4 + kt) * ENT_B + (unsigned)type * OP_B + (unsigned)(f * 2 + h) * 1024u + lane * 16, 0, 17);
      }
    }
  }
  LAS unsigned* ord = (LAS unsigned*)(lds + LDS_ORD);
  LAS unsigned* tbl = (LAS unsigned*)(lds + LDS_TBL);
  LAS unsigned* bc = (LAS unsigned*)(lds + LDS_CTL);
  const int xg0 = (17 * trow) >> 2, wg0 = 4 * tcol;
  auto pub_round = [&](int s, int kt, int i) -> int {
    const int type = i >= 5, gg = type ? wg0 + i - 5 : xg0 + i;
    const int r = p.inv[s][kt * 8 + type * 4 + (gg >> 4)];
    return r + __builtin_ctz(p.pubmask >> r);
  };
  if (wv == 0) {
    const int e = lane & 31, s = e >> 2, kt = e & 3;
    int key = kt;
    if (s != p.rank) {
      int m = 0;
      for (int i = 0; i < 9; ++i) { const int r = pub_round(s, kt, i); m = r > m ? r : m; }
      key = 4 + m * 32 + kt * 8 + ((s - p.rank + g) & 7);
    }
    int rk = 0;
    for (int l = 0; l < 32; ++l) rk += __builtin_amdgcn_readlane(key, l) < key;
    if (lane < 32) ord[rk] = e;
  }
  WAIT_V(0);
  __syncthreads();
  if (threadIdx.x == 0)
    __hip_atomic_store((unsigned*)(ctl + CTL_LFLAG) + g, call, __ATOMIC_RELAXED, __HIP_MEMORY_SCOPE_SYSTEM);
  if (threadIdx.x < 352) {
    int t = threadIdx.x / 9; const int i = threadIdx.x - t * 9; t = t < 32 ? t : 31;
    const int e = ord[t], s = e >> 2, kt = e & 3, type = i >= 5, gg = type ? wg0 + i - 5 : xg0 + i;
    unsigned off;
    if (s == p.rank) off = CTL_LFLAG + 4u * (unsigned)((type * 256 + kt * 64 + gg) % NGEMM);
    else off = CTL_RFLAG + 4u * (unsigned)(s * 512 + pub_round(s, kt, i) * 16 + (gg & 15));
    tbl[threadIdx.x] = off;
  }
  const int ordv = ord[lane & 31];
  const float scale = p.sx[0] * p.sw[0];
  WAIT_V(0);
  __syncthreads();

  const int wr = wv >> 2, wc = wv & 3;
  const unsigned voff = lane * 16;
  const unsigned ldsb = (unsigned)(uintptr_t)lds;
  const unsigned a0o = 9 * wr * 2048 + wc * 2560, a1o = (9 * wr + 5 + wc) * 2048, b0o = (4 * (wv >> 1) + (wv & 1)) * 2048;
  const unsigned gA = 17u * trow * 2048u, gB = OP_B + tcol * 32768u;
  const unsigned ga0 = gA + a0o, ga1 = gA + a1o - ((wv == 7 && trow < 14) ? 2048u : 0u), gb0 = gB + b0o;
  const unsigned la0 = ldsb + LDS_A + a0o, la1 = ldsb + LDS_A + a1o, lb0 = ldsb + LDS_B + b0o;
#define ENT(t) ((unsigned)__builtin_amdgcn_readlane(ordv, (t)) * ENT_B)
#define ST_A0(eb, sl) DMA3(la0 + (sl) * A_SLOT, (eb) + ga0)
#define ST_A1(eb, sl) DMA2(la1 + (sl) * A_SLOT, (eb) + ga1)
#define ST_B0(eb, sl) DMA2(lb0 + (sl) * B_SLOT, (eb) + gb0)
#define ST_B1(eb, sl) DMA2(lb0 + 4096 + (sl) * B_SLOT, (eb) + gb0 + 4096)
#define LDF(dst, base, off) do { const i32x4 _l = *(const LAS i32x4*)((base) + (off)), _h = *(const LAS i32x4*)((base) + (off) + 1024); \
    dst = __builtin_shufflevector(_l, _h, 0, 1, 2, 3, 4, 5, 6, 7); } while (0)
#define MMA(m0_, nm, n0_, Bf) do { __builtin_amdgcn_s_setprio(1); \
    _Pragma("unroll") for (int m = 0; m < nm; ++m) _Pragma("unroll") for (int n = 0; n < 2; ++n) \
      acc[m0_ + m][n0_ + n] = __builtin_amdgcn_mfma_scale_f32_16x16x128_f8f6f4(Bf[n], At[m], acc[m0_ + m][n0_ + n], 1, 1, 0, 0, 0, 0); \
    __builtin_amdgcn_s_setprio(0); } while (0)

  f32x4 acc[9][4];
#pragma unroll
  for (int m = 0; m < 9; ++m)
#pragma unroll
    for (int n = 0; n < 4; ++n) acc[m][n] = (f32x4){0.f, 0.f, 0.f, 0.f};
  i32x8 At[5], B0[2], B1[2];
  int t = 0, ready = 0, nrestart = 0;
  unsigned dead = 0;
  while (t < 32) {
    if (ready <= t) {
      if (wv == 0) {
        const unsigned* fp = (const unsigned*)(ctl + tbl[t * 9 + lane]);
        unsigned* tmo = (unsigned*)(ctl + CTL_TMO);
        const unsigned long long t0 = __builtin_amdgcn_s_memrealtime();
        int r; unsigned dd = 0;
        for (;;) {
          const unsigned v = __hip_atomic_load(fp, __ATOMIC_RELAXED, __HIP_MEMORY_SCOPE_SYSTEM);
          r = ready_from(__builtin_amdgcn_ballot_w64((int)(v - call) >= 0), t);
          if (r > t) break;
          if (__builtin_amdgcn_readfirstlane(__hip_atomic_load(tmo, __ATOMIC_RELAXED, __HIP_MEMORY_SCOPE_AGENT)) == call ||
              __builtin_amdgcn_s_memrealtime() - t0 > DEADLINE_TICKS) {
            if (lane == 0) __hip_atomic_store(tmo, call, __ATOMIC_RELAXED, __HIP_MEMORY_SCOPE_AGENT);
            r = 32; dd = 64; break;
          }
          __builtin_amdgcn_s_sleep(2);
        }
        if (lane == 0) bc[nrestart & 63] = (unsigned)r | (dd << 8);
        asm volatile("s_waitcnt vmcnt(0) lgkmcnt(0)" ::: "memory");
      }
      BAR;
      const unsigned v = __builtin_amdgcn_readfirstlane(bc[nrestart & 63]);
      ready = v & 63; dead |= v >> 8; ++nrestart;
    }
    int s1 = ready > t + 1;
    { const unsigned e0 = ENT(t); const int sl = t & 1;
      ST_B0(e0, sl); ST_A0(e0, sl); ST_B1(e0, sl); ST_A1(e0, sl);
      if (s1) { const unsigned e1 = ENT(t + 1); ST_B0(e1, sl ^ 1); ST_A0(e1, sl ^ 1); ST_B1(e1, sl ^ 1); } }
    if (wr == 1) BAR;
    if (s1) WAIT_V(7); else WAIT_V(0);
    BAR;
    BAR;
    for (;;) {
      const int sl = t & 1;
      const LAS unsigned char* pa = lds + LDS_A + sl * A_SLOT + wr * (9 * 2048) + lane * 16;
      const LAS unsigned char* pb = lds + LDS_B + sl * B_SLOT + wc * (4 * 2048) + lane * 16;
      const int pbase = ready;
      LDF(B0[0], pb, 0); LDF(B0[1], pb, 2048); SCHED;
#pragma unroll
      for (int m = 0; m < 5; ++m) LDF(At[m], pa, m * 2048);
      if (wv == 0 && ready < 32) {
        const unsigned fo = tbl[ready * 9 + lane];
        asm volatile("s_mov_b32 m0, %0\n\ts_nop 0\n\tglobal_load_lds_dword %1, %2 sc0 sc1"
                     :: "s"(ldsb + LDS_SNAP + sl * 256), "v"(fo), "s"(ctl) : "memory");
      }
      if (s1) { ST_A1(ENT(t + 1), sl ^ 1); }
      else if (ready > t + 1) { const unsigned e1 = ENT(t + 1); ST_B0(e1, sl ^ 1); ST_A0(e1, sl ^ 1); ST_B1(e1, sl ^ 1); ST_A1(e1, sl ^ 1); s1 = 1; }
      const int s2 = s1 && ready > t + 2;
      const unsigned e2 = ENT(s2 ? t + 2 : t);
      WAIT_L(10); BAR; WAIT_L(0); SCHED; MMA(0, 5, 0, B0); BAR; SCHED;
      LDF(B1[0], pb, 4096); LDF(B1[1], pb, 6144);
      if (s2) ST_B0(e2, sl);
      BAR; WAIT_L(0); SCHED; MMA(0, 5, 2, B1); BAR; SCHED;
#pragma unroll
      for (int m = 0; m < 4; ++m) LDF(At[m], pa, (5 + m) * 2048);
      if (s2) ST_A0(e2, sl);
      BAR; WAIT_L(0); SCHED; MMA(5, 4, 0, B0); BAR; SCHED;
      if (s2) { ST_B1(e2, sl); WAIT_V(7); } else WAIT_V(0);
      BAR;
      unsigned snap = call;
      if (pbase < 32) snap = *(const LAS unsigned*)(lds + LDS_SNAP + sl * 256 + lane * 4);
      SCHED; MMA(5, 4, 2, B1); BAR; SCHED;
      if (pbase < 32) ready = ready_from(__builtin_amdgcn_ballot_w64((int)(snap - call) >= 0), pbase);
      ++t;
      if (!s1) break;
      s1 = s2;
    }
    if (wr == 0) BAR;
  }
  {
    int tz = threadIdx.x; asm volatile("" : "+v"(tz));
    const int fr = tz & 15, fq = (tz >> 4) & 3;
    const float sc = dead ? __builtin_nanf("") : scale;
    float* o = p.out + (size_t)(trow * 272 + wr * 144 + fr) * MN + tcol * 256 + wc * 64 + fq * 4;
#pragma unroll
    for (int m = 0; m < 9; ++m) {
      if (m == 8 && wr == 1 && trow < 14) break;
#pragma unroll
      for (int n = 0; n < 4; ++n) *(f32x4*)(o + (size_t)m * 16 * MN + n * 16) = acc[m][n] * sc;
    }
  }
}

__global__ void __launch_bounds__(512, 2) gemm_ag_kernel(const Params p) {
  __shared__ __attribute__((aligned(16))) unsigned char lds_raw[LDS_BYTES];
  LAS unsigned char* lds = (LAS unsigned char*)lds_raw;
  const int wv = __builtin_amdgcn_readfirstlane(threadIdx.x >> 6), lane = threadIdx.x & 63;
  if (blockIdx.x < NCOMM) comm_role(p, lds, blockIdx.x, wv, lane);
  else gemm_role(p, lds, blockIdx.x - NCOMM, wv, lane);
  if (threadIdx.x == 0)
    __hip_atomic_fetch_add((unsigned*)(p.ws[p.rank] + CTL_DONE), 1u, __ATOMIC_RELAXED, __HIP_MEMORY_SCOPE_AGENT);
}

static void make_schedule(int s, uint8_t* sc) {
  (void)s;
  int r = 0;
  for (int kt = 0; kt < 4; ++kt) {
    for (int q = 0; q < 4; ++q) sc[r++] = (uint8_t)(kt * 8 + 4 + q);
    for (int q = 0; q < 4; ++q) sc[r++] = (uint8_t)(kt * 8 + q);
  }
}

}

extern "C" void kernel_launch(int rank, int nranks, ncclComm_t comm,
                              hipStream_t stream,
                              void* const* d_in, const int* in_sizes, int n_in,
                              void* d_out, int out_size,
                              const int* in_dtypes, int out_dtype,
                              void* const* d_out_all,
                              void* const* d_ws_all, size_t ws_bytes) {
  (void)comm; (void)in_dtypes; (void)out_dtype; (void)d_out_all;
  if (nranks != NR || n_in != 4 || (unsigned)rank >= (unsigned)NR || ws_bytes < WS_NEED || out_size != MN * MN ||
      in_sizes[0] != MN * KS || in_sizes[1] != MN * KS) {
    fprintf(stderr, "kernel_launch: unexpected problem shape\n");
    return;
  }
  Params p;
  p.x = (const uint8_t*)d_in[0]; p.w = (const uint8_t*)d_in[1];
  p.sx = (const float*)d_in[2]; p.sw = (const float*)d_in[3];
  p.out = (float*)d_out;
  for (int i = 0; i < NR; ++i) p.ws[i] = (uint8_t*)d_ws_all[i];
  p.rank = rank;
  p.pubmask = 0xF0808080u;
  for (int s = 0; s < NR; ++s) {
    uint8_t sc[32];
    make_schedule(s, sc);
    for (int r = 0; r < 32; ++r) p.inv[s][sc[r]] = (uint8_t)r;
    if (s == rank)
      for (int i = 0; i < 8; ++i)
        p.sched[i] = sc[4 * i] | (sc[4 * i + 1] << 8) | (sc[4 * i + 2] << 16) | ((unsigned)sc[4 * i + 3] << 24);
  }
  hipLaunchKernelGGL(gemm_ag_kernel, dim3(NWG), dim3(512), 0, stream, p);
  HIP_CHECK(hipGetLastError());
}
```

```cpp
#include <hip/hip_runtime.h>
#include <rccl/rccl.h>
#include <cstdio>
#include <cstdint>

#define HIP_CHECK(expr) do { hipError_t _e = (expr); if (_e != hipSuccess) { \
    fprintf(stderr, "HIP error %s at %s:%d\n", hipGetErrorString(_e), __FILE__, __LINE__); \
    return; } } while (0)

#define COMM_DRAIN 0
#define DMA_POL "sc1"
#define XMAP 2
#define WINM1 4
#define POLL_SLEEP 2
#define OUT_AUX 16

namespace {

#define LAS __attribute__((address_space(3)))
typedef unsigned u32x4 __attribute__((ext_vector_type(4)));
typedef int i32x2 __attribute__((ext_vector_type(2)));
typedef int i32x4 __attribute__((ext_vector_type(4)));
typedef int i32x8 __attribute__((ext_vector_type(8)));
typedef float f32x4 __attribute__((ext_vector_type(4)));

constexpr int NR = 8;
constexpr int MN = 4096;
constexpr int KS = 512;
constexpr int NCOMM = 16;
constexpr int NGEMM = 240;
constexpr int NWG = NCOMM + NGEMM;
constexpr unsigned ENT_B = 1u << 20;
constexpr unsigned OP_B = 1u << 19;
constexpr unsigned SLOT_B = 32u << 20;
constexpr size_t DATA_OFF = 1u << 20;
constexpr size_t WS_NEED = DATA_OFF + 2ull * SLOT_B;
constexpr unsigned CTL_DONE = 0x0000, CTL_TMO = 0x0080, CTL_TICK = 0x0200, CTL_UFLAG = 0x0800, CTL_RFLAG = 0x1000, CTL_LFLAG = 0x5000;
constexpr unsigned long long DEADLINE_TICKS = 100000000ull;

constexpr int A_SLOT = 18 * 2048, B_SLOT = 16 * 2048;
constexpr int LDS_A = 0, LDS_B = 2 * A_SLOT;
constexpr int LDS_TBL = LDS_B + 2 * B_SLOT;
constexpr int LDS_SNAP = LDS_TBL + 352 * 4;
constexpr int LDS_ORD = LDS_SNAP + 6144;
constexpr int LDS_CTL = LDS_ORD + 128;
constexpr int LDS_BYTES = LDS_CTL + 256;

struct Params {
  const uint8_t* x; const uint8_t* w; const float* sx; const float* sw; float* out;
  uint8_t* ws[NR];
  int rank;
  unsigned pubmask;
  int fine_start;
  unsigned sched[32];
  uint8_t inv[NR][128];
};

#define STAMP(i)
#define STAMPC(i)
#define STAMP_DECL
#define STAMP_FLUSH
#define WAIT_V(n) asm volatile("s_waitcnt vmcnt(" #n ")" ::: "memory")
#define WAIT_L(n) asm volatile("s_waitcnt lgkmcnt(" #n ")" ::: "memory")
#define BAR do { asm volatile("" ::: "memory"); __builtin_amdgcn_s_barrier(); asm volatile("" ::: "memory"); } while (0)
#define SCHED __builtin_amdgcn_sched_barrier(0)

__device__ __forceinline__ unsigned tile_src_off(int type, int kt, int g, int wv, int lane) {
  const int row = wv * 8 + (lane >> 3), ch = (lane & 7) * 16;
  return type ? (unsigned)((kt * 128 + (g & 1) * 64 + row) * MN + (g >> 1) * 128 + ch)
              : (unsigned)((g * 64 + row) * KS + kt * 128 + ch);
}
__device__ __forceinline__ unsigned unit_dst_off(int type, int kt, int g, int wv) {
  const int u = type ? ((g >> 1) * 8 + wv) * 2 + (g & 1) : g * 8 + wv;
  return (unsigned)kt * ENT_B + (unsigned)type * OP_B + (unsigned)u * 1024u;
}
__device__ __forceinline__ u32x4 unit_read(int type, const LAS unsigned char* tile, int wv, int lane) {
  if (type) {
    const LAS unsigned* a = (const LAS unsigned*)(tile + (lane >> 4) * 2048 + (wv >> 2) * 64 + (lane & 15) * 4);
    const unsigned n = wv & 3, s1 = n | ((n + 4u) << 8);
    u32x4 r;
#pragma unroll
    for (int i = 0; i < 4; ++i) {
      const unsigned lo = __builtin_amdgcn_perm(a[(4 * i + 1) * 32], a[(4 * i) * 32], s1);
      const unsigned hi = __builtin_amdgcn_perm(a[(4 * i + 3) * 32], a[(4 * i + 2) * 32], s1);
      r[i] = __builtin_amdgcn_perm(hi, lo, 0x05040100u);
    }
    return r;
  }
  return *(const LAS u32x4*)(tile + ((wv >> 1) * 16 + (lane & 15)) * 128 + (wv & 1) * 64 + (lane >> 4) * 16);
}

#define TLOAD "s_mov_b32 m0, %[lm]\n\ts_nop 3\n\tbuffer_load_dwordx4 %[lo], %[rl], 0 offen lds\n\t"
#define TLOAD_IN [lm] "s"(lm), [lo] "v"(lo), [rl] "s"(rl)
#define STR2(x) #x
#define STR(x) STR2(x)
#define PUSH1(j) asm volatile("s_waitcnt vmcnt(" STR(WINM1) ")\n\tbuffer_store_dwordx4 %0, %1, %2, 0 offen sc0 sc1" \
                              :: "v"(cur), "v"(so), "s"(rs[j]) : "memory")
#define DRAW(vnew) asm volatile("s_mov_b64 exec, 1\n\tglobal_atomic_add " vnew ", %0, %1, %2 sc0\n\ts_mov_b64 exec, -1" \
                                :: "v"(ao), "v"(one), "s"(ctl) : "memory", "v254", "v255")
#define GOT(vold) asm volatile("v_mov_b32 %0, " vold : "=v"(aret) :: "memory", "v254", "v255")

__device__ __forceinline__ void comm_role(const Params& p, LAS unsigned char* lds, int wg, int wv, int lane) {
  STAMP_DECL; STAMP(0);
  LAS unsigned* tick = (LAS unsigned*)(lds + LDS_CTL);
  uint8_t* const ctl = p.ws[p.rank];
  const unsigned ldsw = (unsigned)(uintptr_t)lds + wv * 1024;
  auto code_of = [&](int pos) -> int { return (int)((p.sched[(pos >> 4) & 31] >> (((pos >> 2) & 3) * 8)) & 127u); };
#define TILE_ARGS(pos, slot) const int _c = code_of(pos); \
    const __amdgpu_buffer_rsrc_t rl = __builtin_amdgcn_make_buffer_rsrc((void*)((_c & 16) ? p.w : p.x), 0, MN * KS, 0x00020000); \
    const unsigned lo = tile_src_off((_c >> 4) & 1, _c >> 5, (_c & 15) * 4 + ((pos) & 3), wv, lane), lm = ldsw + ((slot) & 3) * 8192
  int pm1 = 0, p0 = wg, p1 = 16 + wg, p2 = 32 + wg;
  { TILE_ARGS(p0, 0); asm volatile(TLOAD :: TLOAD_IN : "memory"); }
  { TILE_ARGS(p1, 1); asm volatile(TLOAD :: TLOAD_IN : "memory"); }
  const unsigned call = __builtin_amdgcn_readfirstlane(
      __hip_atomic_load((const unsigned*)(ctl + CTL_DONE), __ATOMIC_RELAXED, __HIP_MEMORY_SCOPE_AGENT)) / NWG + 1u;
  __amdgpu_buffer_rsrc_t rs[7];
#pragma unroll
  for (int j = 0; j < 7; ++j) {
    const int peer = (p.rank + 1 + (j + wg + wv) % 7) & (NR - 1);
    rs[j] = __builtin_amdgcn_make_buffer_rsrc(p.ws[peer] + DATA_OFF + (size_t)(call & 1u) * SLOT_B + (size_t)p.rank * 4 * ENT_B,
                                              0, 4 * ENT_B, 0x00020000);
  }
  uint8_t* const pctl = p.ws[(p.rank + 1 + lane % 7) & (NR - 1)];
  const unsigned ao = CTL_TICK + (call & 1u) * 128u, one = 1u;
  WAIT_V(0);
  BAR;
  if (wg == 0 && threadIdx.x == 64)
    __hip_atomic_store((unsigned*)(ctl + CTL_TICK + ((call & 1u) ^ 1u) * 128u), 0u, __ATOMIC_RELAXED, __HIP_MEMORY_SCOPE_AGENT);
  unsigned open = p.pubmask;
  STAMP(1);
  for (int r = 0;; ++r) {
    u32x4 cur; unsigned so = 0;
    if (p0 < 512) {
      const int c = code_of(p0);
      cur = unit_read((c >> 4) & 1, lds + (r & 3) * 8192, wv, lane);
      so = unit_dst_off((c >> 4) & 1, c >> 5, (c & 15) * 4 + (p0 & 3), wv) + lane * 16;
#pragma unroll
      for (int j = 0; j < WINM1; ++j) PUSH1(j);
      asm volatile("s_waitcnt vmcnt(" STR(WINM1) ")" ::: "memory");
      if (wv == 0) {
        unsigned aret;
        if (r & 1) GOT("v254"); else GOT("v255");
        if (lane == 0) tick[r & 1] = aret;
        WAIT_L(0);
      }
    } else {
      WAIT_V(0);
    }
    BAR;
    STAMP(2 + (r < 34 ? r : 34));
    if (r > 0) {
      if ((pm1 >> 4) >= p.fine_start) {
        if (wv == 1 + (r & 3) && lane < 7)
          __hip_atomic_store((unsigned*)(pctl + CTL_RFLAG) + p.rank * 512 + pm1, call * 16u, __ATOMIC_RELAXED, __HIP_MEMORY_SCOPE_SYSTEM);
      }
      unsigned done = (p0 >> 4) >= 32 ? open : open & ((1u << (p0 >> 4)) - 1u);
      open &= ~done;
      for (; done; done &= done - 1u)
        if (wv == 1 + (r & 3) && lane < 7)
          __hip_atomic_fetch_add((unsigned*)(pctl + CTL_UFLAG) + p.rank * 32 + __builtin_ctz(done), 1u, __ATOMIC_RELAXED, __HIP_MEMORY_SCOPE_SYSTEM);
    }
    if (p0 >= 512) break;
    if (r > 0) p2 = p1 < 512 ? 48 + __builtin_amdgcn_readfirstlane(tick[r & 1]) : 512;
#pragma unroll
    for (int j = WINM1; j < 7; ++j) PUSH1(j);
    { const int pn = p2 < 512 ? p2 : 511; TILE_ARGS(pn, r + 2); asm volatile(TLOAD :: TLOAD_IN : "memory"); }
    if (wv == 0) { if (r & 1) DRAW("v255"); else DRAW("v254"); }
    pm1 = p0; p0 = p1; p1 = p2;
  }
  WAIT_V(0);
  STAMP(37); STAMP_FLUSH;
}

#define DMA2(m0v, soff) asm volatile("s_mov_b32 m0, %0\n\ts_nop 0\n\t" \
    "buffer_load_dwordx4 %1, %2, %3 offen " DMA_POL " lds\n\t" \
    "buffer_load_dwordx4 %1, %2, %3 offen offset:1024 " DMA_POL " lds" \
    :: "s"(m0v), "v"(voff), "s"(rdata), "s"(soff) : "memory")
#define DMA3(m0v, soff, m0h, soffh) asm volatile("s_mov_b32 m0, %0\n\ts_nop 0\n\t" \
    "buffer_load_dwordx4 %1, %2, %3 offen " DMA_POL " lds\n\t" \
    "buffer_load_dwordx4 %1, %2, %3 offen offset:1024 " DMA_POL " lds\n\t" \
    "s_mov_b32 m0, %4\n\ts_mov_b32 exec_hi, 0\n\t" \
    "buffer_load_dwordx4 %1, %2, %5 offen " DMA_POL " lds\n\t" \
    "s_mov_b32 exec_hi, -1" \
    :: "s"(m0v), "v"(voff), "s"(rdata), "s"(soff), "s"(m0h), "s"(soffh) : "memory")

__device__ __forceinline__ int ready_from(unsigned long long okmask, int base) {
  const int r = base + (int)((unsigned)__builtin_ctzll(~okmask | (1ull << 63)) * 57u >> 9);
  return r < 32 ? r : 32;
}

__device__ __forceinline__ void gemm_role(const Params& p, LAS unsigned char* lds, int g, int wv, int lane) {
  const int lin = (g & 7) * 30 + (g >> 3), tcol = lin & 15, trow = lin >> 4;
  uint8_t* const ctl = p.ws[p.rank];
  STAMP_DECL; STAMP(0);
#pragma unroll
  for (int i = 0; i < 3; ++i) {
    const int gid = g + i * NGEMM, type = (gid >> 8) & 1, kt = (gid >> 6) & 3;
    if (gid < 512) {
      const __amdgpu_buffer_rsrc_t rl = __builtin_amdgcn_make_buffer_rsrc((void*)(type ? p.w : p.x), 0, MN * KS, 0x00020000);
      const unsigned lo = tile_src_off(type, kt, gid & 63, wv, lane), lm = (unsigned)(uintptr_t)lds + i * 8192 + wv * 1024;
      asm volatile(TLOAD :: TLOAD_IN : "memory");
    }
  }
  const unsigned call = __builtin_amdgcn_readfirstlane(
      __hip_atomic_load((const unsigned*)(ctl + CTL_DONE), __ATOMIC_RELAXED, __HIP_MEMORY_SCOPE_AGENT)) / NWG + 1u;
  const __amdgpu_buffer_rsrc_t rdata =
      __builtin_amdgcn_make_buffer_rsrc(ctl + DATA_OFF + (size_t)(call & 1u) * SLOT_B, 0, SLOT_B, 0x00020000);
  WAIT_V(0);
  BAR;
#pragma unroll
  for (int i = 0; i < 3; ++i) {
    const int gid = g + i * NGEMM, type = (gid >> 8) & 1, kt = (gid >> 6) & 3;
    if (gid < 512)
      __builtin_amdgcn_raw_buffer_store_b128(unit_read(type, lds + i * 8192, wv, lane), rdata,
                                             (unsigned)p.rank * 4u * ENT_B + unit_dst_off(type, kt, gid & 63, wv) + lane * 16, 0, 17);
  }
  LAS unsigned* ord = (LAS unsigned*)(lds + LDS_ORD);
  LAS unsigned* tbl = (LAS unsigned*)(lds + LDS_TBL);
  LAS unsigned* bc = (LAS unsigned*)(lds + LDS_CTL);
  const int xg0 = (17 * trow) >> 2, wg0 = 4 * tcol;
  auto pub_pos = [&](int s, int kt, int i) -> int {
    const int type = i >= 5, gg = type ? wg0 + i - 5 : xg0 + i;
    const int pos = p.inv[s][kt * 32 + type * 16 + (gg >> 2)] * 4 + (gg & 3), r = pos >> 4;
    return r >= p.fine_start ? pos : (r + __builtin_ctz(p.pubmask >> r)) * 16 + 15;
  };
  if (wv == 0) {
    const int e = lane & 31, s = e >> 2, kt = e & 3;
    int key = kt;
    if (s != p.rank) {
      int m = 0;
      for (int i = 0; i < 9; ++i) { const int r = pub_pos(s, kt, i); m = r > m ? r : m; }
      key = 4 + m * 32 + kt * 8 + ((s - p.rank + g) & 7);
    }
    int rk = 0;
    for (int l = 0; l < 32; ++l) rk += __builtin_amdgcn_readlane(key, l) < key;
    if (lane < 32) ord[rk] = e;
  }
  WAIT_V(0);
  __syncthreads();
  if (threadIdx.x == 0)
    __hip_atomic_store((unsigned*)(ctl + CTL_LFLAG) + g, call * 16u, __ATOMIC_RELAXED, __HIP_MEMORY_SCOPE_SYSTEM);
  if (threadIdx.x < 352) {
    int t = threadIdx.x / 9; const int i = threadIdx.x - t * 9; t = t < 32 ? t : 31;
    const int e = ord[t], s = e >> 2, kt = e & 3, type = i >= 5, gg = type ? wg0 + i - 5 : xg0 + i;
    unsigned off;
    if (s == p.rank) off = CTL_LFLAG + 4u * (unsigned)((type * 256 + kt * 64 + gg) % NGEMM);
    else { const int pp = pub_pos(s, kt, i);
           off = (pp >> 4) >= p.fine_start ? CTL_RFLAG + 4u * (unsigned)(s * 512 + pp) : CTL_UFLAG + 4u * (unsigned)(s * 32 + (pp >> 4)); }
    tbl[threadIdx.x] = off;
  }
  const int ordv = ord[lane & 31];
  ((LAS unsigned long long*)(lds + LDS_SNAP))[threadIdx.x] = 0ull;
  ((LAS unsigned*)(lds + LDS_SNAP + 4096))[threadIdx.x] = 0u;
  int scale_bits = __builtin_amdgcn_readfirstlane(__builtin_bit_cast(int, p.sx[0] * p.sw[0]));
  asm volatile("" : "+s"(scale_bits));
  WAIT_V(0);
  __syncthreads();

  STAMP(1);
  const int wr = wv >> 2, wc = wv & 3;
  const unsigned voff = lane * 16;
  const unsigned ldsb = (unsigned)(uintptr_t)lds;
  const unsigned a0o = (9 * wr + wc) * 2048, a8o = (9 * wr + 8) * 2048 + wc * 512, b0o = (4 * (wv >> 1) + (wv & 1)) * 2048;
  const unsigned gA = 17u * trow * 2048u, gB = OP_B + tcol * 32768u;
  const unsigned ga0 = gA + a0o, ga8 = gA + a8o - ((wr == 1 && trow < 14) ? 2048u : 0u), gb0 = gB + b0o;
  const unsigned la0 = ldsb + LDS_A + a0o, la8 = ldsb + LDS_A + a8o, lb0 = ldsb + LDS_B + b0o;
#define ENT(t) ((unsigned)__builtin_amdgcn_readlane(ordv, (t)) * ENT_B)
#define ST_A03(eb, sl) DMA2(la0 + (sl) * A_SLOT, (eb) + ga0)
#define ST_A48(eb, sl) DMA3(la0 + 8192 + (sl) * A_SLOT, (eb) + ga0 + 8192, la8 + (sl) * A_SLOT, (eb) + ga8)
#define ST_B0(eb, sl) DMA2(lb0 + (sl) * B_SLOT, (eb) + gb0)
#define ST_B1(eb, sl) DMA2(lb0 + 4096 + (sl) * B_SLOT, (eb) + gb0 + 4096)
#define ST_ALL(eb, sl) do { ST_B0(eb, sl); ST_B1(eb, sl); ST_A03(eb, sl); ST_A48(eb, sl); } while (0)
#define LDF(dst, base, off) do { const i32x4 _l = *(const LAS i32x4*)((base) + (off)), _h = *(const LAS i32x4*)((base) + (off) + 1024); \
    dst = __builtin_shufflevector(_l, _h, 0, 1, 2, 3, 4, 5, 6, 7); SCHED; } while (0)
#define MM(m, n, Af) do { acc[m][n] = __builtin_amdgcn_mfma_scale_f32_16x16x128_f8f6f4(Af, Bf[n], acc[m][n], 1, 1, 0, 0, 0, 0); SCHED; } while (0)

  f32x4 acc[9][4];
#pragma unroll
  for (int m = 0; m < 9; ++m)
#pragma unroll
    for (int n = 0; n < 4; ++n) acc[m][n] = (f32x4){0.f, 0.f, 0.f, 0.f};
  i32x8 a0, a1, a2, a3, a4, Bf[4];
  int t = 0, ready = 0, nrestart = 0;
  unsigned dead = 0;
  const unsigned thr = call * 16u;
  constexpr unsigned OOB = 0x40000000u;
  const unsigned pollo = tbl[threadIdx.x < 352 ? threadIdx.x : 351];
  const __amdgpu_buffer_rsrc_t rctl = __builtin_amdgcn_make_buffer_rsrc(ctl, 0, 0x10000, 0x00020000);
  const __amdgpu_buffer_rsrc_t rout = __builtin_amdgcn_make_buffer_rsrc(p.out, 0, MN * MN * 4, 0x00020000);
  const LAS unsigned char* const pa0 = lds + LDS_A + wr * (9 * 2048) + lane * 16;
  const LAS unsigned char* const pb0 = lds + LDS_B + wc * (4 * 2048) + lane * 16;
#define POLL(buf) do { const unsigned _po = (unsigned)(ready * 9 + 62 - wv * 64) < 126u && ready < 32 ? 0u : OOB; \
    asm volatile("s_mov_b32 m0, %0\n\ts_nop 0\n\tbuffer_load_dword %1, %2, %3 offen sc0 sc1 lds" \
                 :: "s"(ldsb + LDS_SNAP + (buf) * 2048 + wv * 256), "v"(pollo), "s"(rctl), "s"(_po) : "memory"); } while (0)
#define SNAP_AT(buf) ((const LAS unsigned*)(lds + LDS_SNAP + ((buf) * 512 + ready * 9) * 4) + lane)
#define OKMASK(v) __builtin_amdgcn_ballot_w64((int)((v) - thr) >= 0)
#define STAGE_FROM(u) do { const unsigned _e0 = ENT(u); ST_ALL(_e0, (u) & 1); sn = ready > (u) + 1; \
    if (sn) { const unsigned _e1 = ENT((u) + 1); ST_ALL(_e1, ((u) & 1) ^ 1); } else POLL(((u) & 1) ^ 1); } while (0)
  int sn = -1;
  for (;;) {
    if (sn < 0) {
      if (ready <= t) {
        if (wv == 0) {
          const unsigned* fp = (const unsigned*)(ctl + (lane == 63 ? CTL_TMO : tbl[t * 9 + lane]));
          unsigned* tmo = (unsigned*)(ctl + CTL_TMO);
          const unsigned long long t0 = __builtin_amdgcn_s_memrealtime();
          int r; unsigned dd = 0;
          for (;;) {
            const unsigned v = __hip_atomic_load(fp, __ATOMIC_RELAXED, __HIP_MEMORY_SCOPE_SYSTEM);
            r = ready_from(OKMASK(v), t);
            if (r > t) break;
            if ((unsigned)__builtin_amdgcn_readlane(v, 63) == call || __builtin_amdgcn_s_memrealtime() - t0 > DEADLINE_TICKS) {
              if (lane == 0) __hip_atomic_store(tmo, call, __ATOMIC_RELAXED, __HIP_MEMORY_SCOPE_AGENT);
              r = 32; dd = 64; break;
            }
            __builtin_amdgcn_s_sleep(POLL_SLEEP);
          }
          if (lane == 0) bc[nrestart & 63] = (unsigned)r | (dd << 8);
          asm volatile("s_waitcnt vmcnt(0) lgkmcnt(0)" ::: "memory");
        }
        BAR;
        const unsigned v = __builtin_amdgcn_readfirstlane(bc[nrestart & 63]);
        ready = v & 63; dead |= v >> 8; ++nrestart;
      }
      STAGE_FROM(t);
    }
    int s1 = sn;
    sn = -1;
    if (s1) WAIT_V(12); else WAIT_V(4);
    BAR;
    { const LAS unsigned char* pa = pa0 + (t & 1) * A_SLOT; const LAS unsigned char* pb = pb0 + (t & 1) * B_SLOT;
      LDF(Bf[0], pb, 0); LDF(a0, pa, 0); LDF(a1, pa, 2048); LDF(Bf[1], pb, 2048); LDF(Bf[2], pb, 4096); LDF(a2, pa, 4096); LDF(Bf[3], pb, 6144); }
    if (t == 31) break;
    unsigned en2 = ENT((t + 2) & 31);
    for (;;) {
      const int sl = t & 1;
      const LAS unsigned char* pa = pa0 + sl * A_SLOT;
      const LAS unsigned char* pan = pa0 + (sl ^ 1) * A_SLOT; const LAS unsigned char* pbn = pb0 + (sl ^ 1) * B_SLOT;
      LDF(a3, pa, 3 * 2048);
      MM(0, 0, a0); MM(1, 0, a1);
      const LAS unsigned* psn = SNAP_AT(sl ^ s1 ^ 1);
      asm volatile("" : "+v"(psn), "+v"(pan), "+v"(pbn));
      SCHED;
      MM(0, 1, a0); MM(1, 1, a1); MM(0, 2, a0); MM(1, 2, a1);
      if (s1) WAIT_V(9); else WAIT_V(0);
      WAIT_L(0); BAR; SCHED;
      const unsigned snap = *psn;
      SCHED;
      MM(0, 3, a0);
      LDF(a4, pa, 8 * 2048);
      MM(1, 3, a1);
      LDF(a0, pa, 4 * 2048);
      MM(2, 0, a2);
      const unsigned long long okm = OKMASK(snap);
      SCHED;
      MM(2, 1, a2);
      LDF(a1, pa, 5 * 2048);
      MM(2, 2, a2);
      ready = ready_from(okm, ready);
      if (__builtin_expect(!s1, 0)) {
        if (ready > t + 1) { const unsigned e1 = ENT(t + 1); ST_ALL(e1, sl ^ 1); s1 = 1; } else POLL(2);
      }
      const int s2 = s1 & (ready > t + 2);
      const unsigned e2 = s2 ? en2 : OOB;
      ST_B0(e2, sl);
      SCHED;
      MM(2, 3, a2);
      LDF(a2, pa, 6 * 2048);
      MM(3, 0, a3); MM(3, 1, a3);
      ST_B1(e2, sl);
      SCHED;
      MM(3, 2, a3); MM(3, 3, a3);
      LDF(a3, pa, 7 * 2048);
      MM(4, 0, a0); MM(4, 1, a0);
      ST_A03(e2, sl);
      SCHED;
      MM(4, 2, a0);
      en2 = ENT((t + 3) & 31);
      SCHED;
      MM(4, 3, a0); MM(5, 0, a1); MM(5, 1, a1);
      if (s1) WAIT_V(9); else WAIT_V(0);
      WAIT_L(0); BAR; SCHED;
      MM(5, 2, a1);
      LDF(a0, pan, 0);
      POLL(sl);
      SCHED;
      MM(5, 3, a1);
      LDF(a1, pan, 2048);
      MM(6, 0, a2);
      ST_A48(e2, sl);
      if (__builtin_expect(!s1, 0)) {
        ready = ready_from(OKMASK(*SNAP_AT(2)), ready);
        if (ready > t + 1) STAGE_FROM(t + 1);
      }
      SCHED;
      MM(6, 1, a2); MM(6, 2, a2); MM(6, 3, a2);
      LDF(a2, pan, 4096);
      MM(7, 0, a3); MM(8, 0, a4);
      LDF(Bf[0], pbn, 0);
      MM(7, 1, a3); MM(8, 1, a4);
      LDF(Bf[1], pbn, 2048);
      MM(7, 2, a3); MM(8, 2, a4);
      LDF(Bf[2], pbn, 4096);
      MM(7, 3, a3); MM(8, 3, a4);
      LDF(Bf[3], pbn, 6144);
      STAMP(2 + t); if (t == 24) STAMPC(40); if (t == 29) STAMPC(41);
      ++t;
      if (!s1 || t == 31) break;
      s1 = s2;
    }
    if (s1) break;
    if (sn < 0) {
      asm volatile("s_waitcnt vmcnt(0) lgkmcnt(0)" ::: "memory"); BAR;
      ready = ready_from(OKMASK(*SNAP_AT((t & 1) ^ 1)), ready);
      WAIT_L(0); BAR;
    }
  }
  {
    const float sc = __builtin_bit_cast(float, dead ? 0x7fc00000 : scale_bits);
    const unsigned ov = (unsigned)((lane >> 4) * 4 * MN + (lane & 15) * 4) * 4u;
    const unsigned os = (unsigned)((trow * 272 + wr * 144) * MN + tcol * 256 + wc * 64) * 4u;
#define OUT(m) do { f32x4 o[4]; \
      _Pragma("unroll") for (int j = 0; j < 4; ++j) { o[j][0] = acc[m][0][j] * sc; o[j][1] = acc[m][1][j] * sc; o[j][2] = acc[m][2][j] * sc; o[j][3] = acc[m][3][j] * sc; } \
      SCHED; \
      _Pragma("unroll") for (int j = 0; j < 4; ++j) \
        __builtin_amdgcn_raw_buffer_store_b128(__builtin_bit_cast(u32x4, o[j]), rout, ov, os + (unsigned)((m) * 16 + j) * (MN * 4u), OUT_AUX); \
      asm volatile("s_nop 1" ::: "memory");                     \
      SCHED; } while (0)
    const LAS unsigned char* pa = pa0 + A_SLOT;
    LDF(a3, pa, 3 * 2048);
    MM(0, 0, a0); MM(1, 0, a1); MM(0, 1, a0); MM(1, 1, a1); MM(0, 2, a0); MM(1, 2, a1);
    WAIT_V(0); BAR; SCHED;
    MM(0, 3, a0);
    LDF(a4, pa, 8 * 2048);
    MM(1, 3, a1);
    LDF(a0, pa, 4 * 2048);
    MM(2, 0, a2); MM(2, 1, a2);
    LDF(a1, pa, 5 * 2048);
    MM(2, 2, a2); MM(2, 3, a2);
    LDF(a2, pa, 6 * 2048);
    OUT(0);
    MM(3, 0, a3); MM(3, 1, a3); MM(3, 2, a3); MM(3, 3, a3);
    LDF(a3, pa, 7 * 2048);
    OUT(1);
    MM(4, 0, a0); MM(4, 1, a0); MM(4, 2, a0); MM(4, 3, a0);
    OUT(2);
    MM(5, 0, a1); MM(5, 1, a1); MM(5, 2, a1); MM(5, 3, a1);
    OUT(3);
    MM(6, 0, a2); MM(6, 1, a2); MM(6, 2, a2); MM(6, 3, a2);
    OUT(4);
    MM(7, 0, a3); MM(7, 1, a3); MM(7, 2, a3); MM(7, 3, a3);
    OUT(5);
    MM(8, 0, a4); MM(8, 1, a4); MM(8, 2, a4); MM(8, 3, a4);
    STAMP(33);
    OUT(6); OUT(7);
    if (!(wr == 1 && trow < 14)) OUT(8);
  }
}

__global__ void __launch_bounds__(512, 2) gemm_ag_kernel(const Params p) {
  __shared__ __attribute__((aligned(16))) unsigned char lds_raw[LDS_BYTES];
  LAS unsigned char* lds = (LAS unsigned char*)lds_raw;
  const int wv = __builtin_amdgcn_readfirstlane(threadIdx.x >> 6), lane = threadIdx.x & 63;
  if (blockIdx.x < NCOMM) comm_role(p, lds, blockIdx.x, wv, lane);
  else gemm_role(p, lds, blockIdx.x - NCOMM, wv, lane);
  if (threadIdx.x == 0)
    __hip_atomic_fetch_add((unsigned*)(p.ws[p.rank] + CTL_DONE), 1u, __ATOMIC_RELAXED, __HIP_MEMORY_SCOPE_AGENT);
}

static void make_schedule(int s, uint8_t* sc) {
  static const int nlate[16] = {0, 0, 1, 2, 2, 3, 4, 4, 5, 6, 6, 7, 8, 8, 8, 8};
  int r = 0;
  for (int kt = 0; kt < 3; ++kt)
    for (int q = 0; q < 16; ++q) {
      sc[r++] = (uint8_t)(kt * 32 + 16 + (q + 2 * s) % 16);
      sc[r++] = (uint8_t)(kt * 32 + (q + 2 * s) % 16);
    }
  double due[16]; int late[16], nl = 0;
  for (int c = 0; c < 16; ++c) {
    const int i = ((s - c * 3) % 8 + 8) % 8;
    if (i < nlate[c]) { due[nl] = (0.71 * c < 7.6 ? 0.71 * c : 7.6) - (nlate[c] - i) * 1.15; late[nl++] = c; }
    else sc[r++] = (uint8_t)(96 + 16 + c);
  }
  for (int q = 0; q < 16; ++q) sc[r++] = (uint8_t)(96 + (q + 2 * s) % 16);
  for (int k = 0; k < nl; ++k) {
    int best = -1;
    for (int j = 0; j < nl; ++j) if (late[j] >= 0 && (best < 0 || due[j] < due[best])) best = j;
    sc[r++] = (uint8_t)(96 + 16 + late[best]); late[best] = -1;
  }
}

}

extern "C" void kernel_launch(int rank, int nranks, ncclComm_t comm,
                              hipStream_t stream,
                              void* const* d_in, const int* in_sizes, int n_in,
                              void* d_out, int out_size,
                              const int* in_dtypes, int out_dtype,
                              void* const* d_out_all,
                              void* const* d_ws_all, size_t ws_bytes) {
  (void)comm; (void)in_dtypes; (void)out_dtype; (void)d_out_all;
  if (nranks != NR || n_in != 4 || (unsigned)rank >= (unsigned)NR || ws_bytes < WS_NEED || out_size != MN * MN ||
      in_sizes[0] != MN * KS || in_sizes[1] != MN * KS) {
    fprintf(stderr, "kernel_launch: unexpected problem shape\n");
    return;
  }
  Params p;
  p.x = (const uint8_t*)d_in[0]; p.w = (const uint8_t*)d_in[1];
  p.sx = (const float*)d_in[2]; p.sw = (const float*)d_in[3];
  p.out = (float*)d_out;
  for (int i = 0; i < NR; ++i) p.ws[i] = (uint8_t*)d_ws_all[i];
  p.rank = rank;
  p.pubmask = 0x00AA8080u;
  p.fine_start = 24;
  for (int s = 0; s < NR; ++s) {
    uint8_t sc[128];
    make_schedule(s, sc);
    for (int r = 0; r < 128; ++r) p.inv[s][sc[r]] = (uint8_t)r;
    if (s == rank)
      for (int i = 0; i < 32; ++i)
        p.sched[i] = sc[4 * i] | (sc[4 * i + 1] << 8) | (sc[4 * i + 2] << 16) | ((unsigned)sc[4 * i + 3] << 24);
  }
  hipLaunchKernelGGL(gemm_ag_kernel, dim3(NWG), dim3(512), 0, stream, p);
  HIP_CHECK(hipGetLastError());
}
```

```cpp
#include <hip/hip_runtime.h>
#include <rccl/rccl.h>
#include <cstdio>
#include <cstdint>

#define HIP_CHECK(expr) do { hipError_t _e = (expr); if (_e != hipSuccess) { \
    fprintf(stderr, "HIP error %s at %s:%d\n", hipGetErrorString(_e), __FILE__, __LINE__); \
    return; } } while (0)

#define COMM_DRAIN 0
#define DMA_POL "sc1"
#define XMAP 1
#define WINM1 4
#define POLL_POL "sc0 sc1"
#define POLL_SCOPE __HIP_MEMORY_SCOPE_SYSTEM
#define POLL_SLEEP 2
#define OUT_AUX 16

namespace {

#define LAS __attribute__((address_space(3)))
typedef unsigned u32x4 __attribute__((ext_vector_type(4)));
typedef int i32x2 __attribute__((ext_vector_type(2)));
typedef int i32x4 __attribute__((ext_vector_type(4)));
typedef int i32x8 __attribute__((ext_vector_type(8)));
typedef float f32x4 __attribute__((ext_vector_type(4)));

constexpr int NR = 8;
constexpr int MN = 4096;
constexpr int KS = 512;
constexpr int NCOMM = 16;
constexpr int NGEMM = 240;
constexpr int NWG = NCOMM + NGEMM;
constexpr unsigned ENT_B = 1u << 20;
constexpr unsigned OP_B = 1u << 19;
constexpr unsigned SLOT_B = 32u << 20;
constexpr size_t DATA_OFF = 1u << 20;
constexpr size_t WS_NEED = DATA_OFF + 2ull * SLOT_B;
constexpr unsigned CTL_DONE = 0x0000, CTL_TMO = 0x0080, CTL_TICK = 0x0200, CTL_LFLAG = 0x1000,
                   CTL_UFLAG = 0x8000, CTL_RFLAG = 0x10000, CTL_BYTES = 0x30000;
__host__ __device__ constexpr unsigned uflag_off(int s, int unit) { return CTL_UFLAG + (unsigned)(s * 32 + unit) * 128u; }
__host__ __device__ constexpr unsigned rflag_off(int s, int pos) { return CTL_RFLAG + (unsigned)(s * 128 + (pos >> 2)) * 128u + (unsigned)(pos & 3) * 4u; }
constexpr unsigned long long DEADLINE_TICKS = 100000000ull;

constexpr int A_SLOT = 18 * 2048, B_SLOT = 16 * 2048;
constexpr int LDS_A = 0, LDS_B = 2 * A_SLOT;
constexpr int LDS_TBL = LDS_B + 2 * B_SLOT;
constexpr int LDS_SNAP = LDS_TBL + 352 * 4;
constexpr int LDS_ORD = LDS_SNAP + 6144;
constexpr int LDS_CTL = LDS_ORD + 128;
constexpr int LDS_BYTES = LDS_CTL + 256;

struct Params {
  const uint8_t* x; const uint8_t* w; const float* sx; const float* sw; float* out;
  uint8_t* ws[NR];
  int rank;
  unsigned pubmask;
  int fine_start;
  unsigned sched[32];
  uint8_t inv[NR][128];
};

#define STAMP(i)
#define STAMPC(i)
#define STAMP_DECL
#define STAMP_FLUSH
#define TR(code)
#define TRACE_LDS 0
#define WAIT_V(n) asm volatile("s_waitcnt vmcnt(" #n ")" ::: "memory")
#define WAIT_L(n) asm volatile("s_waitcnt lgkmcnt(" #n ")" ::: "memory")
#define BAR do { asm volatile("" ::: "memory"); __builtin_amdgcn_s_barrier(); asm volatile("" ::: "memory"); } while (0)
#define SCHED __builtin_amdgcn_sched_barrier(0)

__device__ __forceinline__ unsigned tile_src_off(int type, int kt, int g, int wv, int lane) {
  const int row = wv * 8 + (lane >> 3), ch = (lane & 7) * 16;
  return type ? (unsigned)((kt * 128 + (g & 1) * 64 + row) * MN + (g >> 1) * 128 + ch)
              : (unsigned)((g * 64 + row) * KS + kt * 128 + ch);
}
__device__ __forceinline__ unsigned unit_dst_off(int type, int kt, int g, int wv) {
  const int u = type ? ((g >> 1) * 8 + wv) * 2 + (g & 1) : g * 8 + wv;
  return (unsigned)kt * ENT_B + (unsigned)type * OP_B + (unsigned)u * 1024u;
}
__device__ __forceinline__ u32x4 unit_read(int type, const LAS unsigned char* tile, int wv, int lane) {
  if (type) {
    const LAS unsigned* a = (const LAS unsigned*)(tile + (lane >> 4) * 2048 + (wv >> 2) * 64 + (lane & 15) * 4);
    const unsigned n = wv & 3, s1 = n | ((n + 4u) << 8);
    u32x4 r;
#pragma unroll
    for (int i = 0; i < 4; ++i) {
      const unsigned lo = __builtin_amdgcn_perm(a[(4 * i + 1) * 32], a[(4 * i) * 32], s1);
      const unsigned hi = __builtin_amdgcn_perm(a[(4 * i + 3) * 32], a[(4 * i + 2) * 32], s1);
      r[i] = __builtin_amdgcn_perm(hi, lo, 0x05040100u);
    }
    return r;
  }
  return *(const LAS u32x4*)(tile + ((wv >> 1) * 16 + (lane & 15)) * 128 + (wv & 1) * 64 + (lane >> 4) * 16);
}

#define TLOAD "s_mov_b32 m0, %[lm]\n\ts_nop 3\n\tbuffer_load_dwordx4 %[lo], %[rl], 0 offen lds\n\t"
#define TLOAD_IN [lm] "s"(lm), [lo] "v"(lo), [rl] "s"(rl)
#define STR2(x) #x
#define STR(x) STR2(x)
#define PUSH1(j) asm volatile("s_waitcnt vmcnt(" STR(WINM1) ")\n\tbuffer_store_dwordx4 %0, %1, %2, 0 offen sc0 sc1" \
                              :: "v"(cur), "v"(so), "s"(rs[j]) : "memory")
#define DRAW(vnew) asm volatile("s_mov_b64 exec, 1\n\tglobal_atomic_add " vnew ", %0, %1, %2 sc0\n\ts_mov_b64 exec, -1" \
                                :: "v"(ao), "v"(one), "s"(ctl) : "memory", "v254", "v255")
#define GOT(vold) asm volatile("v_mov_b32 %0, " vold : "=v"(aret) :: "memory", "v254", "v255")

__device__ __forceinline__ void comm_role(const Params& p, LAS unsigned char* lds, int wg, int wv, int lane) {
  STAMP_DECL; STAMP(0);
  LAS unsigned* tick = (LAS unsigned*)(lds + LDS_CTL);
  uint8_t* const ctl = p.ws[p.rank];
  const unsigned ldsw = (unsigned)(uintptr_t)lds + wv * 1024;
  auto code_of = [&](int pos) -> int { return (int)((p.sched[(pos >> 4) & 31] >> (((pos >> 2) & 3) * 8)) & 127u); };
#define TILE_ARGS(pos, slot) const int _c = code_of(pos); \
    const __amdgpu_buffer_rsrc_t rl = __builtin_amdgcn_make_buffer_rsrc((void*)((_c & 16) ? p.w : p.x), 0, MN * KS, 0x00020000); \
    const unsigned lo = tile_src_off((_c >> 4) & 1, _c >> 5, (_c & 15) * 4 + ((pos) & 3), wv, lane), lm = ldsw + ((slot) & 3) * 8192
  int pm1 = 0, p0 = wg, p1 = 16 + wg, p2 = 32 + wg;
  { TILE_ARGS(p0, 0); asm volatile(TLOAD :: TLOAD_IN : "memory"); }
  { TILE_ARGS(p1, 1); asm volatile(TLOAD :: TLOAD_IN : "memory"); }
  const unsigned call = __builtin_amdgcn_readfirstlane(
      __hip_atomic_load((const unsigned*)(ctl + CTL_DONE), __ATOMIC_RELAXED, __HIP_MEMORY_SCOPE_AGENT)) / NWG + 1u;
  __amdgpu_buffer_rsrc_t rs[7];
#pragma unroll
  for (int j = 0; j < 7; ++j) {
    const int peer = (p.rank + 1 + (j + wg + wv) % 7) & (NR - 1);
    rs[j] = __builtin_amdgcn_make_buffer_rsrc(p.ws[peer] + DATA_OFF + (size_t)(call & 1u) * SLOT_B + (size_t)p.rank * 4 * ENT_B,
                                              0, 4 * ENT_B, 0x00020000);
  }
  uint8_t* const pctl = p.ws[(p.rank + 1 + lane % 7) & (NR - 1)];
  const unsigned ao = CTL_TICK + (call & 1u) * 128u, one = 1u;
  WAIT_V(0);
  BAR;
  if (wg == 0 && threadIdx.x == 64)
    __hip_atomic_store((unsigned*)(ctl + CTL_TICK + ((call & 1u) ^ 1u) * 128u), 0u, __ATOMIC_RELAXED, __HIP_MEMORY_SCOPE_AGENT);
  unsigned open = p.pubmask;
  STAMP(1);
  for (int r = 0;; ++r) {
    u32x4 cur; unsigned so = 0;
    if (p0 < 512) {
      const int c = code_of(p0);
      cur = unit_read((c >> 4) & 1, lds + (r & 3) * 8192, wv, lane);
      so = unit_dst_off((c >> 4) & 1, c >> 5, (c & 15) * 4 + (p0 & 3), wv) + lane * 16;
#pragma unroll
      for (int j = 0; j < WINM1; ++j) PUSH1(j);
      asm volatile("s_waitcnt vmcnt(" STR(WINM1) ")" ::: "memory");
      if (wv == 0) {
        unsigned aret;
        if (r & 1) GOT("v254"); else GOT("v255");
        if (lane == 0) tick[r & 1] = aret;
        WAIT_L(0);
      }
    } else {
      WAIT_V(0);
    }
    BAR;
    STAMP(2 + (r < 34 ? r : 34));
    if (r > 0) {
      if ((pm1 >> 4) >= p.fine_start) {
        if (wv == 1 + (r & 3) && lane < 7)
          __hip_atomic_store((unsigned*)(pctl + rflag_off(p.rank, pm1)), call * 16u, __ATOMIC_RELAXED, __HIP_MEMORY_SCOPE_SYSTEM);
      }
      unsigned done = (p0 >> 4) >= 32 ? open : open & ((1u << (p0 >> 4)) - 1u);
      open &= ~done;
      for (; done; done &= done - 1u)
        if (wv == 1 + (r & 3) && lane < 7)
          __hip_atomic_fetch_add((unsigned*)(pctl + uflag_off(p.rank, __builtin_ctz(done))), 1u, __ATOMIC_RELAXED, __HIP_MEMORY_SCOPE_SYSTEM);
    }
    if (p0 >= 512) break;
    if (r > 0) p2 = p1 < 512 ? 48 + __builtin_amdgcn_readfirstlane(tick[r & 1]) : 512;
#pragma unroll
    for (int j = WINM1; j < 7; ++j) PUSH1(j);
    { const int pn = p2 < 512 ? p2 : 511; TILE_ARGS(pn, r + 2); asm volatile(TLOAD :: TLOAD_IN : "memory"); }
    if (wv == 0) { if (r & 1) DRAW("v255"); else DRAW("v254"); }
    pm1 = p0; p0 = p1; p1 = p2;
  }
  WAIT_V(0);
  STAMP(37); STAMP_FLUSH;
}

#define DMA2(m0v, soff) asm volatile("s_mov_b32 m0, %0\n\ts_nop 0\n\t" \
    "buffer_load_dwordx4 %1, %2, %3 offen " DMA_POL " lds\n\t" \
    "buffer_load_dwordx4 %1, %2, %3 offen offset:1024 " DMA_POL " lds" \
    :: "s"(m0v), "v"(voff), "s"(rdata), "s"(soff) : "memory")
#define DMA3(m0v, soff, m0h, soffh) asm volatile("s_mov_b32 m0, %0\n\ts_nop 0\n\t" \
    "buffer_load_dwordx4 %1, %2, %3 offen " DMA_POL " lds\n\t" \
    "buffer_load_dwordx4 %1, %2, %3 offen offset:1024 " DMA_POL " lds\n\t" \
    "s_mov_b32 m0, %4\n\ts_mov_b32 exec_hi, 0\n\t" \
    "buffer_load_dwordx4 %1, %2, %5 offen " DMA_POL " lds\n\t" \
    "s_mov_b32 exec_hi, -1" \
    :: "s"(m0v), "v"(voff), "s"(rdata), "s"(soff), "s"(m0h), "s"(soffh) : "memory")

#define DMA1(m0v, soff) asm volatile("s_mov_b32 m0, %0\n\ts_nop 0\n\tbuffer_load_dwordx4 %1, %2, %3 offen " DMA_POL " lds" \
    :: "s"(m0v), "v"(voff), "s"(rdata), "s"(soff) : "memory")
#define DMA1B(soff) asm volatile("buffer_load_dwordx4 %0, %1, %2 offen offset:1024 " DMA_POL " lds"     \
    :: "v"(voff), "s"(rdata), "s"(soff) : "memory")
#define DMA1H(m0v, soff) asm volatile("s_mov_b32 m0, %0\n\ts_mov_b32 exec_hi, 0\n\tbuffer_load_dwordx4 %1, %2, %3 offen " DMA_POL " lds\n\t" \
    "s_mov_b32 exec_hi, -1" :: "s"(m0v), "v"(voff), "s"(rdata), "s"(soff) : "memory")
#define PIN(x) asm volatile("" : "+s"(x))

__device__ __forceinline__ int ready_from(unsigned long long okmask, int base) {
  const int r = base + (int)((unsigned)__builtin_ctzll(~okmask | (1ull << 63)) * 57u >> 9);
  return r < 32 ? r : 32;
}

__device__ __forceinline__ void gemm_role(const Params& p, LAS unsigned char* lds, int g, int wv, int lane) {
  const int lin = (g & 7) * 30 + (g >> 3), band = lin / 80, cb = (lin % 80) / 5;
  const int tcol = (band & 1) ? 15 - cb : cb, trow = band * 5 + lin % 5;
  uint8_t* const ctl = p.ws[p.rank];
  STAMP_DECL; STAMP(0);
#pragma unroll
  for (int i = 0; i < 3; ++i) {
    const int gid = g + i * NGEMM, type = (gid >> 8) & 1, kt = (gid >> 6) & 3;
    if (gid < 512) {
      const __amdgpu_buffer_rsrc_t rl = __builtin_amdgcn_make_buffer_rsrc((void*)(type ? p.w : p.x), 0, MN * KS, 0x00020000);
      const unsigned lo = tile_src_off(type, kt, gid & 63, wv, lane), lm = (unsigned)(uintptr_t)lds + i * 8192 + wv * 1024;
      asm volatile(TLOAD :: TLOAD_IN : "memory");
    }
  }
  const unsigned call = __builtin_amdgcn_readfirstlane(
      __hip_atomic_load((const unsigned*)(ctl + CTL_DONE), __ATOMIC_RELAXED, __HIP_MEMORY_SCOPE_AGENT)) / NWG + 1u;
  const __amdgpu_buffer_rsrc_t rdata =
      __builtin_amdgcn_make_buffer_rsrc(ctl + DATA_OFF + (size_t)(call & 1u) * SLOT_B, 0, SLOT_B, 0x00020000);
  WAIT_V(0);
  BAR;
#pragma unroll
  for (int i = 0; i < 3; ++i) {
    const int gid = g + i * NGEMM, type = (gid >> 8) & 1, kt = (gid >> 6) & 3;
    if (gid < 512)
      __builtin_amdgcn_raw_buffer_store_b128(unit_read(type, lds + i * 8192, wv, lane), rdata,
                                             (unsigned)p.rank * 4u * ENT_B + unit_dst_off(type, kt, gid & 63, wv) + lane * 16, 0, 17);
  }
  LAS unsigned* ord = (LAS unsigned*)(lds + LDS_ORD);
  LAS unsigned* tbl = (LAS unsigned*)(lds + LDS_TBL);
  LAS unsigned* bc = (LAS unsigned*)(lds + LDS_CTL);
  const int xg0 = (17 * trow) >> 2, wg0 = 4 * tcol;
  auto pub_pos = [&](int s, int kt, int i) -> int {
    const int type = i >= 5, gg = type ? wg0 + i - 5 : xg0 + i;
    const int pos = p.inv[s][kt * 32 + type * 16 + (gg >> 2)] * 4 + (gg & 3), r = pos >> 4;
    return r >= p.fine_start ? pos : (r + __builtin_ctz(p.pubmask >> r)) * 16 + 15;
  };
  if (wv == 0) {
    const int e = lane & 31, s = e >> 2, kt = e & 3;
    int key = kt;
    if (s != p.rank) {
      int m = 0;
      for (int i = 0; i < 9; ++i) { const int r = pub_pos(s, kt, i); m = r > m ? r : m; }
      key = 4 + m * 32 + kt * 8 + ((s - p.rank + g) & 7);
    }
    int rk = 0;
    for (int l = 0; l < 32; ++l) rk += __builtin_amdgcn_readlane(key, l) < key;
    if (lane < 32) ord[rk] = e;
  }
  WAIT_V(0);
  __syncthreads();
  if (threadIdx.x == 0)
    __hip_atomic_store((unsigned*)(ctl + CTL_LFLAG) + g, call * 16u, __ATOMIC_RELAXED, __HIP_MEMORY_SCOPE_SYSTEM);
  if (threadIdx.x < 352) {
    int t = threadIdx.x / 9; const int i = threadIdx.x - t * 9; t = t < 32 ? t : 31;
    const int e = ord[t], s = e >> 2, kt = e & 3, type = i >= 5, gg = type ? wg0 + i - 5 : xg0 + i;
    unsigned off;
    if (s == p.rank) off = CTL_LFLAG + 4u * (unsigned)((type * 256 + kt * 64 + gg) % NGEMM);
    else { const int pp = pub_pos(s, kt, i);
           off = (pp >> 4) >= p.fine_start ? rflag_off(s, pp) : uflag_off(s, pp >> 4); }
    tbl[threadIdx.x] = off;
  }
  const int ordv = ord[lane & 31];
  ((LAS unsigned long long*)(lds + LDS_SNAP))[threadIdx.x] = 0ull;
  ((LAS unsigned*)(lds + LDS_SNAP + 4096))[threadIdx.x] = 0u;
  int scale_bits = __builtin_amdgcn_readfirstlane(__builtin_bit_cast(int, p.sx[0] * p.sw[0]));
  asm volatile("" : "+s"(scale_bits));
  WAIT_V(0);
  __syncthreads();

  STAMP(1);
  const int wr = wv >> 2, wc = wv & 3;
  const unsigned voff = lane * 16;
  const unsigned ldsb = (unsigned)(uintptr_t)lds;
  const unsigned a0o = (9 * wr + wc) * 2048, a8o = (9 * wr + 8) * 2048 + wc * 512, b0o = (4 * (wv >> 1) + (wv & 1)) * 2048;
  const unsigned gA = 17u * trow * 2048u, gB = OP_B + tcol * 32768u;
  const bool has9 = !(wr == 1 && trow < 14);
  const unsigned ga0 = gA + a0o, ga8 = has9 ? gA + a8o : 0x40000000u, gb0 = gB + b0o;
  const unsigned la0 = ldsb + LDS_A + a0o, la8 = ldsb + LDS_A + a8o, lb0 = ldsb + LDS_B + b0o;
#define ENT(t) ((unsigned)__builtin_amdgcn_readlane(ordv, (t)) * ENT_B)
#define ST_A03(eb, sl) DMA2(la0 + (sl) * A_SLOT, (eb) + ga0)
#define ST_A48(eb, sl) DMA3(la0 + 8192 + (sl) * A_SLOT, (eb) + ga0 + 8192, la8 + (sl) * A_SLOT, (eb) + ga8)
#define ST_B0(eb, sl) DMA2(lb0 + (sl) * B_SLOT, (eb) + gb0)
#define ST_B1(eb, sl) DMA2(lb0 + 4096 + (sl) * B_SLOT, (eb) + gb0 + 4096)
#define LST(x) x
#define LPOLL(x) x
#define ST_ALL(eb, sl) do { ST_B0(eb, sl); ST_B1(eb, sl); ST_A03(eb, sl); ST_A48(eb, sl); } while (0)
#define LDF(dst, base, off) do { const i32x4 _l = *(const LAS i32x4*)((base) + (off)), _h = *(const LAS i32x4*)((base) + (off) + 1024); \
    dst = __builtin_shufflevector(_l, _h, 0, 1, 2, 3, 4, 5, 6, 7); SCHED; } while (0)
#define MM(m, n, Af) do { acc[m][n] = __builtin_amdgcn_mfma_scale_f32_16x16x128_f8f6f4(Af, Bf[n], acc[m][n], 1, 1, 0, 0, 0, 0); SCHED; } while (0)

  f32x4 acc[9][4];
#pragma unroll
  for (int m = 0; m < 9; ++m)
#pragma unroll
    for (int n = 0; n < 4; ++n) acc[m][n] = (f32x4){0.f, 0.f, 0.f, 0.f};
  i32x8 a0, a1, a2, a3, a4, Bf[4];
  int t = 0, ready = 0, nrestart = 0;
  unsigned dead = 0;
  const unsigned thr = call * 16u;
  constexpr unsigned OOB = 0x40000000u;
  const unsigned pollo = tbl[threadIdx.x < 352 ? threadIdx.x : 351];
  const __amdgpu_buffer_rsrc_t rctl = __builtin_amdgcn_make_buffer_rsrc(ctl, 0, CTL_BYTES, 0x00020000);
  const __amdgpu_buffer_rsrc_t rout = __builtin_amdgcn_make_buffer_rsrc(p.out, 0, MN * MN * 4, 0x00020000);
  const LAS unsigned char* const pa0 = lds + LDS_A + wr * (9 * 2048) + lane * 16;
  const LAS unsigned char* const pb0 = lds + LDS_B + wc * (4 * 2048) + lane * 16;
#define POLL(buf) do { const unsigned _po = (unsigned)(ready * 9 + 62 - wv * 64) < 126u && ready < 32 ? 0u : OOB; \
    asm volatile("s_mov_b32 m0, %0\n\ts_nop 0\n\tbuffer_load_dword %1, %2, %3 offen " POLL_POL " lds" \
                 :: "s"(ldsb + LDS_SNAP + (buf) * 2048 + wv * 256), "v"(pollo), "s"(rctl), "s"(_po) : "memory"); } while (0)
#define SNAP_AT(buf) ((const LAS unsigned*)(lds + LDS_SNAP + ((buf) * 512 + ready * 9) * 4) + lane)
#define OKMASK(v) __builtin_amdgcn_ballot_w64((int)((v) - thr) >= 0)
#define STAGE_BAR BAR
#define STAGE_FROM(u) do { const unsigned _e0 = ENT(u); ST_ALL(_e0, (u) & 1); sn = ready > (u) + 1; \
    if (sn) { STAGE_BAR; const unsigned _e1 = ENT((u) + 1); ST_ALL(_e1, ((u) & 1) ^ 1); } \
    POLL(((u) & 1) ^ 1); } while (0)
  int sn = -1;
  for (;;) {
    if (sn < 0) {
      if (ready <= t) {
        if (wv == 0) {
          const unsigned* fp = (const unsigned*)(ctl + (lane == 63 ? CTL_TMO : tbl[t * 9 + lane]));
          unsigned* tmo = (unsigned*)(ctl + CTL_TMO);
          const unsigned long long t0 = __builtin_amdgcn_s_memrealtime();
          int r; unsigned dd = 0;
          for (;;) {
            const unsigned v = __hip_atomic_load(fp, __ATOMIC_RELAXED, POLL_SCOPE);
            r = ready_from(OKMASK(v), t);
            if (r > t) break;
            if ((unsigned)__builtin_amdgcn_readlane(v, 63) == call || __builtin_amdgcn_s_memrealtime() - t0 > DEADLINE_TICKS) {
              if (lane == 0) __hip_atomic_store(tmo, call, __ATOMIC_RELAXED, __HIP_MEMORY_SCOPE_AGENT);
              r = 32; dd = 64; break;
            }
            __builtin_amdgcn_s_sleep(POLL_SLEEP);
          }
          if (lane == 0) bc[nrestart & 63] = (unsigned)r | (dd << 8);
          asm volatile("s_waitcnt vmcnt(0) lgkmcnt(0)" ::: "memory");
        }
        BAR;
        const unsigned v = __builtin_amdgcn_readfirstlane(bc[nrestart & 63]);
        ready = v & 63; dead |= v >> 8; ++nrestart;
      }
      TR(1);
      STAGE_FROM(t);
    }
    int s1 = sn;
    sn = -1;
    TR(2);
    if (s1) WAIT_V(13); else WAIT_V(4);
    BAR;
    TR(3);
    { const LAS unsigned char* pa = pa0 + (t & 1) * A_SLOT; const LAS unsigned char* pb = pb0 + (t & 1) * B_SLOT;
      LDF(Bf[0], pb, 0); LDF(a0, pa, 0); LDF(a1, pa, 2048); LDF(Bf[1], pb, 2048); LDF(Bf[2], pb, 4096); LDF(a2, pa, 4096); LDF(Bf[3], pb, 6144); }
    if (t == 31) break;
    unsigned en2 = ENT((t + 2) & 31);
    int first = 1;
    for (;;) {
      const int sl = t & 1;
      const LAS unsigned char* pa = pa0 + sl * A_SLOT;
      LDF(a3, pa, 3 * 2048);
      MM(0, 0, a0);
      const int rich = (ready > t + 2 || ready == 32 || (first & s1)) ? 1 : 0;
      const LAS unsigned* psn = SNAP_AT(sl ^ rich ^ 1);
      asm volatile("" : "+v"(psn)); SCHED;
      MM(1, 0, a1);
      const LAS unsigned char* pan = pa0 + (sl ^ 1) * A_SLOT; const LAS unsigned char* pbn = pb0 + (sl ^ 1) * B_SLOT;
      asm volatile("" : "+v"(pan), "+v"(pbn)); SCHED;
      MM(0, 1, a0);
      unsigned mB = lb0 + sl * B_SLOT, mA = la0 + sl * A_SLOT;
      PIN(mB); PIN(mA); SCHED;
      MM(1, 1, a1);
      unsigned mA8 = la8 + sl * A_SLOT, mP = ldsb + LDS_SNAP + sl * 2048 + wv * 256;
      PIN(mA8); PIN(mP); SCHED;
      MM(0, 2, a0); MM(1, 2, a1);
      TR(4);
      WAIT_V(10); if (!rich) { if (first) WAIT_V(0); else WAIT_V(3); }
      first = 0;
      WAIT_L(0); BAR; SCHED;
      TR(5);
      const unsigned snap = *psn;
      SCHED;
      MM(0, 3, a0);
      LDF(a4, pa, 6 * 2048);
      MM(1, 3, a1);
      LDF(a0, pa, 4 * 2048);
      MM(2, 0, a2);
      unsigned long long okm = OKMASK(snap);
      SCHED;
      MM(2, 1, a2);
      LDF(a1, pa, 5 * 2048);
      unsigned fz = (unsigned)__builtin_ctzll(~okm | (1ull << 63)) * 57u;
      PIN(fz); SCHED;
      MM(2, 2, a2);
      ready += (int)(fz >> 9); ready = ready < 32 ? ready : 32;
      if (__builtin_expect(!s1, 0)) {
        if (ready > t + 1) { const unsigned e1 = ENT(t + 1); ST_ALL(e1, sl ^ 1); s1 = 1; }
        POLL(2);
      }
      SCHED;
      MM(2, 3, a2);
      int s2 = ready > t + 2 ? s1 : 0;
      unsigned e2 = s2 ? en2 : OOB;
      PIN(s2); PIN(e2);
      LDF(a2, pa, 8 * 2048);
      MM(3, 0, a3);
      LST(DMA1(mB, e2 + gb0)); SCHED;
      MM(3, 1, a3);
      LST(DMA1B(e2 + gb0)); SCHED;
      MM(3, 2, a3);
      LST(DMA1(mB + 4096, e2 + gb0 + 4096)); SCHED;
      MM(3, 3, a3);
      LDF(a3, pa, 7 * 2048);
      LST(DMA1B(e2 + gb0 + 4096)); SCHED;
      MM(4, 0, a0);
      LST(DMA1(mA, e2 + ga0)); SCHED;
      MM(4, 1, a0);
      LST(DMA1B(e2 + ga0)); SCHED;
      MM(4, 2, a0);
      en2 = ENT((t + 3) & 31);
      SCHED;
      MM(4, 3, a0);
      unsigned po = (unsigned)(ready * 9 + 62 - wv * 64) < 126u && ready < 32 ? 0u : OOB;
      PIN(po); SCHED;
      MM(5, 0, a1); MM(5, 1, a1);
      TR(6);
      WAIT_V(10); if (__builtin_expect(!s1, 0)) WAIT_V(0);
      WAIT_L(0); BAR; SCHED;
      TR(7);
      MM(5, 2, a1);
      LDF(a0, pan, 0);
      LPOLL(asm volatile("s_mov_b32 m0, %0\n\ts_nop 0\n\tbuffer_load_dword %1, %2, %3 offen " POLL_POL " lds"
                   :: "s"(mP), "v"(pollo), "s"(rctl), "s"(po) : "memory"));
      SCHED;
      MM(5, 3, a1);
      LDF(a1, pan, 2048);
      LST(DMA1(mA + 8192, e2 + ga0 + 8192)); SCHED;
      if (has9) { MM(8, 0, a2); MM(8, 1, a2); MM(8, 2, a2); MM(8, 3, a2); }
      LDF(a2, pan, 4096);
      LST(DMA1B(e2 + ga0 + 8192)); SCHED;
      MM(7, 0, a3);
      LST(DMA1H(mA8, e2 + ga8));
      if (__builtin_expect(!s1, 0)) {
        ready = ready_from(OKMASK(*SNAP_AT(2)), ready);
        if (ready > t + 1) STAGE_FROM(t + 1);
      }
      SCHED;
      MM(6, 0, a4);
      LDF(Bf[0], pbn, 0);
      MM(7, 1, a3); MM(6, 1, a4);
      LDF(Bf[1], pbn, 2048);
      MM(7, 2, a3); MM(6, 2, a4);
      LDF(Bf[2], pbn, 4096);
      MM(7, 3, a3); MM(6, 3, a4);
      LDF(Bf[3], pbn, 6144);
      STAMP(2 + t); if (t == 24) STAMPC(40); if (t == 29) STAMPC(41);
      TR(8);
      ++t;
      if (!s1 || t == 31) break;
      s1 = s2;
    }
    if (s1) break;
    if (sn < 0) {
      asm volatile("s_waitcnt vmcnt(0) lgkmcnt(0)" ::: "memory"); BAR;
      ready = ready_from(OKMASK(*SNAP_AT((t & 1) ^ 1)), ready);
      WAIT_L(0); BAR;
    }
  }
  {
    const float sc = __builtin_bit_cast(float, dead ? 0x7fc00000 : scale_bits);
    const unsigned ov = (unsigned)((lane >> 4) * 4 * MN + (lane & 15) * 4) * 4u;
    const unsigned os = (unsigned)((trow * 272 + wr * 144) * MN + tcol * 256 + wc * 64) * 4u;
#define OUT(m) do { f32x4 o[4]; \
      _Pragma("unroll") for (int j = 0; j < 4; ++j) { o[j][0] = acc[m][0][j] * sc; o[j][1] = acc[m][1][j] * sc; o[j][2] = acc[m][2][j] * sc; o[j][3] = acc[m][3][j] * sc; } \
      SCHED; \
      _Pragma("unroll") for (int j = 0; j < 4; ++j) \
        __builtin_amdgcn_raw_buffer_store_b128(__builtin_bit_cast(u32x4, o[j]), rout, ov, os + (unsigned)((m) * 16 + j) * (MN * 4u), OUT_AUX); \
      asm volatile("s_nop 1" ::: "memory");                     \
      SCHED; } while (0)
    const LAS unsigned char* pa = pa0 + A_SLOT;
    LDF(a3, pa, 3 * 2048);
    MM(0, 0, a0); MM(1, 0, a1); MM(0, 1, a0); MM(1, 1, a1); MM(0, 2, a0); MM(1, 2, a1);
    WAIT_V(0); BAR; SCHED;
    MM(0, 3, a0);
    LDF(a4, pa, 6 * 2048);
    MM(1, 3, a1);
    LDF(a0, pa, 4 * 2048);
    MM(2, 0, a2); MM(2, 1, a2);
    LDF(a1, pa, 5 * 2048);
    MM(2, 2, a2); MM(2, 3, a2);
    LDF(a2, pa, 8 * 2048);
    OUT(0);
    MM(3, 0, a3); MM(3, 1, a3); MM(3, 2, a3); MM(3, 3, a3);
    LDF(a3, pa, 7 * 2048);
    OUT(1);
    MM(4, 0, a0); MM(4, 1, a0); MM(4, 2, a0); MM(4, 3, a0);
    OUT(2);
    MM(5, 0, a1); MM(5, 1, a1); MM(5, 2, a1); MM(5, 3, a1);
    OUT(3);
    MM(6, 0, a4); MM(6, 1, a4); MM(6, 2, a4); MM(6, 3, a4);
    OUT(4);
    MM(7, 0, a3); MM(7, 1, a3); MM(7, 2, a3); MM(7, 3, a3);
    OUT(5);
    if (has9) { MM(8, 0, a2); MM(8, 1, a2); MM(8, 2, a2); MM(8, 3, a2); }
    STAMP(33);
    OUT(6); OUT(7);
    if (has9) OUT(8);
  }
}

__global__ void __launch_bounds__(512, 2) gemm_ag_kernel(const Params p) {
  __shared__ __attribute__((aligned(16))) unsigned char lds_raw[LDS_BYTES + TRACE_LDS];
  LAS unsigned char* lds = (LAS unsigned char*)lds_raw;
  const int wv = __builtin_amdgcn_readfirstlane(threadIdx.x >> 6), lane = threadIdx.x & 63;
  if (blockIdx.x < NCOMM) comm_role(p, lds, blockIdx.x, wv, lane);
  else gemm_role(p, lds, blockIdx.x - NCOMM, wv, lane);
  if (threadIdx.x == 0)
    __hip_atomic_fetch_add((unsigned*)(p.ws[p.rank] + CTL_DONE), 1u, __ATOMIC_RELAXED, __HIP_MEMORY_SCOPE_AGENT);
}

static void make_schedule(int s, uint8_t* sc) {
  static const int nlate[16] = {0, 0, 1, 2, 2, 3, 4, 4, 5, 6, 6, 7, 8, 8, 8, 8};
  int r = 0;
  for (int kt = 0; kt < 3; ++kt)
    for (int q = 0; q < 16; ++q) {
      sc[r++] = (uint8_t)(kt * 32 + 16 + (q + 2 * s) % 16);
      sc[r++] = (uint8_t)(kt * 32 + (q + 2 * s) % 16);
    }
  double due[16]; int late[16], nl = 0;
  for (int c = 0; c < 16; ++c) {
    const int k = (c * 5) % 16, i = ((s - k * 3) % 8 + 8) % 8;
    if (i < nlate[k]) { due[nl] = (0.71 * k < 7.6 ? 0.71 * k : 7.6) - (nlate[k] - i) * 1.15; late[nl++] = c; }
    else sc[r++] = (uint8_t)(96 + 16 + c);
  }
  for (int q = 0; q < 16; ++q) sc[r++] = (uint8_t)(96 + (q + 2 * s) % 16);
  for (int k = 0; k < nl; ++k) {
    int best = -1;
    for (int j = 0; j < nl; ++j) if (late[j] >= 0 && (best < 0 || due[j] < due[best])) best = j;
    sc[r++] = (uint8_t)(96 + 16 + late[best]); late[best] = -1;
  }
}

}

extern "C" void kernel_launch(int rank, int nranks, ncclComm_t comm,
                              hipStream_t stream,
                              void* const* d_in, const int* in_sizes, int n_in,
                              void* d_out, int out_size,
                              const int* in_dtypes, int out_dtype,
                              void* const* d_out_all,
                              void* const* d_ws_all, size_t ws_bytes) {
  (void)comm; (void)in_dtypes; (void)out_dtype; (void)d_out_all;
  if (nranks != NR || n_in != 4 || (unsigned)rank >= (unsigned)NR || ws_bytes < WS_NEED || out_size != MN * MN ||
      in_sizes[0] != MN * KS || in_sizes[1] != MN * KS) {
    fprintf(stderr, "kernel_launch: unexpected problem shape\n");
    return;
  }
  Params p;
  p.x = (const uint8_t*)d_in[0]; p.w = (const uint8_t*)d_in[1];
  p.sx = (const float*)d_in[2]; p.sw = (const float*)d_in[3];
  p.out = (float*)d_out;
  for (int i = 0; i < NR; ++i) p.ws[i] = (uint8_t*)d_ws_all[i];
  p.rank = rank;
  p.pubmask = 0x00AA8080u;
  p.fine_start = 24;
  for (int s = 0; s < NR; ++s) {
    uint8_t sc[128];
    make_schedule(s, sc);
    for (int r = 0; r < 128; ++r) p.inv[s][sc[r]] = (uint8_t)r;
    if (s == rank)
      for (int i = 0; i < 32; ++i)
        p.sched[i] = sc[4 * i] | (sc[4 * i + 1] << 8) | (sc[4 * i + 2] << 16) | ((unsigned)sc[4 * i + 3] << 24);
  }
  hipLaunchKernelGGL(gemm_ag_kernel, dim3(NWG), dim3(512), 0, stream, p);
  HIP_CHECK(hipGetLastError());
}
```

```cpp
#include <hip/hip_runtime.h>
#include <rccl/rccl.h>
#include <cstdio>
#include <cstdint>

#define HIP_CHECK(expr) do { hipError_t _e = (expr); if (_e != hipSuccess) { \
    fprintf(stderr, "HIP error %s at %s:%d\n", hipGetErrorString(_e), __FILE__, __LINE__); \
    return; } } while (0)

#define COMM_DRAIN 0
#define DMA_POL "sc1"
#define XMAP 1
#define WINM1 4
#define POLL_POL "sc0 sc1"
#define POLL_SCOPE __HIP_MEMORY_SCOPE_SYSTEM
#define SHARE_WIN 6
#define POLL_SLEEP 2
#define OUT_AUX 16

namespace {

#define LAS __attribute__((address_space(3)))
#define GAS __attribute__((address_space(1)))
typedef unsigned u32x4 __attribute__((ext_vector_type(4)));
typedef int i32x2 __attribute__((ext_vector_type(2)));
typedef int i32x4 __attribute__((ext_vector_type(4)));
typedef int i32x8 __attribute__((ext_vector_type(8)));
typedef float f32x4 __attribute__((ext_vector_type(4)));

constexpr int NR = 8;
constexpr int MN = 4096;
constexpr int KS = 512;
constexpr int NCOMM = 16;
constexpr int NGEMM = 240;
constexpr int NWG = NCOMM + NGEMM;
constexpr unsigned ENT_B = 1u << 20;
constexpr unsigned OP_B = 1u << 19;
constexpr unsigned SLOT_B = 32u << 20;
constexpr size_t DATA_OFF = 1u << 20;
constexpr size_t WS_NEED = DATA_OFF + 2ull * SLOT_B;
constexpr unsigned CTL_DONE = 0x0000, CTL_TMO = 0x0080, CTL_TICK = 0x0200, CTL_SLICE = 0x0400, CTL_LFLAG = 0x1000,
                   CTL_UFLAG = 0x8000, CTL_RFLAG = 0x10000, CTL_BYTES = 0x30000;
__host__ __device__ constexpr unsigned uflag_off(int s, int unit) { return CTL_UFLAG + (unsigned)(s * 32 + unit) * 128u; }
__host__ __device__ constexpr unsigned rflag_off(int s, int pos) { return CTL_RFLAG + (unsigned)(s * 128 + (pos >> 2)) * 128u + (unsigned)(pos & 3) * 4u; }
constexpr unsigned OOB = 0x40000000u;
constexpr unsigned long long DEADLINE_TICKS = 100000000ull;

constexpr int A_SLOT = 18 * 2048, B_SLOT = 16 * 2048;
constexpr int LDS_A = 0, LDS_B = 2 * A_SLOT;
constexpr int LDS_TBL = LDS_B + 2 * B_SLOT;
constexpr int LDS_SNAP = LDS_TBL + 352 * 4;
constexpr int LDS_ORD = LDS_SNAP + 6144;
constexpr int LDS_CTL = LDS_ORD + 128;
constexpr int LDS_BYTES = LDS_CTL + 256;

struct Params {
  const uint8_t* x; const uint8_t* w; const float* sx; const float* sw; float* out;
  uint8_t* ws[NR];
  float* outs[NR];
  int rank;
  unsigned pubmask;
  int fine_start;
  unsigned sched[32];
  uint8_t inv[NR][128];
};

#define STAMP(i)
#define STAMPC(i)
#define STAMP_DECL
#define STAMP_FLUSH
#define TR(code)
#define TRACE_LDS 0
#define WAIT_V(n) asm volatile("s_waitcnt vmcnt(" #n ")" ::: "memory")
#define WAIT_L(n) asm volatile("s_waitcnt lgkmcnt(" #n ")" ::: "memory")
#define BAR do { asm volatile("" ::: "memory"); __builtin_amdgcn_s_barrier(); asm volatile("" ::: "memory"); } while (0)
#define SCHED __builtin_amdgcn_sched_barrier(0)

__device__ __forceinline__ unsigned tile_src_off(int type, int kt, int g, int wv, int lane) {
  const int row = wv * 8 + (lane >> 3), ch = (lane & 7) * 16;
  return type ? (unsigned)((kt * 128 + (g & 1) * 64 + row) * MN + (g >> 1) * 128 + ch)
              : (unsigned)((g * 64 + row) * KS + kt * 128 + ch);
}
__device__ __forceinline__ unsigned unit_dst_off(int type, int kt, int g, int wv) {
  const int u = type ? ((g >> 1) * 8 + wv) * 2 + (g & 1) : g * 8 + wv;
  return (unsigned)kt * ENT_B + (unsigned)type * OP_B + (unsigned)u * 1024u;
}
__device__ __forceinline__ u32x4 unit_read(int type, const LAS unsigned char* tile, int wv, int lane) {
  if (type) {
    const LAS unsigned* a = (const LAS unsigned*)(tile + (lane >> 4) * 2048 + (wv >> 2) * 64 + (lane & 15) * 4);
    const unsigned n = wv & 3, s1 = n | ((n + 4u) << 8);
    u32x4 r;
#pragma unroll
    for (int i = 0; i < 4; ++i) {
      const unsigned lo = __builtin_amdgcn_perm(a[(4 * i + 1) * 32], a[(4 * i) * 32], s1);
      const unsigned hi = __builtin_amdgcn_perm(a[(4 * i + 3) * 32], a[(4 * i + 2) * 32], s1);
      r[i] = __builtin_amdgcn_perm(hi, lo, 0x05040100u);
    }
    return r;
  }
  return *(const LAS u32x4*)(tile + ((wv >> 1) * 16 + (lane & 15)) * 128 + (wv & 1) * 64 + (lane >> 4) * 16);
}

#define TLOAD "s_mov_b32 m0, %[lm]\n\ts_nop 3\n\tbuffer_load_dwordx4 %[lo], %[rl], 0 offen lds\n\t"
#define TLOAD_IN [lm] "s"(lm), [lo] "v"(lo), [rl] "s"(rl)
#define STR2(x) #x
#define STR(x) STR2(x)
#define PUSH1(j) asm volatile("s_waitcnt vmcnt(" STR(WINM1) ")\n\tbuffer_store_dwordx4 %0, %1, %2, %3 offen sc0 sc1" \
                              :: "v"(cur), "v"(so), "s"(rs[j]), "s"((skip >> j) & 1u ? OOB : 0u) : "memory")
#define DRAW(vnew) asm volatile("s_mov_b64 exec, 1\n\tglobal_atomic_add " vnew ", %0, %1, %2 sc0\n\ts_mov_b64 exec, -1" \
                                :: "v"(ao), "v"(one), "s"(ctl) : "memory", "v254", "v255")
#define GOT(vold) asm volatile("v_mov_b32 %0, " vold : "=v"(aret) :: "memory", "v254", "v255")

__device__ __forceinline__ void comm_role(const Params& p, LAS unsigned char* lds, int wg, int wv, int lane) {
  STAMP_DECL; STAMP(0);
  LAS unsigned* tick = (LAS unsigned*)(lds + LDS_CTL);
  uint8_t* const ctl = p.ws[p.rank];
  const unsigned ldsw = (unsigned)(uintptr_t)lds + wv * 1024;
  auto code_of = [&](int pos) -> int { return (int)((p.sched[(pos >> 4) & 31] >> (((pos >> 2) & 3) * 8)) & 127u); };
#define TILE_ARGS(pos, slot) const int _c = code_of(pos); \
    const __amdgpu_buffer_rsrc_t rl = __builtin_amdgcn_make_buffer_rsrc((void*)((_c & 16) ? p.w : p.x), 0, MN * KS, 0x00020000); \
    const unsigned lo = tile_src_off((_c >> 4) & 1, _c >> 5, (_c & 15) * 4 + ((pos) & 3), wv, lane), lm = ldsw + ((slot) & 3) * 8192
  int pm1 = 0, p0 = wg, p1 = 16 + wg, p2 = 32 + wg;
  { TILE_ARGS(p0, 0); asm volatile(TLOAD :: TLOAD_IN : "memory"); }
  { TILE_ARGS(p1, 1); asm volatile(TLOAD :: TLOAD_IN : "memory"); }
  const unsigned call = __builtin_amdgcn_readfirstlane(
      __hip_atomic_load((const unsigned*)(ctl + CTL_DONE), __ATOMIC_RELAXED, __HIP_MEMORY_SCOPE_AGENT)) / NWG + 1u;
  __amdgpu_buffer_rsrc_t rs[7];
#pragma unroll
  for (int j = 0; j < 7; ++j) {
    const int peer = (p.rank + 1 + (j + wg + wv) % 7) & (NR - 1);
    rs[j] = __builtin_amdgcn_make_buffer_rsrc(p.ws[peer] + DATA_OFF + (size_t)(call & 1u) * SLOT_B + (size_t)p.rank * 4 * ENT_B,
                                              0, 4 * ENT_B, 0x00020000);
  }
  uint8_t* const pctl = p.ws[(p.rank + 1 + lane % 7) & (NR - 1)];
  const unsigned ao = CTL_TICK + (call & 1u) * 128u, one = 1u;
  WAIT_V(0);
  BAR;
  if (wg == 0 && threadIdx.x == 64)
    __hip_atomic_store((unsigned*)(ctl + CTL_TICK + ((call & 1u) ^ 1u) * 128u), 0u, __ATOMIC_RELAXED, __HIP_MEMORY_SCOPE_AGENT);
  unsigned open = p.pubmask;
  STAMP(1);
  for (int r = 0;; ++r) {
    u32x4 cur; unsigned so = 0, skip = 0;
    if (p0 < 512) {
      const int c = code_of(p0);
      cur = unit_read((c >> 4) & 1, lds + (r & 3) * 8192, wv, lane);
      so = unit_dst_off((c >> 4) & 1, c >> 5, (c & 15) * 4 + (p0 & 3), wv) + lane * 16;
      if ((c & 31) == 15) {
        const int d = ((p0 & 3) * 2 + (wv >> 2) - p.rank - 1) & (NR - 1);
        skip = d == 7 ? 0x7fu : 0x7fu & ~(1u << ((d + 70 - wg - wv) % 7));
      }
#pragma unroll
      for (int j = 0; j < WINM1; ++j) PUSH1(j);
      asm volatile("s_waitcnt vmcnt(" STR(WINM1) ")" ::: "memory");
      if (wv == 0) {
        unsigned aret;
        if (r & 1) GOT("v254"); else GOT("v255");
        if (lane == 0) tick[r & 1] = aret;
        WAIT_L(0);
      }
    } else {
      WAIT_V(0);
    }
    BAR;
    STAMP(2 + (r < 34 ? r : 34));
    if (r > 0) {
      if ((pm1 >> 4) >= p.fine_start) {
        if (wv == 1 + (r & 3) && lane < 7)
          __hip_atomic_store((unsigned*)(pctl + rflag_off(p.rank, pm1)), call * 16u, __ATOMIC_RELAXED, __HIP_MEMORY_SCOPE_SYSTEM);
      }
      unsigned done = (p0 >> 4) >= 32 ? open : open & ((1u << (p0 >> 4)) - 1u);
      open &= ~done;
      for (; done; done &= done - 1u)
        if (wv == 1 + (r & 3) && lane < 7)
          __hip_atomic_fetch_add((unsigned*)(pctl + uflag_off(p.rank, __builtin_ctz(done))), 1u, __ATOMIC_RELAXED, __HIP_MEMORY_SCOPE_SYSTEM);
    }
    if (p0 >= 512) break;
    if (r > 0) p2 = p1 < 512 ? 48 + __builtin_amdgcn_readfirstlane(tick[r & 1]) : 512;
#pragma unroll
    for (int j = WINM1; j < 7; ++j) PUSH1(j);
    { const int pn = p2 < 512 ? p2 : 511; TILE_ARGS(pn, r + 2); asm volatile(TLOAD :: TLOAD_IN : "memory"); }
    if (wv == 0) { if (r & 1) DRAW("v255"); else DRAW("v254"); }
    pm1 = p0; p0 = p1; p1 = p2;
  }
  WAIT_V(0);
  STAMP(37);
  if (wg == 0 && wv == 0) {
    const unsigned* cnt = (const unsigned*)(ctl + (lane & 1 ? CTL_TMO : CTL_SLICE));
    const unsigned long long t0 = __builtin_amdgcn_s_memrealtime();
    for (;;) {
      const unsigned v = __hip_atomic_load(cnt, __ATOMIC_RELAXED, __HIP_MEMORY_SCOPE_SYSTEM);
      if ((int)((unsigned)__builtin_amdgcn_readlane(v, 0) - call * (7u * 32u)) >= 0) break;
      if ((unsigned)__builtin_amdgcn_readlane(v, 1) == call || __builtin_amdgcn_s_memrealtime() - t0 > DEADLINE_TICKS) {
        if (lane == 0) {
          __hip_atomic_store((unsigned*)(ctl + CTL_TMO), call, __ATOMIC_RELAXED, __HIP_MEMORY_SCOPE_AGENT);
          p.out[0] = __builtin_nanf("");
        }
        break;
      }
      __builtin_amdgcn_s_sleep(4);
    }
  }
  STAMP(38); STAMP_FLUSH;
}

#define DMA2(m0v, soff) asm volatile("s_mov_b32 m0, %0\n\ts_nop 0\n\t" \
    "buffer_load_dwordx4 %1, %2, %3 offen " DMA_POL " lds\n\t" \
    "buffer_load_dwordx4 %1, %2, %3 offen offset:1024 " DMA_POL " lds" \
    :: "s"(m0v), "v"(voff), "s"(rdata), "s"(soff) : "memory")
#define DMA3(m0v, soff, m0h, soffh) asm volatile("s_mov_b32 m0, %0\n\ts_nop 0\n\t" \
    "buffer_load_dwordx4 %1, %2, %3 offen " DMA_POL " lds\n\t" \
    "buffer_load_dwordx4 %1, %2, %3 offen offset:1024 " DMA_POL " lds\n\t" \
    "s_mov_b32 m0, %4\n\ts_mov_b32 exec_hi, 0\n\t" \
    "buffer_load_dwordx4 %1, %2, %5 offen " DMA_POL " lds\n\t" \
    "s_mov_b32 exec_hi, -1" \
    :: "s"(m0v), "v"(voff), "s"(rdata), "s"(soff), "s"(m0h), "s"(soffh) : "memory")

#define DMA1(m0v, soff) asm volatile("s_mov_b32 m0, %0\n\ts_nop 0\n\tbuffer_load_dwordx4 %1, %2, %3 offen " DMA_POL " lds" \
    :: "s"(m0v), "v"(voff), "s"(rdata), "s"(soff) : "memory")
#define DMA1B(soff) asm volatile("buffer_load_dwordx4 %0, %1, %2 offen offset:1024 " DMA_POL " lds"     \
    :: "v"(voff), "s"(rdata), "s"(soff) : "memory")
#define DMA1H(m0v, soff) asm volatile("s_mov_b32 m0, %0\n\ts_mov_b32 exec_hi, 0\n\tbuffer_load_dwordx4 %1, %2, %3 offen " DMA_POL " lds\n\t" \
    "s_mov_b32 exec_hi, -1" :: "s"(m0v), "v"(voff), "s"(rdata), "s"(soff) : "memory")
#define PIN(x) asm volatile("" : "+s"(x))

__device__ __forceinline__ int ready_from(unsigned long long okmask, int base) {
  const int r = base + (int)((unsigned)__builtin_ctzll(~okmask | (1ull << 63)) * 57u >> 9);
  return r < 32 ? r : 32;
}

__device__ __forceinline__ void gemm_role(const Params& p, LAS unsigned char* lds, int g, int wv, int lane) {
  const int lin = (g & 7) * 30 + (g >> 3), band = lin / 80, cb = (lin % 80) / 5;
  const int tcol = (band & 1) ? 15 - cb : cb, trow = band * 5 + lin % 5;
  uint8_t* const ctl = p.ws[p.rank];
  STAMP_DECL; STAMP(0);
#pragma unroll
  for (int i = 0; i < 3; ++i) {
    const int gid = g + i * NGEMM, type = (gid >> 8) & 1, kt = (gid >> 6) & 3;
    if (gid < 512) {
      const __amdgpu_buffer_rsrc_t rl = __builtin_amdgcn_make_buffer_rsrc((void*)(type ? p.w : p.x), 0, MN * KS, 0x00020000);
      const unsigned lo = tile_src_off(type, kt, gid & 63, wv, lane), lm = (unsigned)(uintptr_t)lds + i * 8192 + wv * 1024;
      asm volatile(TLOAD :: TLOAD_IN : "memory");
    }
  }
  const unsigned call = __builtin_amdgcn_readfirstlane(
      __hip_atomic_load((const unsigned*)(ctl + CTL_DONE), __ATOMIC_RELAXED, __HIP_MEMORY_SCOPE_AGENT)) / NWG + 1u;
  const __amdgpu_buffer_rsrc_t rdata =
      __builtin_amdgcn_make_buffer_rsrc(ctl + DATA_OFF + (size_t)(call & 1u) * SLOT_B, 0, SLOT_B, 0x00020000);
  WAIT_V(0);
  BAR;
#pragma unroll
  for (int i = 0; i < 3; ++i) {
    const int gid = g + i * NGEMM, type = (gid >> 8) & 1, kt = (gid >> 6) & 3;
    if (gid < 512)
      __builtin_amdgcn_raw_buffer_store_b128(unit_read(type, lds + i * 8192, wv, lane), rdata,
                                             (unsigned)p.rank * 4u * ENT_B + unit_dst_off(type, kt, gid & 63, wv) + lane * 16, 0, 17);
  }
  LAS unsigned* ord = (LAS unsigned*)(lds + LDS_ORD);
  LAS unsigned* tbl = (LAS unsigned*)(lds + LDS_TBL);
  LAS unsigned* bc = (LAS unsigned*)(lds + LDS_CTL);
  const int hj = (trow / 5 + 2 - tcol % 3) % 3;
  const bool host = trow % 5 == 0 && hj < 2;
  auto group_of = [&](int i) -> int { return i < 4 ? 4 * trow + i : i < 8 ? 4 * tcol + i - 4 : host ? 60 + (p.rank >> 1) : 4 * trow; };
  auto pub_pos = [&](int s, int kt, int i) -> int {
    const int type = (i >> 2) & 1, gg = group_of(i);
    const int pos = p.inv[s][kt * 32 + type * 16 + (gg >> 2)] * 4 + (gg & 3), r = pos >> 4;
    return r >= p.fine_start ? pos : (r + __builtin_ctz(p.pubmask >> r)) * 16 + 15;
  };
  if (wv == 0) {
    const int e = lane & 31, s = e >> 2, kt = e & 3;
    int key = kt;
    if (s != p.rank) {
      int m = 0;
      for (int i = 0; i < 9; ++i) { const int r = pub_pos(s, kt, i); m = r > m ? r : m; }
      key = 4 + m * 32 + kt * 8 + ((s - p.rank + g) & 7);
    }
    int rk = 0;
    for (int l = 0; l < 32; ++l) rk += __builtin_amdgcn_readlane(key, l) < key;
    if (lane < 32) ord[rk] = e;
  }
  WAIT_V(0);
  __syncthreads();
  if (threadIdx.x == 0)
    __hip_atomic_store((unsigned*)(ctl + CTL_LFLAG) + g, call * 16u, __ATOMIC_RELAXED, __HIP_MEMORY_SCOPE_SYSTEM);
  if (threadIdx.x < 352) {
    int t = threadIdx.x / 9; const int i = threadIdx.x - t * 9; t = t < 32 ? t : 31;
    const int e = ord[t], s = e >> 2, kt = e & 3, type = (i >> 2) & 1, gg = group_of(i);
    unsigned off;
    if (s == p.rank) off = CTL_LFLAG + 4u * (unsigned)((type * 256 + kt * 64 + gg) % NGEMM);
    else { const int pp = pub_pos(s, kt, i);
           off = (pp >> 4) >= p.fine_start ? rflag_off(s, pp) : uflag_off(s, pp >> 4); }
    tbl[threadIdx.x] = off;
  }
  const int ordv = ord[lane & 31];
  ((LAS unsigned long long*)(lds + LDS_SNAP))[threadIdx.x] = 0ull;
  ((LAS unsigned*)(lds + LDS_SNAP + 4096))[threadIdx.x] = 0u;
  if (threadIdx.x == 0) bc[63] = 0u;
  if (threadIdx.x < 7) ((LAS unsigned long long*)(bc + 32))[threadIdx.x] = (unsigned long long)(p.ws[(p.rank + 1 + threadIdx.x) & (NR - 1)] + CTL_SLICE);
  int scale_bits = __builtin_amdgcn_readfirstlane(__builtin_bit_cast(int, p.sx[0] * p.sw[0]));
  asm volatile("" : "+s"(scale_bits));
  WAIT_V(0);
  __syncthreads();

  STAMP(1);
  const int wr = wv >> 2, wc = wv & 3;
  const unsigned voff = lane * 16;
  const unsigned ldsb = (unsigned)(uintptr_t)lds;
  const unsigned a0o = (9 * wr + wc) * 2048, a8o = (9 * wr + 8) * 2048 + wc * 512, b0o = (4 * (wv >> 1) + (wv & 1)) * 2048;
  const bool has9 = host && wr == 0;
  const unsigned ga0 = (unsigned)(16 * trow + 8 * wr + wc) * 2048u, gb0 = OP_B + tcol * 32768u + b0o;
  const unsigned ga8 = has9 ? (unsigned)(240 + 2 * p.rank + hj) * 2048u + wc * 512u : OOB;
  const unsigned la0 = ldsb + LDS_A + a0o, la8 = ldsb + LDS_A + a8o, lb0 = ldsb + LDS_B + b0o;
#define ENT(t) ((unsigned)__builtin_amdgcn_readlane(ordv, (t)) * ENT_B)
#define ST_A03(eb, sl) DMA2(la0 + (sl) * A_SLOT, (eb) + ga0)
#define ST_A48(eb, sl) DMA3(la0 + 8192 + (sl) * A_SLOT, (eb) + ga0 + 8192, la8 + (sl) * A_SLOT, (eb) + ga8)
#define ST_B0(eb, sl) DMA2(lb0 + (sl) * B_SLOT, (eb) + gb0)
#define ST_B1(eb, sl) DMA2(lb0 + 4096 + (sl) * B_SLOT, (eb) + gb0 + 4096)
#define LST(x) x
#define LPOLL(x) x
#define ST_ALL(eb, sl) do { ST_B0(eb, sl); ST_B1(eb, sl); ST_A03(eb, sl); ST_A48(eb, sl); } while (0)
#define LDF(dst, base, off) do { const i32x4 _l = *(const LAS i32x4*)((base) + (off)), _h = *(const LAS i32x4*)((base) + (off) + 1024); \
    dst = __builtin_shufflevector(_l, _h, 0, 1, 2, 3, 4, 5, 6, 7); SCHED; } while (0)
#define MM(m, n, Af) do { acc[m][n] = __builtin_amdgcn_mfma_scale_f32_16x16x128_f8f6f4(Af, Bf[n], acc[m][n], 1, 1, 0, 0, 0, 0); SCHED; } while (0)

  f32x4 acc[9][4];
#pragma unroll
  for (int m = 0; m < 9; ++m)
#pragma unroll
    for (int n = 0; n < 4; ++n) acc[m][n] = (f32x4){0.f, 0.f, 0.f, 0.f};
  i32x8 a0, a1, a2, a3, a4, Bf[4];
  int t = 0, ready = 0, nrestart = 0;
  unsigned dead = 0;
  const unsigned thr = call * 16u;
  const unsigned pollo = tbl[threadIdx.x < 352 ? threadIdx.x : 351];
  const __amdgpu_buffer_rsrc_t rctl = __builtin_amdgcn_make_buffer_rsrc(ctl, 0, CTL_BYTES, 0x00020000);
  const __amdgpu_buffer_rsrc_t rout = __builtin_amdgcn_make_buffer_rsrc(p.out, 0, MN * MN * 4, 0x00020000);
  const LAS unsigned char* const pa0 = lds + LDS_A + wr * (9 * 2048) + lane * 16;
  const LAS unsigned char* const pb0 = lds + LDS_B + wc * (4 * 2048) + lane * 16;
#define POLL(buf) do { const unsigned _po = (unsigned)(ready * 9 + 62 - wv * 64) < 126u && ready < 32 ? 0u : OOB; \
    asm volatile("s_mov_b32 m0, %0\n\ts_nop 0\n\tbuffer_load_dword %1, %2, %3 offen " POLL_POL " lds" \
                 :: "s"(ldsb + LDS_SNAP + (buf) * 2048 + wv * 256), "v"(pollo), "s"(rctl), "s"(_po) : "memory"); } while (0)
#define SNAP_AT(buf) ((const LAS unsigned*)(lds + LDS_SNAP + ((buf) * 512 + ready * 9) * 4) + lane)
#define OKMASK(v) __builtin_amdgcn_ballot_w64((int)((v) - thr) >= 0)
#define STAGE_BAR BAR
#define STAGE_FROM(u) do { const unsigned _e0 = ENT(u); ST_ALL(_e0, (u) & 1); sn = ready > (u) + 1; \
    if (sn) { STAGE_BAR; const unsigned _e1 = ENT((u) + 1); ST_ALL(_e1, ((u) & 1) ^ 1); } \
    POLL(((u) & 1) ^ 1); } while (0)
  int sn = -1;
  for (;;) {
    if (sn < 0) {
      if (ready <= t) {
        if (wv == 0) {
          const unsigned* fp = (const unsigned*)(ctl + (lane == 63 ? CTL_TMO : tbl[t * 9 + lane]));
          unsigned* tmo = (unsigned*)(ctl + CTL_TMO);
          const unsigned long long t0 = __builtin_amdgcn_s_memrealtime();
          int r; unsigned dd = 0;
          for (;;) {
            const unsigned v = __hip_atomic_load(fp, __ATOMIC_RELAXED, POLL_SCOPE);
            r = ready_from(OKMASK(v), t);
            if (r > t) break;
            if ((unsigned)__builtin_amdgcn_readlane(v, 63) == call || __builtin_amdgcn_s_memrealtime() - t0 > DEADLINE_TICKS) {
              if (lane == 0) __hip_atomic_store(tmo, call, __ATOMIC_RELAXED, __HIP_MEMORY_SCOPE_AGENT);
              r = 32; dd = 64; break;
            }
            __builtin_amdgcn_s_sleep(POLL_SLEEP);
          }
          if (lane == 0) bc[nrestart & 31] = (unsigned)r | (dd << 8);
          asm volatile("s_waitcnt vmcnt(0) lgkmcnt(0)" ::: "memory");
        }
        BAR;
        const unsigned v = __builtin_amdgcn_readfirstlane(bc[nrestart & 31]);
        ready = v & 63; dead |= v >> 8; ++nrestart;
      }
      TR(1);
      STAGE_FROM(t);
    }
    int s1 = sn;
    sn = -1;
    TR(2);
    if (s1) WAIT_V(13); else WAIT_V(4);
    BAR;
    TR(3);
    { const LAS unsigned char* pa = pa0 + (t & 1) * A_SLOT; const LAS unsigned char* pb = pb0 + (t & 1) * B_SLOT;
      LDF(Bf[0], pb, 0); LDF(a0, pa, 0); LDF(a1, pa, 2048); LDF(Bf[1], pb, 2048); LDF(Bf[2], pb, 4096); LDF(a2, pa, 4096); LDF(Bf[3], pb, 6144); }
    if (t == 31) break;
    unsigned en2 = ENT((t + 2) & 31);
    int first = 1;
    for (;;) {
      const int sl = t & 1;
      const LAS unsigned char* pa = pa0 + sl * A_SLOT;
      LDF(a3, pa, 3 * 2048);
      MM(0, 0, a0);
      const int rich = (ready > t + 2 || ready == 32 || (first & s1)) ? 1 : 0;
      const LAS unsigned* psn = SNAP_AT(sl ^ rich ^ 1);
      asm volatile("" : "+v"(psn)); SCHED;
      MM(1, 0, a1);
      const LAS unsigned char* pan = pa0 + (sl ^ 1) * A_SLOT; const LAS unsigned char* pbn = pb0 + (sl ^ 1) * B_SLOT;
      asm volatile("" : "+v"(pan), "+v"(pbn)); SCHED;
      MM(0, 1, a0);
      unsigned mB = lb0 + sl * B_SLOT, mA = la0 + sl * A_SLOT;
      PIN(mB); PIN(mA); SCHED;
      MM(1, 1, a1);
      unsigned mA8 = la8 + sl * A_SLOT, mP = ldsb + LDS_SNAP + sl * 2048 + wv * 256;
      PIN(mA8); PIN(mP); SCHED;
      MM(0, 2, a0); MM(1, 2, a1);
      TR(4);
      WAIT_V(10); if (!rich) { if (first) WAIT_V(0); else WAIT_V(3); }
      first = 0;
      WAIT_L(0); BAR; SCHED;
      TR(5);
      const unsigned snap = *psn;
      SCHED;
      MM(0, 3, a0);
      LDF(a4, pa, 6 * 2048);
      MM(1, 3, a1);
      LDF(a0, pa, 4 * 2048);
      MM(2, 0, a2);
      unsigned long long okm = OKMASK(snap);
      SCHED;
      MM(2, 1, a2);
      LDF(a1, pa, 5 * 2048);
      unsigned fz = (unsigned)__builtin_ctzll(~okm | (1ull << 63)) * 57u;
      PIN(fz); SCHED;
      MM(2, 2, a2);
      ready += (int)(fz >> 9); ready = ready < 32 ? ready : 32;
      if (__builtin_expect(!s1, 0)) {
        if (ready > t + 1) { const unsigned e1 = ENT(t + 1); ST_ALL(e1, sl ^ 1); s1 = 1; }
        POLL(2);
      }
      SCHED;
      MM(2, 3, a2);
      int s2 = ready > t + 2 ? s1 : 0;
      unsigned e2 = s2 ? en2 : OOB;
      PIN(s2); PIN(e2);
      LDF(a2, pa, 8 * 2048);
      MM(3, 0, a3);
      LST(DMA1(mB, e2 + gb0)); SCHED;
      MM(3, 1, a3);
      LST(DMA1B(e2 + gb0)); SCHED;
      MM(3, 2, a3);
      LST(DMA1(mB + 4096, e2 + gb0 + 4096)); SCHED;
      MM(3, 3, a3);
      LDF(a3, pa, 7 * 2048);
      LST(DMA1B(e2 + gb0 + 4096)); SCHED;
      MM(4, 0, a0);
      LST(DMA1(mA, e2 + ga0)); SCHED;
      MM(4, 1, a0);
      LST(DMA1B(e2 + ga0)); SCHED;
      MM(4, 2, a0);
      en2 = ENT((t + 3) & 31);
      SCHED;
      MM(4, 3, a0);
      unsigned po = (unsigned)(ready * 9 + 62 - wv * 64) < 126u && ready < 32 ? 0u : OOB;
      PIN(po); SCHED;
      MM(5, 0, a1); MM(5, 1, a1);
      TR(6);
      WAIT_V(10); if (__builtin_expect(!s1, 0)) WAIT_V(0);
      WAIT_L(0); BAR; SCHED;
      TR(7);
      MM(5, 2, a1);
      LDF(a0, pan, 0);
      LPOLL(asm volatile("s_mov_b32 m0, %0\n\ts_nop 0\n\tbuffer_load_dword %1, %2, %3 offen " POLL_POL " lds"
                   :: "s"(mP), "v"(pollo), "s"(rctl), "s"(po) : "memory"));
      SCHED;
      MM(5, 3, a1);
      LDF(a1, pan, 2048);
      LST(DMA1(mA + 8192, e2 + ga0 + 8192)); SCHED;
      if (has9) { MM(8, 0, a2); MM(8, 1, a2); MM(8, 2, a2); MM(8, 3, a2); }
      LDF(a2, pan, 4096);
      LST(DMA1B(e2 + ga0 + 8192)); SCHED;
      MM(7, 0, a3);
      LST(DMA1H(mA8, e2 + ga8));
      if (__builtin_expect(!s1, 0)) {
        ready = ready_from(OKMASK(*SNAP_AT(2)), ready);
        if (ready > t + 1) STAGE_FROM(t + 1);
      }
      SCHED;
      MM(6, 0, a4);
      LDF(Bf[0], pbn, 0);
      MM(7, 1, a3); MM(6, 1, a4);
      LDF(Bf[1], pbn, 2048);
      MM(7, 2, a3); MM(6, 2, a4);
      LDF(Bf[2], pbn, 4096);
      MM(7, 3, a3); MM(6, 3, a4);
      LDF(Bf[3], pbn, 6144);
      STAMP(2 + t); if (t == 24) STAMPC(40); if (t == 29) STAMPC(41);
      TR(8);
      ++t;
      if (!s1 || t == 31) break;
      s1 = s2;
    }
    if (s1) break;
    if (sn < 0) {
      asm volatile("s_waitcnt vmcnt(0) lgkmcnt(0)" ::: "memory"); BAR;
      ready = ready_from(OKMASK(*SNAP_AT((t & 1) ^ 1)), ready);
      WAIT_L(0); BAR;
    }
  }
  {
    const float sc = __builtin_bit_cast(float, dead ? 0x7fc00000 : scale_bits);
    const unsigned ov = (unsigned)((lane >> 4) * 4 * MN + (lane & 15) * 4) * 4u;
    const unsigned os = (unsigned)((trow * 256 + wr * 128) * MN + tcol * 256 + wc * 64) * 4u;
#define OUT(m) do { f32x4 o[4]; \
      asm volatile("" : "+v"(acc[m][0]), "+v"(acc[m][1]), "+v"(acc[m][2]), "+v"(acc[m][3])); \
      _Pragma("unroll") for (int j = 0; j < 4; ++j) { o[j][0] = acc[m][0][j] * sc; o[j][1] = acc[m][1][j] * sc; o[j][2] = acc[m][2][j] * sc; o[j][3] = acc[m][3][j] * sc; } \
      SCHED; \
      _Pragma("unroll") for (int j = 0; j < 4; ++j) \
        __builtin_amdgcn_raw_buffer_store_b128(__builtin_bit_cast(u32x4, o[j]), rout, ov, os + (unsigned)((m) * 16 + j) * (MN * 4u), OUT_AUX); \
      asm volatile("s_nop 1" ::: "memory");                     \
      SCHED; } while (0)
    const LAS unsigned char* pa = pa0 + A_SLOT;
    LDF(a3, pa, 3 * 2048);
    MM(0, 0, a0); MM(1, 0, a1); MM(0, 1, a0); MM(1, 1, a1); MM(0, 2, a0); MM(1, 2, a1);
    WAIT_V(0); BAR; SCHED;
    MM(0, 3, a0);
    LDF(a4, pa, 8 * 2048);
    MM(1, 3, a1);
    LDF(a0, pa, 4 * 2048);
    MM(2, 0, a2); MM(2, 1, a2);
    LDF(a1, pa, 5 * 2048);
    MM(2, 2, a2); MM(2, 3, a2);
    LDF(a2, pa, 6 * 2048);
    f32x4 o8[4];
    const unsigned os8 = (unsigned)((3840 + 32 * p.rank + 16 * hj) * MN + tcol * 256 + wc * 64) * 4u;
#define SHARE(d) do { if (has9) { \
      const __amdgpu_buffer_rsrc_t rd = __builtin_amdgcn_make_buffer_rsrc(p.outs[(p.rank + 1 + ((d) < 7 ? ((d) + tcol + wc) % 7 : 7)) & (NR - 1)], 0, MN * MN * 4, 0x00020000); \
      asm volatile("s_waitcnt vmcnt(" STR(SHARE_WIN) ")" ::: "memory"); \
      _Pragma("unroll") for (int j = 0; j < 4; ++j) \
        __builtin_amdgcn_raw_buffer_store_b128(__builtin_bit_cast(u32x4, o8[j]), rd, ov, os8 + (unsigned)j * (MN * 4u), 17); } \
      SCHED; } while (0)
    if (has9) { MM(8, 0, a4); MM(8, 1, a4); MM(8, 2, a4); MM(8, 3, a4); }
    OUT(0);
#pragma unroll
    for (int j = 0; j < 4; ++j) { o8[j][0] = acc[8][0][j] * sc; o8[j][1] = acc[8][1][j] * sc; o8[j][2] = acc[8][2][j] * sc; o8[j][3] = acc[8][3][j] * sc; }
    SCHED;
    SHARE(0);
    MM(3, 0, a3); MM(3, 1, a3); MM(3, 2, a3); MM(3, 3, a3);
    LDF(a3, pa, 7 * 2048);
    OUT(1); SHARE(1);
    MM(4, 0, a0); MM(4, 1, a0); MM(4, 2, a0); MM(4, 3, a0);
    OUT(2); SHARE(2);
    MM(5, 0, a1); MM(5, 1, a1); MM(5, 2, a1); MM(5, 3, a1);
    OUT(3); SHARE(3);
    MM(6, 0, a2); MM(6, 1, a2); MM(6, 2, a2); MM(6, 3, a2);
    OUT(4); SHARE(4);
    MM(7, 0, a3); MM(7, 1, a3); MM(7, 2, a3); MM(7, 3, a3);
    STAMP(33);
    OUT(5); SHARE(5); OUT(6); SHARE(6); OUT(7); SHARE(7);
    if (has9) {
      WAIT_V(8);
      unsigned old = 0;
      if (lane == 0) old = __hip_atomic_fetch_add(bc + 63, 1u, __ATOMIC_RELAXED, __HIP_MEMORY_SCOPE_WORKGROUP);
      if (__builtin_amdgcn_readfirstlane(old) == 3u && lane < 7)
        __hip_atomic_fetch_add((GAS unsigned*)((LAS unsigned long long*)(bc + 32))[lane], 1u, __ATOMIC_RELAXED, __HIP_MEMORY_SCOPE_SYSTEM);
    }
  }
}

__global__ void __launch_bounds__(512, 2) gemm_ag_kernel(const Params p) {
  __shared__ __attribute__((aligned(16))) unsigned char lds_raw[LDS_BYTES + TRACE_LDS];
  LAS unsigned char* lds = (LAS unsigned char*)lds_raw;
  const int wv = __builtin_amdgcn_readfirstlane(threadIdx.x >> 6), lane = threadIdx.x & 63;
  if (blockIdx.x < NCOMM) comm_role(p, lds, blockIdx.x, wv, lane);
  else gemm_role(p, lds, blockIdx.x - NCOMM, wv, lane);
  if (threadIdx.x == 0)
    __hip_atomic_fetch_add((unsigned*)(p.ws[p.rank] + CTL_DONE), 1u, __ATOMIC_RELAXED, __HIP_MEMORY_SCOPE_AGENT);
}

static void make_schedule(int s, uint8_t* sc) {
  static const int nlate[15] = {0, 0, 0, 2, 3, 3, 4, 5, 6, 6, 7, 8, 8, 8, 8};
  int r = 0;
  for (int kt = 0; kt < 3; ++kt)
    for (int q = 0; q < 16; ++q) {
      sc[r++] = (uint8_t)(kt * 32 + 16 + (q + 2 * s) % 16);
      sc[r++] = (uint8_t)(kt * 32 + (q + 2 * s) % 16);
    }
  double due[16]; int late[16], nl = 0;
  sc[r++] = (uint8_t)(96 + 15);
  for (int c = 0; c < 15; ++c) {
    const int k = (c % 5) * 3 + c / 5, i = ((s - k * 3) % 8 + 8) % 8;
    if (i < nlate[k]) { due[nl] = (0.75 * k < 7.6 ? 0.75 * k : 7.6) - (nlate[k] - i) * 1.15; late[nl++] = c; }
    else sc[r++] = (uint8_t)(96 + c);
  }
  for (int q = 0; q < 16; ++q) sc[r++] = (uint8_t)(96 + 16 + (q + 2 * s) % 16);
  for (int k = 0; k < nl; ++k) {
    int best = -1;
    for (int j = 0; j < nl; ++j) if (late[j] >= 0 && (best < 0 || due[j] < due[best])) best = j;
    sc[r++] = (uint8_t)(96 + late[best]); late[best] = -1;
  }
}

}

extern "C" void kernel_launch(int rank, int nranks, ncclComm_t comm,
                              hipStream_t stream,
                              void* const* d_in, const int* in_sizes, int n_in,
                              void* d_out, int out_size,
                              const int* in_dtypes, int out_dtype,
                              void* const* d_out_all,
                              void* const* d_ws_all, size_t ws_bytes) {
  (void)comm; (void)in_dtypes; (void)out_dtype;
  if (nranks != NR || n_in != 4 || (unsigned)rank >= (unsigned)NR || ws_bytes < WS_NEED || out_size != MN * MN ||
      in_sizes[0] != MN * KS || in_sizes[1] != MN * KS) {
    fprintf(stderr, "kernel_launch: unexpected problem shape\n");
    return;
  }
  Params p;
  p.x = (const uint8_t*)d_in[0]; p.w = (const uint8_t*)d_in[1];
  p.sx = (const float*)d_in[2]; p.sw = (const float*)d_in[3];
  p.out = (float*)d_out;
  for (int i = 0; i < NR; ++i) { p.ws[i] = (uint8_t*)d_ws_all[i]; p.outs[i] = (float*)d_out_all[i]; }
  p.rank = rank;
  p.pubmask = 0x00AA8080u;
  p.fine_start = 24;
  for (int s = 0; s < NR; ++s) {
    uint8_t sc[128];
    make_schedule(s, sc);
    for (int r = 0; r < 128; ++r) p.inv[s][sc[r]] = (uint8_t)r;
    if (s == rank)
      for (int i = 0; i < 32; ++i)
        p.sched[i] = sc[4 * i] | (sc[4 * i + 1] << 8) | (sc[4 * i + 2] << 16) | ((unsigned)sc[4 * i + 3] << 24);
  }
  hipLaunchKernelGGL(gemm_ag_kernel, dim3(NWG), dim3(512), 0, stream, p);
  HIP_CHECK(hipGetLastError());
}
```

```cpp
#include <hip/hip_runtime.h>
#include <rccl/rccl.h>
#include <cstdio>
#include <cstdint>

#define HIP_CHECK(expr) do { hipError_t _e = (expr); if (_e != hipSuccess) { \
    fprintf(stderr, "HIP error %s at %s:%d\n", hipGetErrorString(_e), __FILE__, __LINE__); \
    return; } } while (0)

#define DMA_POL "sc0 sc1"
#define WINM1 4
#define SHARE_WIN 6
#define OUT_AUX 16

namespace {

#define LAS __attribute__((address_space(3)))
#define GAS __attribute__((address_space(1)))
typedef unsigned u32x4 __attribute__((ext_vector_type(4)));
typedef int i32x4 __attribute__((ext_vector_type(4)));
typedef int i32x8 __attribute__((ext_vector_type(8)));
typedef float f32x4 __attribute__((ext_vector_type(4)));

constexpr int NR = 8;
constexpr int MN = 4096;
constexpr int KS = 512;
constexpr int NCOMM = 16;
constexpr int NGEMM = 240;
constexpr int NWG = NCOMM + NGEMM;
constexpr unsigned ENT_B = 1u << 20;
constexpr unsigned OP_B = 1u << 19;
constexpr unsigned SLOT_B = 32u << 20;
constexpr size_t DATA_OFF = 1u << 20;
constexpr size_t WS_NEED = DATA_OFF + 2ull * SLOT_B;
constexpr unsigned CTL_DONE = 0x0000, CTL_TMO = 0x0080, CTL_TICK = 0x0200, CTL_SLICE = 0x0400, CTL_LFLAG = 0x1000,
                   CTL_UFLAG = 0x8000, CTL_RFLAG = 0x10000, CTL_BYTES = 0x30000;
__host__ __device__ constexpr unsigned uflag_off(int s, int unit) { return CTL_UFLAG + (unsigned)(s * 32 + unit) * 128u; }
__host__ __device__ constexpr unsigned rflag_off(int s, int pos) { return CTL_RFLAG + (unsigned)(s * 128 + (pos >> 2)) * 128u + (unsigned)(pos & 3) * 4u; }
constexpr unsigned OOB = 0x40000000u;
constexpr unsigned long long DEADLINE_TICKS = 100000000ull;

constexpr int A_SLOT = 18 * 2048, B_SLOT = 16 * 2048;
constexpr int LDS_A = 0, LDS_B = 2 * A_SLOT;
constexpr int LDS_TBL = LDS_B + 2 * B_SLOT;
constexpr int LDS_SNAP = LDS_TBL + 352 * 4;
constexpr int LDS_ORD = LDS_SNAP + 6144;
constexpr int LDS_CTL = LDS_ORD + 128;
constexpr int LDS_BYTES = LDS_CTL + 256;

struct Params {
  const uint8_t* x; const uint8_t* w; const float* sx; const float* sw; float* out;
  uint8_t* ws[NR];
  float* outs[NR];
  int rank;
  unsigned pubmask;
  int fine_start;
  unsigned sched[32];
  uint8_t inv[NR][128];
};

#define COUNT_IN(ctl) __hip_atomic_fetch_add((unsigned*)((ctl) + CTL_DONE), 1u, __ATOMIC_RELAXED, __HIP_MEMORY_SCOPE_AGENT)
#define WAIT_V(n) asm volatile("s_waitcnt vmcnt(" #n ")" ::: "memory")
#define WAIT_L(n) asm volatile("s_waitcnt lgkmcnt(" #n ")" ::: "memory")
#define BAR do { asm volatile("" ::: "memory"); __builtin_amdgcn_s_barrier(); asm volatile("" ::: "memory"); } while (0)
#define SCHED __builtin_amdgcn_sched_barrier(0)

__device__ __forceinline__ unsigned tile_src_off(int type, int kt, int g, int wv, int lane) {
  const int row = wv * 8 + (lane >> 3), ch = (lane & 7) * 16;
  return type ? (unsigned)((kt * 128 + (g & 1) * 64 + row) * MN + (g >> 1) * 128 + ch)
              : (unsigned)((g * 64 + row) * KS + kt * 128 + ch);
}
__device__ __forceinline__ unsigned unit_dst_off(int type, int kt, int g, int wv) {
  const int u = type ? ((g >> 1) * 8 + wv) * 2 + (g & 1) : g * 8 + wv;
  return (unsigned)kt * ENT_B + (unsigned)type * OP_B + (unsigned)u * 1024u;
}
__device__ __forceinline__ u32x4 unit_read(int type, const LAS unsigned char* tile, int wv, int lane) {
  if (type) {
    const LAS unsigned* a = (const LAS unsigned*)(tile + (lane >> 4) * 2048 + (wv >> 2) * 64 + (lane & 15) * 4);
    const unsigned n = wv & 3, s1 = n | ((n + 4u) << 8);
    u32x4 r;
#pragma unroll
    for (int i = 0; i < 4; ++i) {
      const unsigned lo = __builtin_amdgcn_perm(a[(4 * i + 1) * 32], a[(4 * i) * 32], s1);
      const unsigned hi = __builtin_amdgcn_perm(a[(4 * i + 3) * 32], a[(4 * i + 2) * 32], s1);
      r[i] = __builtin_amdgcn_perm(hi, lo, 0x05040100u);
    }
    return r;
  }
  return *(const LAS u32x4*)(tile + ((wv >> 1) * 16 + (lane & 15)) * 128 + (wv & 1) * 64 + (lane >> 4) * 16);
}

#define TLOAD "s_mov_b32 m0, %[lm]\n\ts_nop 3\n\tbuffer_load_dwordx4 %[lo], %[rl], 0 offen lds\n\t"
#define TLOAD_IN [lm] "s"(lm), [lo] "v"(lo), [rl] "s"(rl)
#define STR2(x) #x
#define STR(x) STR2(x)
#define PUSH1(j) asm volatile("s_waitcnt vmcnt(" STR(WINM1) ")\n\tbuffer_store_dwordx4 %0, %1, %2, %3 offen sc0 sc1" \
                              :: "v"(cur), "v"(so), "s"(rs[j]), "s"((skip >> j) & 1u ? OOB : 0u) : "memory")
#define DRAW(vnew) asm volatile("s_mov_b64 exec, 1\n\tglobal_atomic_add " vnew ", %0, %1, %2 sc0\n\ts_mov_b64 exec, -1" \
                                :: "v"(ao), "v"(one), "s"(ctl) : "memory", "v254", "v255")
#define GOT(vold) asm volatile("v_mov_b32 %0, " vold : "=v"(aret) :: "memory", "v254", "v255")

__device__ __forceinline__ void comm_role(const Params& p, LAS unsigned char* lds, int wg, int wv, int lane) {
  LAS unsigned* tick = (LAS unsigned*)(lds + LDS_CTL);
  uint8_t* const ctl = p.ws[p.rank];
  const unsigned ldsw = (unsigned)(uintptr_t)lds + wv * 1024;
  auto code_of = [&](int pos) -> int { return (int)((p.sched[(pos >> 4) & 31] >> (((pos >> 2) & 3) * 8)) & 127u); };
#define TILE_ARGS(pos, slot) const int _c = code_of(pos); \
    const __amdgpu_buffer_rsrc_t rl = __builtin_amdgcn_make_buffer_rsrc((void*)((_c & 16) ? p.w : p.x), 0, MN * KS, 0x00020000); \
    const unsigned lo = tile_src_off((_c >> 4) & 1, _c >> 5, (_c & 15) * 4 + ((pos) & 3), wv, lane), lm = ldsw + ((slot) & 3) * 8192
  int pm1 = 0, p0 = wg, p1 = 16 + wg, p2 = 32 + wg;
  { TILE_ARGS(p0, 0); asm volatile(TLOAD :: TLOAD_IN : "memory"); }
  { TILE_ARGS(p1, 1); asm volatile(TLOAD :: TLOAD_IN : "memory"); }
  const unsigned call = __builtin_amdgcn_readfirstlane(
      __hip_atomic_load((const unsigned*)(ctl + CTL_DONE), __ATOMIC_RELAXED, __HIP_MEMORY_SCOPE_AGENT)) / NWG + 1u;
  __amdgpu_buffer_rsrc_t rs[7];
#pragma unroll
  for (int j = 0; j < 7; ++j) {
    const int peer = (p.rank + 1 + (j + wg + wv) % 7) & (NR - 1);
    rs[j] = __builtin_amdgcn_make_buffer_rsrc(p.ws[peer] + DATA_OFF + (size_t)(call & 1u) * SLOT_B + (size_t)p.rank * 4 * ENT_B,
                                              0, 4 * ENT_B, 0x00020000);
  }
  uint8_t* const pctl = p.ws[(p.rank + 1 + lane % 7) & (NR - 1)];
  const unsigned ao = CTL_TICK + (call & 1u) * 128u, one = 1u;
  WAIT_V(0);
  BAR;
  if (threadIdx.x == 128) COUNT_IN(ctl);
  if (wg == 0 && threadIdx.x == 64)
    __hip_atomic_store((unsigned*)(ctl + CTL_TICK + ((call & 1u) ^ 1u) * 128u), 0u, __ATOMIC_RELAXED, __HIP_MEMORY_SCOPE_AGENT);
  unsigned open = p.pubmask;
  for (int r = 0;; ++r) {
    u32x4 cur; unsigned so = 0, skip = 0;
    if (p0 < 512) {
      const int c = code_of(p0);
      cur = unit_read((c >> 4) & 1, lds + (r & 3) * 8192, wv, lane);
      so = unit_dst_off((c >> 4) & 1, c >> 5, (c & 15) * 4 + (p0 & 3), wv) + lane * 16;
      if ((c & 31) == 15) {
        const int d = ((p0 & 3) * 2 + (wv >> 2) - p.rank - 1) & (NR - 1);
        skip = d == 7 ? 0x7fu : 0x7fu & ~(1u << ((d + 70 - wg - wv) % 7));
      }
#pragma unroll
      for (int j = 0; j < WINM1; ++j) PUSH1(j);
      asm volatile("s_waitcnt vmcnt(" STR(WINM1) ")" ::: "memory");
      if (skip) WAIT_V(0);
      if (wv == 0) {
        unsigned aret;
        if (r & 1) GOT("v254"); else GOT("v255");
        if (lane == 0) tick[r & 1] = aret;
        WAIT_L(0);
      }
    } else {
      WAIT_V(0);
    }
    BAR;
    if (r > 0) {
      if ((pm1 >> 4) >= p.fine_start) {
        if (wv == 1 + (r & 3) && lane < 7)
          __hip_atomic_store((unsigned*)(pctl + rflag_off(p.rank, pm1)), call * 16u, __ATOMIC_RELAXED, __HIP_MEMORY_SCOPE_SYSTEM);
      }
      unsigned done = (p0 >> 4) >= 32 ? open : open & ((1u << (p0 >> 4)) - 1u);
      open &= ~done;
      for (; done; done &= done - 1u)
        if (wv == 1 + (r & 3) && lane < 7)
          __hip_atomic_fetch_add((unsigned*)(pctl + uflag_off(p.rank, __builtin_ctz(done))), 1u, __ATOMIC_RELAXED, __HIP_MEMORY_SCOPE_SYSTEM);
    }
    if (p0 >= 512) break;
    if (r > 0) p2 = p1 < 512 ? 48 + __builtin_amdgcn_readfirstlane(tick[r & 1]) : 512;
#pragma unroll
    for (int j = WINM1; j < 7; ++j) PUSH1(j);
    { const int pn = p2 < 512 ? p2 : 511; TILE_ARGS(pn, r + 2); asm volatile(TLOAD :: TLOAD_IN : "memory"); }
    if (wv == 0) { if (r & 1) DRAW("v255"); else DRAW("v254"); }
    pm1 = p0; p0 = p1; p1 = p2;
  }
  WAIT_V(0);
  if (wg == 0 && wv == 0) {
    const unsigned* cnt = (const unsigned*)(ctl + (lane & 1 ? CTL_TMO : CTL_SLICE));
    const unsigned long long t0 = __builtin_amdgcn_s_memrealtime();
    for (;;) {
      const unsigned v = __hip_atomic_load(cnt, __ATOMIC_RELAXED, __HIP_MEMORY_SCOPE_SYSTEM);
      if ((int)((unsigned)__builtin_amdgcn_readlane(v, 0) - call * (7u * 32u)) >= 0) break;
      if ((unsigned)__builtin_amdgcn_readlane(v, 1) == call || __builtin_amdgcn_s_memrealtime() - t0 > DEADLINE_TICKS) {
        if (lane == 0) {
          __hip_atomic_store((unsigned*)(ctl + CTL_TMO), call, __ATOMIC_RELAXED, __HIP_MEMORY_SCOPE_AGENT);
          p.out[0] = __builtin_nanf("");
        }
        break;
      }
      __builtin_amdgcn_s_sleep(4);
    }
  }
}

#define DMA2(m0v, soff) asm volatile("s_mov_b32 m0, %0\n\ts_nop 0\n\t" \
    "buffer_load_dwordx4 %1, %2, %3 offen " DMA_POL " lds\n\t" \
    "buffer_load_dwordx4 %1, %2, %3 offen offset:1024 " DMA_POL " lds" \
    :: "s"(m0v), "v"(voff), "s"(rdata), "s"(soff) : "memory")
#define DMA3(m0v, soff, m0h, soffh) asm volatile("s_mov_b32 m0, %0\n\ts_nop 0\n\t" \
    "buffer_load_dwordx4 %1, %2, %3 offen " DMA_POL " lds\n\t" \
    "buffer_load_dwordx4 %1, %2, %3 offen offset:1024 " DMA_POL " lds\n\t" \
    "s_mov_b32 m0, %4\n\ts_mov_b32 exec_hi, 0\n\t" \
    "buffer_load_dwordx4 %1, %2, %5 offen " DMA_POL " lds\n\t" \
    "s_mov_b32 exec_hi, -1" \
    :: "s"(m0v), "v"(voff), "s"(rdata), "s"(soff), "s"(m0h), "s"(soffh) : "memory")

#define DMA1(m0v, soff) asm volatile("s_mov_b32 m0, %0\n\ts_nop 0\n\tbuffer_load_dwordx4 %1, %2, %3 offen " DMA_POL " lds" \
    :: "s"(m0v), "v"(voff), "s"(rdata), "s"(soff) : "memory")
#define DMA1B(m0v, soff) asm volatile("s_mov_b32 m0, %0\n\ts_nop 0\n\tbuffer_load_dwordx4 %1, %2, %3 offen offset:1024 " DMA_POL " lds" \
    :: "s"(m0v), "v"(voff), "s"(rdata), "s"(soff) : "memory")
#define DMA1H(m0v, soff) asm volatile("s_mov_b32 m0, %0\n\ts_mov_b32 exec_hi, 0\n\tbuffer_load_dwordx4 %1, %2, %3 offen " DMA_POL " lds\n\t" \
    "s_mov_b32 exec_hi, -1" :: "s"(m0v), "v"(voff), "s"(rdata), "s"(soff) : "memory")
#define PIN(x) asm volatile("" : "+s"(x))

__device__ __forceinline__ int ready_from(unsigned long long okmask, int base) {
  const int r = base + (int)((unsigned)__builtin_ctzll(~okmask | (1ull << 63)) * 57u >> 9);
  return r < 32 ? r : 32;
}

__device__ __forceinline__ void gemm_role(const Params& p, LAS unsigned char* lds, int g, int wv, int lane) {
  const int lin = (g & 7) * 30 + (g >> 3), band = lin / 80, cb = (lin % 80) / 5;
  const int tcol = (band & 1) ? 15 - cb : cb, trow = band * 5 + lin % 5;
  uint8_t* const ctl = p.ws[p.rank];
#pragma unroll
  for (int i = 0; i < 3; ++i) {
    const int gid = g + i * NGEMM, type = (gid >> 8) & 1, kt = (gid >> 6) & 3;
    if (gid < 512) {
      const __amdgpu_buffer_rsrc_t rl = __builtin_amdgcn_make_buffer_rsrc((void*)(type ? p.w : p.x), 0, MN * KS, 0x00020000);
      const unsigned lo = tile_src_off(type, kt, gid & 63, wv, lane), lm = (unsigned)(uintptr_t)lds + i * 8192 + wv * 1024;
      asm volatile(TLOAD :: TLOAD_IN : "memory");
    }
  }
  const unsigned call = __builtin_amdgcn_readfirstlane(
      __hip_atomic_load((const unsigned*)(ctl + CTL_DONE), __ATOMIC_RELAXED, __HIP_MEMORY_SCOPE_AGENT)) / NWG + 1u;
  const __amdgpu_buffer_rsrc_t rdata =
      __builtin_amdgcn_make_buffer_rsrc(ctl + DATA_OFF + (size_t)(call & 1u) * SLOT_B, 0, SLOT_B, 0x00020000);
  WAIT_V(0);
  BAR;
#pragma unroll
  for (int i = 0; i < 3; ++i) {
    const int gid = g + i * NGEMM, type = (gid >> 8) & 1, kt = (gid >> 6) & 3;
    if (gid < 512)
      __builtin_amdgcn_raw_buffer_store_b128(unit_read(type, lds + i * 8192, wv, lane), rdata,
                                             (unsigned)p.rank * 4u * ENT_B + unit_dst_off(type, kt, gid & 63, wv) + lane * 16, 0, 17);
  }
  LAS unsigned* ord = (LAS unsigned*)(lds + LDS_ORD);
  LAS unsigned* tbl = (LAS unsigned*)(lds + LDS_TBL);
  LAS unsigned* bc = (LAS unsigned*)(lds + LDS_CTL);
  const int hj = (trow / 5 + 2 - tcol % 3) % 3;
  const bool host = trow % 5 == 0 && hj < 2;
  auto group_of = [&](int i) -> int { return i < 4 ? 4 * trow + i : i < 8 ? 4 * tcol + i - 4 : host ? 60 + (p.rank >> 1) : 4 * trow; };
  auto pub_pos = [&](int s, int kt, int i) -> int {
    const int type = (i >> 2) & 1, gg = group_of(i);
    const int pos = p.inv[s][kt * 32 + type * 16 + (gg >> 2)] * 4 + (gg & 3), r = pos >> 4;
    return r >= p.fine_start ? pos : (r + __builtin_ctz(p.pubmask >> r)) * 16 + 15;
  };
  if (wv == 0) {
    const int e = lane & 31, s = e >> 2, kt = e & 3;
    int key = kt;
    if (s != p.rank) {
      int m = 0;
      for (int i = 0; i < 9; ++i) { const int r = pub_pos(s, kt, i); m = r > m ? r : m; }
      key = 4 + m * 32 + kt * 8 + ((s - p.rank + g) & 7);
    }
    int rk = 0;
    for (int l = 0; l < 32; ++l) rk += __builtin_amdgcn_readlane(key, l) < key;
    if (lane < 32) ord[rk] = e;
  }
  WAIT_V(0);
  __syncthreads();
  if (threadIdx.x == 64) COUNT_IN(ctl);
  if (threadIdx.x == 0)
    __hip_atomic_store((unsigned*)(ctl + CTL_LFLAG) + g, call * 16u, __ATOMIC_RELAXED, __HIP_MEMORY_SCOPE_SYSTEM);
  if (threadIdx.x < 352) {
    int t = threadIdx.x / 9; const int i = threadIdx.x - t * 9; t = t < 32 ? t : 31;
    const int e = ord[t], s = e >> 2, kt = e & 3, type = (i >> 2) & 1, gg = group_of(i);
    unsigned off;
    if (s == p.rank) off = CTL_LFLAG + 4u * (unsigned)((type * 256 + kt * 64 + gg) % NGEMM);
    else { const int pp = pub_pos(s, kt, i);
           off = (pp >> 4) >= p.fine_start ? rflag_off(s, pp) : uflag_off(s, pp >> 4); }
    tbl[threadIdx.x] = off;
  }
  const int ordv = ord[lane & 31];
  ((LAS unsigned long long*)(lds + LDS_SNAP))[threadIdx.x] = 0ull;
  ((LAS unsigned*)(lds + LDS_SNAP + 4096))[threadIdx.x] = 0u;
  if (threadIdx.x == 0) bc[63] = 0u;
  if (threadIdx.x < 7) ((LAS unsigned long long*)(bc + 32))[threadIdx.x] = (unsigned long long)(p.ws[(p.rank + 1 + threadIdx.x) & (NR - 1)] + CTL_SLICE);
  int scale_bits = __builtin_amdgcn_readfirstlane(__builtin_bit_cast(int, p.sx[0] * p.sw[0]));
  asm volatile("" : "+s"(scale_bits));
  WAIT_V(0);
  __syncthreads();

  const int wr = wv >> 2, wc = wv & 3;
  const unsigned voff = lane * 16;
  const unsigned ldsb = (unsigned)(uintptr_t)lds;
  const unsigned a0o = (9 * wr + wc) * 2048, a8o = (9 * wr + 8) * 2048 + wc * 512, b0o = (4 * (wv >> 1) + (wv & 1)) * 2048;
  const bool has9 = host && wr == 0;
  const unsigned ga0 = (unsigned)(16 * trow + 8 * wr + wc) * 2048u, gb0 = OP_B + tcol * 32768u + b0o;
  const unsigned ga8 = has9 ? (unsigned)(240 + 2 * p.rank + hj) * 2048u + wc * 512u : OOB;
  const unsigned la0 = ldsb + LDS_A + a0o, la8 = ldsb + LDS_A + a8o, lb0 = ldsb + LDS_B + b0o;
#define ENT(t) ((unsigned)__builtin_amdgcn_readlane(ordv, (t)) * ENT_B)
#define ST_A03(eb, sl) DMA2(la0 + (sl) * A_SLOT, (eb) + ga0)
#define ST_A48(eb, sl) DMA3(la0 + 8192 + (sl) * A_SLOT, (eb) + ga0 + 8192, la8 + (sl) * A_SLOT, (eb) + ga8)
#define ST_B0(eb, sl) DMA2(lb0 + (sl) * B_SLOT, (eb) + gb0)
#define ST_B1(eb, sl) DMA2(lb0 + 4096 + (sl) * B_SLOT, (eb) + gb0 + 4096)
#define ST_ALL(eb, sl) do { ST_B0(eb, sl); ST_B1(eb, sl); ST_A03(eb, sl); ST_A48(eb, sl); } while (0)
#define LDF(dst, base, off) do { const i32x4 _l = *(const LAS i32x4*)((base) + (off)), _h = *(const LAS i32x4*)((base) + (off) + 1024); \
    dst = __builtin_shufflevector(_l, _h, 0, 1, 2, 3, 4, 5, 6, 7); SCHED; } while (0)
#define MM(m, n, Af) do { acc[m][n] = __builtin_amdgcn_mfma_scale_f32_16x16x128_f8f6f4(Af, Bf[n], acc[m][n], 1, 1, 0, 0, 0, 0); SCHED; } while (0)

  f32x4 acc[9][4];
#pragma unroll
  for (int m = 0; m < 9; ++m)
#pragma unroll
    for (int n = 0; n < 4; ++n) acc[m][n] = (f32x4){0.f, 0.f, 0.f, 0.f};
  i32x8 a0, a1, a2, a3, a4, Bf[4];
  int t = 0, ready = 0, nrestart = 0;
  unsigned dead = 0;
  const unsigned thr = call * 16u;
  const unsigned pollo = tbl[threadIdx.x < 352 ? threadIdx.x : 351];
  const __amdgpu_buffer_rsrc_t rctl = __builtin_amdgcn_make_buffer_rsrc(ctl, 0, CTL_BYTES, 0x00020000);
  const __amdgpu_buffer_rsrc_t rout = __builtin_amdgcn_make_buffer_rsrc(p.out, 0, MN * MN * 4, 0x00020000);
  const LAS unsigned char* const pa0 = lds + LDS_A + wr * (9 * 2048) + lane * 16;
  const LAS unsigned char* const pb0 = lds + LDS_B + wc * (4 * 2048) + lane * 16;
#define POLL(buf) do { const unsigned _po = (unsigned)(ready * 9 + 62 - wv * 64) < 126u && ready < 32 ? 0u : OOB; \
    asm volatile("s_mov_b32 m0, %0\n\ts_nop 0\n\tbuffer_load_dword %1, %2, %3 offen " DMA_POL " lds" \
                 :: "s"(ldsb + LDS_SNAP + (buf) * 2048 + wv * 256), "v"(pollo), "s"(rctl), "s"(_po) : "memory"); } while (0)
#define SNAP_AT(buf) ((const LAS unsigned*)(lds + LDS_SNAP + ((buf) * 512 + ready * 9) * 4) + lane)
#define OKMASK(v) __builtin_amdgcn_ballot_w64((int)((v) - thr) >= 0)
#define STAGE_FROM(u) do { const unsigned _e0 = ENT(u); ST_ALL(_e0, (u) & 1); sn = ready > (u) + 1; \
    if (sn) { BAR;   const unsigned _e1 = ENT((u) + 1); ST_ALL(_e1, ((u) & 1) ^ 1); } \
    POLL(((u) & 1) ^ 1); } while (0)
  int sn = -1;
  for (;;) {
    if (sn < 0) {
      if (ready <= t) {
        if (wv == 0) {
          const unsigned* fp = (const unsigned*)(ctl + (lane == 63 ? CTL_TMO : tbl[t * 9 + lane]));
          unsigned* tmo = (unsigned*)(ctl + CTL_TMO);
          const unsigned long long t0 = __builtin_amdgcn_s_memrealtime();
          int r; unsigned dd = 0;
          for (;;) {
            const unsigned v = __hip_atomic_load(fp, __ATOMIC_RELAXED, __HIP_MEMORY_SCOPE_SYSTEM);
            r = ready_from(OKMASK(v), t);
            if (r > t) break;
            if ((unsigned)__builtin_amdgcn_readlane(v, 63) == call || __builtin_amdgcn_s_memrealtime() - t0 > DEADLINE_TICKS) {
              if (lane == 0) __hip_atomic_store(tmo, call, __ATOMIC_RELAXED, __HIP_MEMORY_SCOPE_AGENT);
              r = 32; dd = 64; break;
            }
            __builtin_amdgcn_s_sleep(2);
          }
          if (lane == 0) bc[nrestart & 31] = (unsigned)r | (dd << 8);
          asm volatile("s_waitcnt vmcnt(0) lgkmcnt(0)" ::: "memory");
        }
        BAR;
        const unsigned v = __builtin_amdgcn_readfirstlane(bc[nrestart & 31]);
        ready = v & 63; dead |= v >> 8; ++nrestart;
      }
      STAGE_FROM(t);
    }
    int s1 = sn;
    sn = -1;
    if (s1) WAIT_V(13); else WAIT_V(4);
    BAR;
    { const LAS unsigned char* pa = pa0 + (t & 1) * A_SLOT; const LAS unsigned char* pb = pb0 + (t & 1) * B_SLOT;
      LDF(Bf[0], pb, 0); LDF(a0, pa, 0); LDF(a1, pa, 2048); LDF(Bf[1], pb, 2048); LDF(Bf[2], pb, 4096); LDF(a2, pa, 4096); LDF(Bf[3], pb, 6144); }
    if (t == 31) break;
    unsigned en2 = ENT((t + 2) & 31);
    int first = 1;
    for (;;) {
      const int sl = t & 1;
      const LAS unsigned char* pa = pa0 + sl * A_SLOT;
      LDF(a3, pa, 3 * 2048);
      MM(0, 0, a0);
      const int rich = (ready > t + 2 || ready == 32 || (first & s1)) ? 1 : 0;
      const LAS unsigned* psn = SNAP_AT(sl ^ rich ^ 1);
      asm volatile("" : "+v"(psn)); SCHED;
      MM(1, 0, a1);
      const LAS unsigned char* pan = pa0 + (sl ^ 1) * A_SLOT; const LAS unsigned char* pbn = pb0 + (sl ^ 1) * B_SLOT;
      asm volatile("" : "+v"(pan), "+v"(pbn)); SCHED;
      MM(0, 1, a0);
      unsigned mB = lb0 + sl * B_SLOT, mA = la0 + sl * A_SLOT;
      PIN(mB); PIN(mA); SCHED;
      MM(1, 1, a1);
      unsigned mA8 = la8 + sl * A_SLOT, mP = ldsb + LDS_SNAP + sl * 2048 + wv * 256;
      PIN(mA8); PIN(mP); SCHED;
      MM(0, 2, a0); MM(1, 2, a1);
      WAIT_V(10); if (!rich) { if (first) WAIT_V(0); else WAIT_V(3); }
      first = 0;
      WAIT_L(0); BAR; SCHED;
      const unsigned snap = *psn;
      SCHED;
      MM(0, 3, a0);
      LDF(a4, pa, 6 * 2048);
      MM(1, 3, a1);
      LDF(a0, pa, 4 * 2048);
      MM(2, 0, a2);
      unsigned long long okm = OKMASK(snap);
      SCHED;
      MM(2, 1, a2);
      LDF(a1, pa, 5 * 2048);
      unsigned fz = (unsigned)__builtin_ctzll(~okm | (1ull << 63)) * 57u;
      PIN(fz); SCHED;
      MM(2, 2, a2);
      ready += (int)(fz >> 9); ready = ready < 32 ? ready : 32;
      if (__builtin_expect(!s1, 0)) {
        if (ready > t + 1) { const unsigned e1 = ENT(t + 1); ST_ALL(e1, sl ^ 1); s1 = 1; }
        POLL(2);
      }
      SCHED;
      MM(2, 3, a2);
      int s2 = ready > t + 2 ? s1 : 0;
      unsigned e2 = s2 ? en2 : OOB;
      PIN(s2); PIN(e2);
      LDF(a2, pa, 8 * 2048);
      MM(3, 0, a3);
      DMA1(mB, e2 + gb0); SCHED;
      MM(3, 1, a3);
      DMA1B(mB, e2 + gb0); SCHED;
      MM(3, 2, a3);
      DMA1(mB + 4096, e2 + gb0 + 4096); SCHED;
      MM(3, 3, a3);
      LDF(a3, pa, 7 * 2048);
      DMA1B(mB + 4096, e2 + gb0 + 4096); SCHED;
      MM(4, 0, a0);
      DMA1(mA, e2 + ga0); SCHED;
      MM(4, 1, a0);
      DMA1B(mA, e2 + ga0); SCHED;
      MM(4, 2, a0);
      en2 = ENT((t + 3) & 31);
      SCHED;
      MM(4, 3, a0);
      unsigned po = (unsigned)(ready * 9 + 62 - wv * 64) < 126u && ready < 32 ? 0u : OOB;
      PIN(po); SCHED;
      MM(5, 0, a1); MM(5, 1, a1);
      WAIT_V(10); if (__builtin_expect(!s1, 0)) WAIT_V(0);
      WAIT_L(0); BAR; SCHED;
      MM(5, 2, a1);
      LDF(a0, pan, 0);
      asm volatile("s_mov_b32 m0, %0\n\ts_nop 0\n\tbuffer_load_dword %1, %2, %3 offen " DMA_POL " lds"
                   :: "s"(mP), "v"(pollo), "s"(rctl), "s"(po) : "memory");
      SCHED;
      MM(5, 3, a1);
      LDF(a1, pan, 2048);
      DMA1(mA + 8192, e2 + ga0 + 8192); SCHED;
      if (has9) { MM(8, 0, a2); MM(8, 1, a2); MM(8, 2, a2); MM(8, 3, a2); }
      LDF(a2, pan, 4096);
      DMA1B(mA + 8192, e2 + ga0 + 8192); SCHED;
      MM(7, 0, a3);
      DMA1H(mA8, e2 + ga8);
      if (__builtin_expect(!s1, 0)) {
        ready = ready_from(OKMASK(*SNAP_AT(2)), ready);
        if (ready > t + 1) STAGE_FROM(t + 1);
      }
      SCHED;
      MM(6, 0, a4);
      LDF(Bf[0], pbn, 0);
      MM(7, 1, a3); MM(6, 1, a4);
      LDF(Bf[1], pbn, 2048);
      MM(7, 2, a3); MM(6, 2, a4);
      LDF(Bf[2], pbn, 4096);
      MM(7, 3, a3); MM(6, 3, a4);
      LDF(Bf[3], pbn, 6144);
      ++t;
      if (!s1 || t == 31) break;
      s1 = s2;
    }
    if (s1) break;
    if (sn < 0) {
      asm volatile("s_waitcnt vmcnt(0) lgkmcnt(0)" ::: "memory"); BAR;
      ready = ready_from(OKMASK(*SNAP_AT((t & 1) ^ 1)), ready);
      WAIT_L(0); BAR;
    }
  }
  {
    const float sc = __builtin_bit_cast(float, dead ? 0x7fc00000 : scale_bits);
    const unsigned ov = (unsigned)((lane >> 4) * 4 * MN + (lane & 15) * 4) * 4u;
    const unsigned os = (unsigned)((trow * 256 + wr * 128) * MN + tcol * 256 + wc * 64) * 4u;
#define OUT(m) do { f32x4 o[4]; \
      asm volatile("" : "+v"(acc[m][0]), "+v"(acc[m][1]), "+v"(acc[m][2]), "+v"(acc[m][3])); \
      _Pragma("unroll") for (int j = 0; j < 4; ++j) { o[j][0] = acc[m][0][j] * sc; o[j][1] = acc[m][1][j] * sc; o[j][2] = acc[m][2][j] * sc; o[j][3] = acc[m][3][j] * sc; } \
      SCHED; \
      _Pragma("unroll") for (int j = 0; j < 4; ++j) \
        __builtin_amdgcn_raw_buffer_store_b128(__builtin_bit_cast(u32x4, o[j]), rout, ov, os + (unsigned)((m) * 16 + j) * (MN * 4u), OUT_AUX); \
      asm volatile("s_nop 1" ::: "memory");                     \
      SCHED; } while (0)
    const LAS unsigned char* pa = pa0 + A_SLOT;
    LDF(a3, pa, 3 * 2048);
    MM(0, 0, a0); MM(1, 0, a1); MM(0, 1, a0); MM(1, 1, a1); MM(0, 2, a0); MM(1, 2, a1);
    WAIT_V(0); BAR; SCHED;
    MM(0, 3, a0);
    LDF(a4, pa, 8 * 2048);
    MM(1, 3, a1);
    LDF(a0, pa, 4 * 2048);
    MM(2, 0, a2); MM(2, 1, a2);
    LDF(a1, pa, 5 * 2048);
    MM(2, 2, a2); MM(2, 3, a2);
    LDF(a2, pa, 6 * 2048);
    f32x4 o8[4];
    const unsigned os8 = (unsigned)((3840 + 32 * p.rank + 16 * hj) * MN + tcol * 256 + wc * 64) * 4u;
#define SHARE(d) do { if (has9) { \
      const __amdgpu_buffer_rsrc_t rd = __builtin_amdgcn_make_buffer_rsrc(p.outs[(p.rank + 1 + ((d) < 7 ? ((d) + tcol + wc) % 7 : 7)) & (NR - 1)], 0, MN * MN * 4, 0x00020000); \
      asm volatile("s_waitcnt vmcnt(" STR(SHARE_WIN) ")" ::: "memory"); \
      _Pragma("unroll") for (int j = 0; j < 4; ++j) \
        __builtin_amdgcn_raw_buffer_store_b128(__builtin_bit_cast(u32x4, o8[j]), rd, ov, os8 + (unsigned)j * (MN * 4u), 17); } \
      SCHED; } while (0)
    if (has9) { MM(8, 0, a4); MM(8, 1, a4); MM(8, 2, a4); MM(8, 3, a4); }
    OUT(0);
#pragma unroll
    for (int j = 0; j < 4; ++j) { o8[j][0] = acc[8][0][j] * sc; o8[j][1] = acc[8][1][j] * sc; o8[j][2] = acc[8][2][j] * sc; o8[j][3] = acc[8][3][j] * sc; }
    SCHED;
    SHARE(0);
    MM(3, 0, a3); MM(3, 1, a3); MM(3, 2, a3); MM(3, 3, a3);
    LDF(a3, pa, 7 * 2048);
    OUT(1); SHARE(1);
    MM(4, 0, a0); MM(4, 1, a0); MM(4, 2, a0); MM(4, 3, a0);
    OUT(2); SHARE(2);
    MM(5, 0, a1); MM(5, 1, a1); MM(5, 2, a1); MM(5, 3, a1);
    OUT(3); SHARE(3);
    MM(6, 0, a2); MM(6, 1, a2); MM(6, 2, a2); MM(6, 3, a2);
    OUT(4); SHARE(4);
    MM(7, 0, a3); MM(7, 1, a3); MM(7, 2, a3); MM(7, 3, a3);
    OUT(5); SHARE(5); OUT(6); SHARE(6); OUT(7); SHARE(7);
    if (has9) {
      WAIT_V(8);
      unsigned old = 0;
      if (lane == 0) old = __hip_atomic_fetch_add(bc + 63, 1u, __ATOMIC_RELAXED, __HIP_MEMORY_SCOPE_WORKGROUP);
      if (__builtin_amdgcn_readfirstlane(old) == 3u && lane < 7)
        __hip_atomic_fetch_add((GAS unsigned*)((LAS unsigned long long*)(bc + 32))[lane], 1u, __ATOMIC_RELAXED, __HIP_MEMORY_SCOPE_SYSTEM);
    }
  }
}

__global__ void __launch_bounds__(512, 2) gemm_ag_kernel(const Params p) {
  __shared__ __attribute__((aligned(16))) unsigned char lds_raw[LDS_BYTES];
  LAS unsigned char* lds = (LAS unsigned char*)lds_raw;
  const int wv = __builtin_amdgcn_readfirstlane(threadIdx.x >> 6), lane = threadIdx.x & 63;
  if (blockIdx.x < NCOMM) comm_role(p, lds, blockIdx.x, wv, lane);
  else gemm_role(p, lds, blockIdx.x - NCOMM, wv, lane);
}

static void make_schedule(int s, uint8_t* sc) {
  static const int nlate[5] = {0, 3, 5, 8, 8};
  static const double fin[5] = {0.0, 3.0, 5.2, 7.6, 7.6};
  int r = 0;
  for (int kt = 0; kt < 3; ++kt)
    for (int q = 0; q < 16; ++q) {
      sc[r++] = (uint8_t)(kt * 32 + 16 + (q + 2 * s) % 16);
      sc[r++] = (uint8_t)(kt * 32 + (q + 2 * s) % 16);
    }
  double due[16]; int late[16], nl = 0;
  sc[r++] = (uint8_t)(96 + 15);
  for (int c = 0; c < 15; ++c) {
    const int lv = c % 5, i = ((s - (lv * 3 + c / 5) * 3) % 8 + 8) % 8;
    if (i < nlate[lv]) { due[nl] = fin[lv] - (nlate[lv] - i) * 1.15 + 0.01 * (c / 5); late[nl++] = c; }
    else sc[r++] = (uint8_t)(96 + c);
  }
  for (int q = 0; q < 16; ++q) sc[r++] = (uint8_t)(96 + 16 + (q + 2 * s) % 16);
  for (int k = 0; k < nl; ++k) {
    int best = -1;
    for (int j = 0; j < nl; ++j) if (late[j] >= 0 && (best < 0 || due[j] < due[best])) best = j;
    sc[r++] = (uint8_t)(96 + late[best]); late[best] = -1;
  }
}

}

extern "C" void kernel_launch(int rank, int nranks, ncclComm_t comm,
                              hipStream_t stream,
                              void* const* d_in, const int* in_sizes, int n_in,
                              void* d_out, int out_size,
                              const int* in_dtypes, int out_dtype,
                              void* const* d_out_all,
                              void* const* d_ws_all, size_t ws_bytes) {
  (void)comm; (void)in_dtypes; (void)out_dtype;
  if (nranks != NR || n_in != 4 || (unsigned)rank >= (unsigned)NR || ws_bytes < WS_NEED || out_size != MN * MN ||
      in_sizes[0] != MN * KS || in_sizes[1] != MN * KS) {
    fprintf(stderr, "kernel_launch: unexpected problem shape\n");
    return;
  }
  Params p;
  p.x = (const uint8_t*)d_in[0]; p.w = (const uint8_t*)d_in[1];
  p.sx = (const float*)d_in[2]; p.sw = (const float*)d_in[3];
  p.out = (float*)d_out;
  for (int i = 0; i < NR; ++i) { p.ws[i] = (uint8_t*)d_ws_all[i]; p.outs[i] = (float*)d_out_all[i]; }
  p.rank = rank;
  p.pubmask = 0x00AA8080u;
  p.fine_start = 24;
  for (int s = 0; s < NR; ++s) {
    uint8_t sc[128];
    make_schedule(s, sc);
    for (int r = 0; r < 128; ++r) p.inv[s][sc[r]] = (uint8_t)r;
    if (s == rank)
      for (int i = 0; i < 32; ++i)
        p.sched[i] = sc[4 * i] | (sc[4 * i + 1] << 8) | (sc[4 * i + 2] << 16) | ((unsigned)sc[4 * i + 3] << 24);
  }
  hipLaunchKernelGGL(gemm_ag_kernel, dim3(NWG), dim3(512), 0, stream, p);
  HIP_CHECK(hipGetLastError());
}
```

```cpp
#include <hip/hip_runtime.h>
#include <rccl/rccl.h>
#include <cstdio>
#include <cstdint>

#define HIP_CHECK(expr) do { hipError_t _e = (expr); if (_e != hipSuccess) { \
    fprintf(stderr, "HIP error %s at %s:%d\n", hipGetErrorString(_e), __FILE__, __LINE__); \
    return; } } while (0)

#define DMA_POL "sc0 sc1"
#define WINM1 4
#define SHARE_WIN 6
#define OUT_AUX 16

namespace {

#define LAS __attribute__((address_space(3)))
#define GAS __attribute__((address_space(1)))
typedef unsigned u32x4 __attribute__((ext_vector_type(4)));
typedef int i32x4 __attribute__((ext_vector_type(4)));
typedef int i32x8 __attribute__((ext_vector_type(8)));
typedef float f32x4 __attribute__((ext_vector_type(4)));

constexpr int NR = 8;
constexpr int MN = 4096;
constexpr int KS = 512;
constexpr int NCOMM = 16;
constexpr int NGEMM = 240;
constexpr int NWG = NCOMM + NGEMM;
constexpr unsigned ENT_B = 1u << 20;
constexpr unsigned OP_B = 1u << 19;
constexpr unsigned SLOT_B = 32u << 20;
constexpr size_t DATA_OFF = 1u << 20;
constexpr size_t WS_NEED = DATA_OFF + 2ull * SLOT_B;
constexpr unsigned CTL_DONE = 0x0000, CTL_TMO = 0x0080, CTL_TICK = 0x0200, CTL_SLICE = 0x0400, CTL_LFLAG = 0x1000,
                   CTL_UFLAG = 0x8000, CTL_RFLAG = 0x10000, CTL_BYTES = 0x30000;
__host__ __device__ constexpr unsigned uflag_off(int s, int unit) { return CTL_UFLAG + (unsigned)(s * 32 + unit) * 128u; }
__host__ __device__ constexpr unsigned rflag_off(int s, int pos) { return CTL_RFLAG + (unsigned)(s * 128 + (pos >> 2)) * 128u + (unsigned)(pos & 3) * 4u; }
constexpr unsigned OOB = 0x40000000u;
constexpr unsigned long long DEADLINE_TICKS = 100000000ull;

constexpr int A_SLOT = 18 * 2048, B_SLOT = 16 * 2048;
constexpr int LDS_A = 0, LDS_B = 2 * A_SLOT;
constexpr int LDS_TBL = LDS_B + 2 * B_SLOT;
constexpr int LDS_SNAP = LDS_TBL + 352 * 4;
constexpr int LDS_ORD = LDS_SNAP + 6144;
constexpr int LDS_CTL = LDS_ORD + 128;
constexpr int LDS_BYTES = LDS_CTL + 256;

struct Params {
  const uint8_t* x; const uint8_t* w; uint8_t* ctl;
  unsigned sched01[2];
  uint8_t* peer[14];
  const float* sx; const float* sw; float* out;
  float* outs[NR];
  int rank;
  unsigned pubmask;
  int fine_start;
  unsigned sched[32];
  uint8_t inv[NR][128];
};

#define COUNT_IN(ctl) __hip_atomic_fetch_add((unsigned*)((ctl) + CTL_DONE), 1u, __ATOMIC_RELAXED, __HIP_MEMORY_SCOPE_AGENT)
#define WAIT_V(n) asm volatile("s_waitcnt vmcnt(" #n ")" ::: "memory")
#define WAIT_L(n) asm volatile("s_waitcnt lgkmcnt(" #n ")" ::: "memory")
#define BAR do { asm volatile("" ::: "memory"); __builtin_amdgcn_s_barrier(); asm volatile("" ::: "memory"); } while (0)
#define SCHED __builtin_amdgcn_sched_barrier(0)

__device__ __forceinline__ unsigned tile_src_off(int type, int kt, int g, int wv, int lane) {
  const int row = wv * 8 + (lane >> 3), ch = (lane & 7) * 16;
  return type ? (unsigned)((kt * 128 + (g & 1) * 64 + row) * MN + (g >> 1) * 128 + ch)
              : (unsigned)((g * 64 + row) * KS + kt * 128 + ch);
}
__device__ __forceinline__ unsigned unit_dst_off(int type, int kt, int g, int wv) {
  const int u = type ? ((g >> 1) * 8 + wv) * 2 + (g & 1) : g * 8 + wv;
  return (unsigned)kt * ENT_B + (unsigned)type * OP_B + (unsigned)u * 1024u;
}
__device__ __forceinline__ u32x4 unit_read(int type, const LAS unsigned char* tile, int wv, int lane) {
  if (type) {
    const LAS unsigned* a = (const LAS unsigned*)(tile + (lane >> 4) * 2048 + (wv >> 2) * 64 + (lane & 15) * 4);
    const unsigned n = wv & 3, s1 = n | ((n + 4u) << 8);
    u32x4 r;
#pragma unroll
    for (int i = 0; i < 4; ++i) {
      const unsigned lo = __builtin_amdgcn_perm(a[(4 * i + 1) * 32], a[(4 * i) * 32], s1);
      const unsigned hi = __builtin_amdgcn_perm(a[(4 * i + 3) * 32], a[(4 * i + 2) * 32], s1);
      r[i] = __builtin_amdgcn_perm(hi, lo, 0x05040100u);
    }
    return r;
  }
  return *(const LAS u32x4*)(tile + ((wv >> 1) * 16 + (lane & 15)) * 128 + (wv & 1) * 64 + (lane >> 4) * 16);
}

#define TLOAD "s_mov_b32 m0, %[lm]\n\ts_nop 3\n\tbuffer_load_dwordx4 %[lo], %[rl], 0 offen lds\n\t"
#define TLOAD_IN [lm] "s"(lm), [lo] "v"(lo), [rl] "s"(rl)
#define STR2(x) #x
#define STR(x) STR2(x)
#define PUSH1(j) asm volatile("s_waitcnt vmcnt(" STR(WINM1) ")\n\tbuffer_store_dwordx4 %0, %1, %2, %3 offen sc0 sc1" \
                              :: "v"(cur), "v"(so), "s"(rs[j]), "s"((skip >> j) & 1u ? OOB : 0u) : "memory")
#define DRAW(vnew) asm volatile("s_mov_b64 exec, 1\n\tglobal_atomic_add " vnew ", %0, %1, %2 sc0\n\ts_mov_b64 exec, -1" \
                                :: "v"(ao), "v"(one), "s"(ctl) : "memory", "v254", "v255")
#define GOT(vold) asm volatile("v_mov_b32 %0, " vold : "=v"(aret) :: "memory", "v254", "v255")

__device__ __forceinline__ void comm_role(const Params& p, LAS unsigned char* lds, int wg, int wv, int lane) {
  LAS unsigned* tick = (LAS unsigned*)(lds + LDS_CTL);
  uint8_t* const ctl = p.ctl;
  const unsigned ldsw = (unsigned)(uintptr_t)lds + wv * 1024;
  auto code_of = [&](int pos) -> int { return (int)((p.sched[(pos >> 4) & 31] >> (((pos >> 2) & 3) * 8)) & 127u); };
#define TILE_ARGS(_c, pos, slot) \
    const __amdgpu_buffer_rsrc_t rl = __builtin_amdgcn_make_buffer_rsrc((void*)(((_c) & 16) ? p.w : p.x), 0, MN * KS, 0x00020000); \
    const unsigned lo = tile_src_off(((_c) >> 4) & 1, (_c) >> 5, ((_c) & 15) * 4 + ((pos) & 3), wv, lane), lm = ldsw + ((slot) & 3) * 8192
  int pm1 = 0, p0 = wg, p1 = 16 + wg, p2 = 32 + wg;
  int c0 = (int)((p.sched01[0] >> ((wg >> 2) * 8)) & 127u), c1 = (int)((p.sched01[1] >> ((wg >> 2) * 8)) & 127u);
  { TILE_ARGS(c0, p0, 0); asm volatile(TLOAD :: TLOAD_IN : "memory"); }
  unsigned done0;
  asm volatile("global_load_dword %0, %1, %2 sc1" : "=v"(done0) : "v"(CTL_DONE), "s"(ctl) : "memory");
  { TILE_ARGS(c1, p1, 1); asm volatile(TLOAD :: TLOAD_IN : "memory"); }
  __amdgpu_buffer_rsrc_t rs[7];
  const int rot = (wg + wv) % 7;
#pragma unroll
  for (int j = 0; j < 7; ++j)
    rs[j] = __builtin_amdgcn_make_buffer_rsrc(p.peer[rot + j] + DATA_OFF + (size_t)p.rank * 4 * ENT_B, 0, SLOT_B + 4 * ENT_B, 0x00020000);
  if (threadIdx.x < 7) ((LAS unsigned long long*)(tick + 8))[threadIdx.x] = (unsigned long long)p.peer[threadIdx.x];
  unsigned open = p.pubmask;
  const int fine_start = p.fine_start, rank = p.rank;
  asm volatile("s_waitcnt vmcnt(0) lgkmcnt(0)" : "+v"(done0) :: "memory");
  BAR;
  const unsigned call = __builtin_amdgcn_readfirstlane(done0) / NWG + 1u;
  const unsigned ao = CTL_TICK + (call & 1u) * 128u, one = 1u;
  const unsigned sbase = (call & 1u) * SLOT_B + lane * 16;
  for (int r = 0;; ++r) {
    u32x4 cur; unsigned so = 0, skip = 0;
    if (p0 < 512) {
      const int c = c0;
      cur = unit_read((c >> 4) & 1, lds + (r & 3) * 8192, wv, lane);
      so = unit_dst_off((c >> 4) & 1, c >> 5, (c & 15) * 4 + (p0 & 3), wv) + sbase;
      if ((c & 31) == 15) {
        const int d = ((p0 & 3) * 2 + (wv >> 2) - rank - 1) & (NR - 1);
        skip = d == 7 ? 0x7fu : 0x7fu & ~(1u << ((d + 7 - rot) % 7));
      }
#pragma unroll
      for (int j = 0; j < WINM1; ++j) PUSH1(j);
      asm volatile("s_waitcnt vmcnt(" STR(WINM1) ")" ::: "memory");
      if (skip) WAIT_V(0);
      if (wv == 0) {
        unsigned aret;
        if (r & 1) GOT("v254"); else GOT("v255");
        if (lane == 0) tick[r & 1] = aret;
        WAIT_L(0);
      }
    } else {
      WAIT_V(0);
    }
    BAR;
    if (r == 0) {
      if (threadIdx.x == 128) COUNT_IN(ctl);
      if (wg == 0 && threadIdx.x == 64)
        __hip_atomic_store((unsigned*)(ctl + CTL_TICK + ((call & 1u) ^ 1u) * 128u), 0u, __ATOMIC_RELAXED, __HIP_MEMORY_SCOPE_AGENT);
    } else {
      GAS uint8_t* const pctl = (GAS uint8_t*)((LAS unsigned long long*)(tick + 8))[lane < 7 ? lane : 0];
      if ((pm1 >> 4) >= fine_start) {
        if (wv == 1 + (r & 3) && lane < 7)
          __hip_atomic_store((GAS unsigned*)(pctl + rflag_off(rank, pm1)), call * 16u, __ATOMIC_RELAXED, __HIP_MEMORY_SCOPE_SYSTEM);
      }
      unsigned done = (p0 >> 4) >= 32 ? open : open & ((1u << (p0 >> 4)) - 1u);
      open &= ~done;
      for (; done; done &= done - 1u)
        if (wv == 1 + (r & 3) && lane < 7)
          __hip_atomic_fetch_add((GAS unsigned*)(pctl + uflag_off(rank, __builtin_ctz(done))), 1u, __ATOMIC_RELAXED, __HIP_MEMORY_SCOPE_SYSTEM);
    }
    if (p0 >= 512) break;
    if (r > 0) p2 = p1 < 512 ? 48 + __builtin_amdgcn_readfirstlane(tick[r & 1]) : 512;
#pragma unroll
    for (int j = WINM1; j < 7; ++j) PUSH1(j);
    const int pn = p2 < 512 ? p2 : 511, c2 = code_of(pn);
    { TILE_ARGS(c2, pn, r + 2); asm volatile(TLOAD :: TLOAD_IN : "memory"); }
    if (wv == 0) { if (r & 1) DRAW("v255"); else DRAW("v254"); }
    pm1 = p0; p0 = p1; p1 = p2; c0 = c1; c1 = c2;
  }
  WAIT_V(0);
  if (wg == 0 && wv == 0) {
    const unsigned* cnt = (const unsigned*)(ctl + (lane & 1 ? CTL_TMO : CTL_SLICE));
    const unsigned long long t0 = __builtin_amdgcn_s_memrealtime();
    for (;;) {
      const unsigned v = __hip_atomic_load(cnt, __ATOMIC_RELAXED, __HIP_MEMORY_SCOPE_SYSTEM);
      if ((int)((unsigned)__builtin_amdgcn_readlane(v, 0) - call * (7u * 32u)) >= 0) break;
      if ((unsigned)__builtin_amdgcn_readlane(v, 1) == call || __builtin_amdgcn_s_memrealtime() - t0 > DEADLINE_TICKS) {
        if (lane == 0) {
          __hip_atomic_store((unsigned*)(ctl + CTL_TMO), call, __ATOMIC_RELAXED, __HIP_MEMORY_SCOPE_AGENT);
          p.out[0] = __builtin_nanf("");
        }
        break;
      }
      __builtin_amdgcn_s_sleep(4);
    }
  }
}

#define DMA2(m0v, soff) asm volatile("s_mov_b32 m0, %0\n\ts_nop 0\n\t" \
    "buffer_load_dwordx4 %1, %2, %3 offen " DMA_POL " lds\n\t" \
    "buffer_load_dwordx4 %1, %2, %3 offen offset:1024 " DMA_POL " lds" \
    :: "s"(m0v), "v"(voff), "s"(rdata), "s"(soff) : "memory")
#define DMA3(m0v, soff, m0h, soffh) asm volatile("s_mov_b32 m0, %0\n\ts_nop 0\n\t" \
    "buffer_load_dwordx4 %1, %2, %3 offen " DMA_POL " lds\n\t" \
    "buffer_load_dwordx4 %1, %2, %3 offen offset:1024 " DMA_POL " lds\n\t" \
    "s_mov_b32 m0, %4\n\ts_mov_b32 exec_hi, 0\n\t" \
    "buffer_load_dwordx4 %1, %2, %5 offen " DMA_POL " lds\n\t" \
    "s_mov_b32 exec_hi, -1" \
    :: "s"(m0v), "v"(voff), "s"(rdata), "s"(soff), "s"(m0h), "s"(soffh) : "memory")

#define DMA1(m0v, soff) asm volatile("s_mov_b32 m0, %0\n\ts_nop 0\n\tbuffer_load_dwordx4 %1, %2, %3 offen " DMA_POL " lds" \
    :: "s"(m0v), "v"(voff), "s"(rdata), "s"(soff) : "memory")
#define DMA1B(m0v, soff) asm volatile("s_mov_b32 m0, %0\n\ts_nop 0\n\tbuffer_load_dwordx4 %1, %2, %3 offen offset:1024 " DMA_POL " lds" \
    :: "s"(m0v), "v"(voff), "s"(rdata), "s"(soff) : "memory")
#define DMA1H(m0v, soff) asm volatile("s_mov_b32 m0, %0\n\ts_mov_b32 exec_hi, 0\n\tbuffer_load_dwordx4 %1, %2, %3 offen " DMA_POL " lds\n\t" \
    "s_mov_b32 exec_hi, -1" :: "s"(m0v), "v"(voff), "s"(rdata), "s"(soff) : "memory")
#define PIN(x) asm volatile("" : "+s"(x))

__device__ __forceinline__ int ready_from(unsigned long long okmask, int base) {
  const int r = base + (int)((unsigned)__builtin_ctzll(~okmask | (1ull << 63)) * 57u >> 9);
  return r < 32 ? r : 32;
}

__device__ __forceinline__ void gemm_role(const Params& p, LAS unsigned char* lds, int g, int wv, int lane) {
  const int lin = (g & 7) * 30 + (g >> 3), band = lin / 80, cb = (lin % 80) / 5;
  const int tcol = (band & 1) ? 15 - cb : cb, trow = band * 5 + lin % 5;
  uint8_t* const ctl = p.ctl;
  __builtin_amdgcn_s_sleep(40);
#pragma unroll
  for (int i = 0; i < 3; ++i) {
    const int gid = g + i * NGEMM, type = (gid >> 8) & 1, kt = (gid >> 6) & 3;
    if (gid < 512) {
      const __amdgpu_buffer_rsrc_t rl = __builtin_amdgcn_make_buffer_rsrc((void*)(type ? p.w : p.x), 0, MN * KS, 0x00020000);
      const unsigned lo = tile_src_off(type, kt, gid & 63, wv, lane), lm = (unsigned)(uintptr_t)lds + i * 8192 + wv * 1024;
      asm volatile(TLOAD :: TLOAD_IN : "memory");
    }
  }
  const unsigned call = __builtin_amdgcn_readfirstlane(
      __hip_atomic_load((const unsigned*)(ctl + CTL_DONE), __ATOMIC_RELAXED, __HIP_MEMORY_SCOPE_AGENT)) / NWG + 1u;
  const __amdgpu_buffer_rsrc_t rdata =
      __builtin_amdgcn_make_buffer_rsrc(ctl + DATA_OFF + (size_t)(call & 1u) * SLOT_B, 0, SLOT_B, 0x00020000);
  WAIT_V(0);
  BAR;
#pragma unroll
  for (int i = 0; i < 3; ++i) {
    const int gid = g + i * NGEMM, type = (gid >> 8) & 1, kt = (gid >> 6) & 3;
    if (gid < 512)
      __builtin_amdgcn_raw_buffer_store_b128(unit_read(type, lds + i * 8192, wv, lane), rdata,
                                             (unsigned)p.rank * 4u * ENT_B + unit_dst_off(type, kt, gid & 63, wv) + lane * 16, 0, 17);
  }
  LAS unsigned* ord = (LAS unsigned*)(lds + LDS_ORD);
  LAS unsigned* tbl = (LAS unsigned*)(lds + LDS_TBL);
  LAS unsigned* bc = (LAS unsigned*)(lds + LDS_CTL);
  const int hj = (trow / 5 + 2 - tcol % 3) % 3;
  const bool host = trow % 5 == 0 && hj < 2;
  auto group_of = [&](int i) -> int { return i < 4 ? 4 * trow + i : i < 8 ? 4 * tcol + i - 4 : host ? 60 + (p.rank >> 1) : 4 * trow; };
  auto pub_pos = [&](int s, int kt, int i) -> int {
    const int type = (i >> 2) & 1, gg = group_of(i);
    const int pos = p.inv[s][kt * 32 + type * 16 + (gg >> 2)] * 4 + (gg & 3), r = pos >> 4;
    return r >= p.fine_start ? pos : (r + __builtin_ctz(p.pubmask >> r)) * 16 + 15;
  };
  if (wv == 0) {
    const int e = lane & 31, s = e >> 2, kt = e & 3;
    int key = kt;
    if (s != p.rank) {
      int m = 0;
      for (int i = 0; i < 9; ++i) { const int r = pub_pos(s, kt, i); m = r > m ? r : m; }
      key = 4 + m * 32 + kt * 8 + ((s - p.rank + g) & 7);
    }
    int rk = 0;
    for (int l = 0; l < 32; ++l) rk += __builtin_amdgcn_readlane(key, l) < key;
    if (lane < 32) ord[rk] = e;
  }
  WAIT_V(0);
  __syncthreads();
  if (threadIdx.x == 64) COUNT_IN(ctl);
  if (threadIdx.x == 0)
    __hip_atomic_store((unsigned*)(ctl + CTL_LFLAG) + g, call * 16u, __ATOMIC_RELAXED, __HIP_MEMORY_SCOPE_SYSTEM);
  if (threadIdx.x < 352) {
    int t = threadIdx.x / 9; const int i = threadIdx.x - t * 9; t = t < 32 ? t : 31;
    const int e = ord[t], s = e >> 2, kt = e & 3, type = (i >> 2) & 1, gg = group_of(i);
    unsigned off;
    if (s == p.rank) off = CTL_LFLAG + 4u * (unsigned)((type * 256 + kt * 64 + gg) % NGEMM);
    else { const int pp = pub_pos(s, kt, i);
           off = (pp >> 4) >= p.fine_start ? rflag_off(s, pp) : uflag_off(s, pp >> 4); }
    tbl[threadIdx.x] = off;
  }
  const int ordv = ord[lane & 31];
  ((LAS unsigned long long*)(lds + LDS_SNAP))[threadIdx.x] = 0ull;
  ((LAS unsigned*)(lds + LDS_SNAP + 4096))[threadIdx.x] = 0u;
  if (threadIdx.x == 0) bc[63] = 0u;
  if (threadIdx.x < 7) ((LAS unsigned long long*)(bc + 32))[threadIdx.x] = (unsigned long long)(p.peer[threadIdx.x] + CTL_SLICE);
  int scale_bits = __builtin_amdgcn_readfirstlane(__builtin_bit_cast(int, p.sx[0] * p.sw[0]));
  asm volatile("" : "+s"(scale_bits));
  WAIT_V(0);
  __syncthreads();

  const int wr = wv >> 2, wc = wv & 3;
  const unsigned voff = lane * 16;
  const unsigned ldsb = (unsigned)(uintptr_t)lds;
  const unsigned a0o = (9 * wr + wc) * 2048, a8o = (9 * wr + 8) * 2048 + wc * 512, b0o = (4 * (wv >> 1) + (wv & 1)) * 2048;
  const bool has9 = host && wr == 0;
  const unsigned ga0 = (unsigned)(16 * trow + 8 * wr + wc) * 2048u, gb0 = OP_B + tcol * 32768u + b0o;
  const unsigned ga8 = has9 ? (unsigned)(240 + 2 * p.rank + hj) * 2048u + wc * 512u : OOB;
  const unsigned la0 = ldsb + LDS_A + a0o, la8 = ldsb + LDS_A + a8o, lb0 = ldsb + LDS_B + b0o;
#define ENT(t) ((unsigned)__builtin_amdgcn_readlane(ordv, (t)) * ENT_B)
#define ST_A03(eb, sl) DMA2(la0 + (sl) * A_SLOT, (eb) + ga0)
#define ST_A48(eb, sl) DMA3(la0 + 8192 + (sl) * A_SLOT, (eb) + ga0 + 8192, la8 + (sl) * A_SLOT, (eb) + ga8)
#define ST_B0(eb, sl) DMA2(lb0 + (sl) * B_SLOT, (eb) + gb0)
#define ST_B1(eb, sl) DMA2(lb0 + 4096 + (sl) * B_SLOT, (eb) + gb0 + 4096)
#define ST_ALL(eb, sl) do { ST_B0(eb, sl); ST_B1(eb, sl); ST_A03(eb, sl); ST_A48(eb, sl); } while (0)
#define LDF(dst, base, off) do { const i32x4 _l = *(const LAS i32x4*)((base) + (off)), _h = *(const LAS i32x4*)((base) + (off) + 1024); \
    dst = __builtin_shufflevector(_l, _h, 0, 1, 2, 3, 4, 5, 6, 7); SCHED; } while (0)
#define MM(m, n, Af) do { acc[m][n] = __builtin_amdgcn_mfma_scale_f32_16x16x128_f8f6f4(Af, Bf[n], acc[m][n], 1, 1, 0, 0, 0, 0); SCHED; } while (0)

  f32x4 acc[9][4];
#pragma unroll
  for (int m = 0; m < 9; ++m)
#pragma unroll
    for (int n = 0; n < 4; ++n) acc[m][n] = (f32x4){0.f, 0.f, 0.f, 0.f};
  i32x8 a0, a1, a2, a3, a4, Bf[4];
  int t = 0, ready = 0, nrestart = 0;
  unsigned dead = 0;
  const unsigned thr = call * 16u;
  const unsigned pollo = tbl[threadIdx.x < 352 ? threadIdx.x : 351];
  const __amdgpu_buffer_rsrc_t rctl = __builtin_amdgcn_make_buffer_rsrc(ctl, 0, CTL_BYTES, 0x00020000);
  const __amdgpu_buffer_rsrc_t rout = __builtin_amdgcn_make_buffer_rsrc(p.out, 0, MN * MN * 4, 0x00020000);
  const LAS unsigned char* const pa0 = lds + LDS_A + wr * (9 * 2048) + lane * 16;
  const LAS unsigned char* const pb0 = lds + LDS_B + wc * (4 * 2048) + lane * 16;
#define POLL(buf) do { const unsigned _po = (unsigned)(ready * 9 + 62 - wv * 64) < 126u && ready < 32 ? 0u : OOB; \
    asm volatile("s_mov_b32 m0, %0\n\ts_nop 0\n\tbuffer_load_dword %1, %2, %3 offen " DMA_POL " lds" \
                 :: "s"(ldsb + LDS_SNAP + (buf) * 2048 + wv * 256), "v"(pollo), "s"(rctl), "s"(_po) : "memory"); } while (0)
#define SNAP_AT(buf) ((const LAS unsigned*)(lds + LDS_SNAP + ((buf) * 512 + ready * 9) * 4) + lane)
#define OKMASK(v) __builtin_amdgcn_ballot_w64((int)((v) - thr) >= 0)
#define STAGE_FROM(u) do { const unsigned _e0 = ENT(u); ST_ALL(_e0, (u) & 1); sn = ready > (u) + 1; \
    if (sn) { BAR;   const unsigned _e1 = ENT((u) + 1); ST_ALL(_e1, ((u) & 1) ^ 1); } \
    POLL(((u) & 1) ^ 1); } while (0)
  int sn = -1;
  for (;;) {
    if (sn < 0) {
      if (ready <= t) {
        if (wv == 0) {
          const unsigned* fp = (const unsigned*)(ctl + (lane == 63 ? CTL_TMO : tbl[t * 9 + lane]));
          unsigned* tmo = (unsigned*)(ctl + CTL_TMO);
          const unsigned long long t0 = __builtin_amdgcn_s_memrealtime();
          int r; unsigned dd = 0;
          for (;;) {
            const unsigned v = __hip_atomic_load(fp, __ATOMIC_RELAXED, __HIP_MEMORY_SCOPE_SYSTEM);
            r = ready_from(OKMASK(v), t);
            if (r > t) break;
            if ((unsigned)__builtin_amdgcn_readlane(v, 63) == call || __builtin_amdgcn_s_memrealtime() - t0 > DEADLINE_TICKS) {
              if (lane == 0) __hip_atomic_store(tmo, call, __ATOMIC_RELAXED, __HIP_MEMORY_SCOPE_AGENT);
              r = 32; dd = 64; break;
            }
            __builtin_amdgcn_s_sleep(2);
          }
          if (lane == 0) bc[nrestart & 31] = (unsigned)r | (dd << 8);
          asm volatile("s_waitcnt vmcnt(0) lgkmcnt(0)" ::: "memory");
        }
        BAR;
        const unsigned v = __builtin_amdgcn_readfirstlane(bc[nrestart & 31]);
        ready = v & 63; dead |= v >> 8; ++nrestart;
      }
      STAGE_FROM(t);
    }
    int s1 = sn;
    sn = -1;
    if (s1) WAIT_V(13); else WAIT_V(4);
    BAR;
    { const LAS unsigned char* pa = pa0 + (t & 1) * A_SLOT; const LAS unsigned char* pb = pb0 + (t & 1) * B_SLOT;
      LDF(Bf[0], pb, 0); LDF(a0, pa, 0); LDF(a1, pa, 2048); LDF(Bf[1], pb, 2048); LDF(Bf[2], pb, 4096); LDF(a2, pa, 4096); LDF(Bf[3], pb, 6144); }
    if (t == 31) break;
    unsigned en2 = ENT((t + 2) & 31);
    int first = 1;
    for (;;) {
      const int sl = t & 1;
      const LAS unsigned char* pa = pa0 + sl * A_SLOT;
      LDF(a3, pa, 3 * 2048);
      MM(0, 0, a0);
      const int rich = (ready > t + 2 || ready == 32 || (first & s1)) ? 1 : 0;
      const LAS unsigned* psn = SNAP_AT(sl ^ rich ^ 1);
      asm volatile("" : "+v"(psn)); SCHED;
      MM(1, 0, a1);
      const LAS unsigned char* pan = pa0 + (sl ^ 1) * A_SLOT; const LAS unsigned char* pbn = pb0 + (sl ^ 1) * B_SLOT;
      asm volatile("" : "+v"(pan), "+v"(pbn)); SCHED;
      MM(0, 1, a0);
      unsigned mB = lb0 + sl * B_SLOT, mA = la0 + sl * A_SLOT;
      PIN(mB); PIN(mA); SCHED;
      MM(1, 1, a1);
      unsigned mA8 = la8 + sl * A_SLOT, mP = ldsb + LDS_SNAP + sl * 2048 + wv * 256;
      PIN(mA8); PIN(mP); SCHED;
      MM(0, 2, a0); MM(1, 2, a1);
      WAIT_V(10); if (!rich) { if (first) WAIT_V(0); else WAIT_V(3); }
      first = 0;
      WAIT_L(0); BAR; SCHED;
      const unsigned snap = *psn;
      SCHED;
      MM(0, 3, a0);
      LDF(a4, pa, 6 * 2048);
      MM(1, 3, a1);
      LDF(a0, pa, 4 * 2048);
      MM(2, 0, a2);
      unsigned long long okm = OKMASK(snap);
      SCHED;
      MM(2, 1, a2);
      LDF(a1, pa, 5 * 2048);
      unsigned fz = (unsigned)__builtin_ctzll(~okm | (1ull << 63)) * 57u;
      PIN(fz); SCHED;
      MM(2, 2, a2);
      ready += (int)(fz >> 9); ready = ready < 32 ? ready : 32;
      if (__builtin_expect(!s1, 0)) {
        if (ready > t + 1) { const unsigned e1 = ENT(t + 1); ST_ALL(e1, sl ^ 1); s1 = 1; }
        POLL(2);
      }
      SCHED;
      MM(2, 3, a2);
      int s2 = ready > t + 2 ? s1 : 0;
      unsigned e2 = s2 ? en2 : OOB;
      PIN(s2); PIN(e2);
      LDF(a2, pa, 8 * 2048);
      MM(3, 0, a3);
      DMA1(mB, e2 + gb0); SCHED;
      MM(3, 1, a3);
      DMA1B(mB, e2 + gb0); SCHED;
      MM(3, 2, a3);
      DMA1(mB + 4096, e2 + gb0 + 4096); SCHED;
      MM(3, 3, a3);
      LDF(a3, pa, 7 * 2048);
      DMA1B(mB + 4096, e2 + gb0 + 4096); SCHED;
      MM(4, 0, a0);
      DMA1(mA, e2 + ga0); SCHED;
      MM(4, 1, a0);
      DMA1B(mA, e2 + ga0); SCHED;
      MM(4, 2, a0);
      en2 = ENT((t + 3) & 31);
      SCHED;
      MM(4, 3, a0);
      unsigned po = (unsigned)(ready * 9 + 62 - wv * 64) < 126u && ready < 32 ? 0u : OOB;
      PIN(po); SCHED;
      MM(5, 0, a1); MM(5, 1, a1);
      WAIT_V(10); if (__builtin_expect(!s1, 0)) WAIT_V(0);
      WAIT_L(0); BAR; SCHED;
      MM(5, 2, a1);
      LDF(a0, pan, 0);
      asm volatile("s_mov_b32 m0, %0\n\ts_nop 0\n\tbuffer_load_dword %1, %2, %3 offen " DMA_POL " lds"
                   :: "s"(mP), "v"(pollo), "s"(rctl), "s"(po) : "memory");
      SCHED;
      MM(5, 3, a1);
      LDF(a1, pan, 2048);
      DMA1(mA + 8192, e2 + ga0 + 8192); SCHED;
      if (has9) { MM(8, 0, a2); MM(8, 1, a2); MM(8, 2, a2); MM(8, 3, a2); }
      LDF(a2, pan, 4096);
      DMA1B(mA + 8192, e2 + ga0 + 8192); SCHED;
      MM(7, 0, a3);
      DMA1H(mA8, e2 + ga8);
      if (__builtin_expect(!s1, 0)) {
        ready = ready_from(OKMASK(*SNAP_AT(2)), ready);
        if (ready > t + 1) STAGE_FROM(t + 1);
      }
      SCHED;
      MM(6, 0, a4);
      LDF(Bf[0], pbn, 0);
      MM(7, 1, a3); MM(6, 1, a4);
      LDF(Bf[1], pbn, 2048);
      MM(7, 2, a3); MM(6, 2, a4);
      LDF(Bf[2], pbn, 4096);
      MM(7, 3, a3); MM(6, 3, a4);
      LDF(Bf[3], pbn, 6144);
      ++t;
      if (!s1 || t == 31) break;
      s1 = s2;
    }
    if (s1) break;
    if (sn < 0) {
      asm volatile("s_waitcnt vmcnt(0) lgkmcnt(0)" ::: "memory"); BAR;
      ready = ready_from(OKMASK(*SNAP_AT((t & 1) ^ 1)), ready);
      WAIT_L(0); BAR;
    }
  }
  {
    const float sc = __builtin_bit_cast(float, dead ? 0x7fc00000 : scale_bits);
    const unsigned ov = (unsigned)((lane >> 4) * 4 * MN + (lane & 15) * 4) * 4u;
    const unsigned os = (unsigned)((trow * 256 + wr * 128) * MN + tcol * 256 + wc * 64) * 4u;
#define OUT(m) do { f32x4 o[4]; \
      asm volatile("" : "+v"(acc[m][0]), "+v"(acc[m][1]), "+v"(acc[m][2]), "+v"(acc[m][3])); \
      _Pragma("unroll") for (int j = 0; j < 4; ++j) { o[j][0] = acc[m][0][j] * sc; o[j][1] = acc[m][1][j] * sc; o[j][2] = acc[m][2][j] * sc; o[j][3] = acc[m][3][j] * sc; } \
      SCHED; \
      _Pragma("unroll") for (int j = 0; j < 4; ++j) \
        __builtin_amdgcn_raw_buffer_store_b128(__builtin_bit_cast(u32x4, o[j]), rout, ov, os + (unsigned)((m) * 16 + j) * (MN * 4u), OUT_AUX); \
      asm volatile("s_nop 1" ::: "memory");                     \
      SCHED; } while (0)
    const LAS unsigned char* pa = pa0 + A_SLOT;
    LDF(a3, pa, 3 * 2048);
    MM(0, 0, a0); MM(1, 0, a1); MM(0, 1, a0); MM(1, 1, a1); MM(0, 2, a0); MM(1, 2, a1);
    WAIT_V(0); BAR; SCHED;
    MM(0, 3, a0);
    LDF(a4, pa, 8 * 2048);
    MM(1, 3, a1);
    LDF(a0, pa, 4 * 2048);
    MM(2, 0, a2); MM(2, 1, a2);
    LDF(a1, pa, 5 * 2048);
    MM(2, 2, a2); MM(2, 3, a2);
    LDF(a2, pa, 6 * 2048);
    f32x4 o8[4];
    const unsigned os8 = (unsigned)((3840 + 32 * p.rank + 16 * hj) * MN + tcol * 256 + wc * 64) * 4u;
#define SHARE(d) do { if (has9) { \
      const __amdgpu_buffer_rsrc_t rd = __builtin_amdgcn_make_buffer_rsrc(p.outs[(p.rank + 1 + ((d) < 7 ? ((d) + tcol + wc) % 7 : 7)) & (NR - 1)], 0, MN * MN * 4, 0x00020000); \
      asm volatile("s_waitcnt vmcnt(" STR(SHARE_WIN) ")" ::: "memory"); \
      _Pragma("unroll") for (int j = 0; j < 4; ++j) \
        __builtin_amdgcn_raw_buffer_store_b128(__builtin_bit_cast(u32x4, o8[j]), rd, ov, os8 + (unsigned)j * (MN * 4u), 17); } \
      SCHED; } while (0)
    if (has9) { MM(8, 0, a4); MM(8, 1, a4); MM(8, 2, a4); MM(8, 3, a4); }
    OUT(0);
#pragma unroll
    for (int j = 0; j < 4; ++j) { o8[j][0] = acc[8][0][j] * sc; o8[j][1] = acc[8][1][j] * sc; o8[j][2] = acc[8][2][j] * sc; o8[j][3] = acc[8][3][j] * sc; }
    SCHED;
    SHARE(0);
    MM(3, 0, a3); MM(3, 1, a3); MM(3, 2, a3); MM(3, 3, a3);
    LDF(a3, pa, 7 * 2048);
    OUT(1); SHARE(1);
    MM(4, 0, a0); MM(4, 1, a0); MM(4, 2, a0); MM(4, 3, a0);
    OUT(2); SHARE(2);
    MM(5, 0, a1); MM(5, 1, a1); MM(5, 2, a1); MM(5, 3, a1);
    OUT(3); SHARE(3);
    MM(6, 0, a2); MM(6, 1, a2); MM(6, 2, a2); MM(6, 3, a2);
    OUT(4); SHARE(4);
    MM(7, 0, a3); MM(7, 1, a3); MM(7, 2, a3); MM(7, 3, a3);
    OUT(5); SHARE(5); OUT(6); SHARE(6); OUT(7); SHARE(7);
    if (has9) {
      WAIT_V(8);
      unsigned old = 0;
      if (lane == 0) old = __hip_atomic_fetch_add(bc + 63, 1u, __ATOMIC_RELAXED, __HIP_MEMORY_SCOPE_WORKGROUP);
      if (__builtin_amdgcn_readfirstlane(old) == 3u && lane < 7)
        __hip_atomic_fetch_add((GAS unsigned*)((LAS unsigned long long*)(bc + 32))[lane], 1u, __ATOMIC_RELAXED, __HIP_MEMORY_SCOPE_SYSTEM);
    }
  }
}

__global__ void __launch_bounds__(512, 2) gemm_ag_kernel(const Params p) {
  __shared__ __attribute__((aligned(16))) unsigned char lds_raw[LDS_BYTES];
  LAS unsigned char* lds = (LAS unsigned char*)lds_raw;
  const int wv = __builtin_amdgcn_readfirstlane(threadIdx.x >> 6), lane = threadIdx.x & 63;
  if (blockIdx.x < NCOMM) comm_role(p, lds, blockIdx.x, wv, lane);
  else gemm_role(p, lds, blockIdx.x - NCOMM, wv, lane);
}

static void make_schedule(int s, uint8_t* sc) {
  static const int nlate[5] = {0, 3, 5, 8, 8};
  static const double fin[5] = {0.0, 3.0, 5.2, 7.6, 7.6};
  int r = 0;
  for (int kt = 0; kt < 3; ++kt)
    for (int q = 0; q < 16; ++q) {
      sc[r++] = (uint8_t)(kt * 32 + 16 + (q + 2 * s) % 16);
      sc[r++] = (uint8_t)(kt * 32 + (q + 2 * s) % 16);
    }
  double due[16]; int late[16], nl = 0;
  sc[r++] = (uint8_t)(96 + 15);
  for (int c = 0; c < 15; ++c) {
    const int lv = c % 5, i = ((s - (lv * 3 + c / 5) * 3) % 8 + 8) % 8;
    if (i < nlate[lv]) { due[nl] = fin[lv] - (nlate[lv] - i) * 1.15 + 0.01 * (c / 5); late[nl++] = c; }
    else sc[r++] = (uint8_t)(96 + c);
  }
  for (int q = 0; q < 16; ++q) sc[r++] = (uint8_t)(96 + 16 + (q + 2 * s) % 16);
  for (int k = 0; k < nl; ++k) {
    int best = -1;
    for (int j = 0; j < nl; ++j) if (late[j] >= 0 && (best < 0 || due[j] < due[best])) best = j;
    sc[r++] = (uint8_t)(96 + late[best]); late[best] = -1;
  }
}

}

extern "C" void kernel_launch(int rank, int nranks, ncclComm_t comm,
                              hipStream_t stream,
                              void* const* d_in, const int* in_sizes, int n_in,
                              void* d_out, int out_size,
                              const int* in_dtypes, int out_dtype,
                              void* const* d_out_all,
                              void* const* d_ws_all, size_t ws_bytes) {
  (void)comm; (void)in_dtypes; (void)out_dtype;
  if (nranks != NR || n_in != 4 || (unsigned)rank >= (unsigned)NR || ws_bytes < WS_NEED || out_size != MN * MN ||
      in_sizes[0] != MN * KS || in_sizes[1] != MN * KS) {
    fprintf(stderr, "kernel_launch: unexpected problem shape\n");
    return;
  }
  Params p;
  p.x = (const uint8_t*)d_in[0]; p.w = (const uint8_t*)d_in[1];
  p.sx = (const float*)d_in[2]; p.sw = (const float*)d_in[3];
  p.out = (float*)d_out;
  for (int i = 0; i < NR; ++i) p.outs[i] = (float*)d_out_all[i];
  p.ctl = (uint8_t*)d_ws_all[rank];
  for (int i = 0; i < 14; ++i) p.peer[i] = (uint8_t*)d_ws_all[(rank + 1 + i % 7) % NR];
  p.rank = rank;
  p.pubmask = 0x00AA8080u;
  p.fine_start = 24;
  for (int s = 0; s < NR; ++s) {
    uint8_t sc[128];
    make_schedule(s, sc);
    for (int r = 0; r < 128; ++r) p.inv[s][sc[r]] = (uint8_t)r;
    if (s == rank)
      for (int i = 0; i < 32; ++i)
        p.sched[i] = sc[4 * i] | (sc[4 * i + 1] << 8) | (sc[4 * i + 2] << 16) | ((unsigned)sc[4 * i + 3] << 24);
  }
  p.sched01[0] = p.sched[0]; p.sched01[1] = p.sched[1];
  hipLaunchKernelGGL(gemm_ag_kernel, dim3(NWG), dim3(512), 0, stream, p);
  HIP_CHECK(hipGetLastError());
}
```

```cpp
#include <hip/hip_runtime.h>
#include <rccl/rccl.h>
#include <cstdio>
#include <cstdint>

#define HIP_CHECK(expr) do { hipError_t _e = (expr); if (_e != hipSuccess) { \
    fprintf(stderr, "HIP error %s at %s:%d\n", hipGetErrorString(_e), __FILE__, __LINE__); \
    return; } } while (0)

#define DMA_POL "sc0 sc1"
#define TOUCH_MASK 0x22
#define GDELAY 127
#define TOUCH_WGS 8
#define WINM1 4
#define SHARE_WIN 6
#define OUT_AUX 16

namespace {

#define LAS __attribute__((address_space(3)))
#define GAS __attribute__((address_space(1)))
typedef unsigned u32x4 __attribute__((ext_vector_type(4)));
typedef int i32x4 __attribute__((ext_vector_type(4)));
typedef int i32x8 __attribute__((ext_vector_type(8)));
typedef float f32x4 __attribute__((ext_vector_type(4)));

constexpr int NR = 8;
constexpr int MN = 4096;
constexpr int KS = 512;
constexpr int NCOMM = 16;
constexpr int NGEMM = 240;
constexpr int NWG = NCOMM + NGEMM;
constexpr unsigned ENT_B = 1u << 20;
constexpr unsigned OP_B = 1u << 19;
constexpr unsigned SLOT_B = 32u << 20;
constexpr size_t DATA_OFF = 1u << 20;
constexpr size_t WS_NEED = DATA_OFF + 2ull * SLOT_B;
constexpr unsigned CTL_DONE = 0x0000, CTL_TMO = 0x0080, CTL_TICK = 0x0200, CTL_SLICE = 0x0400, CTL_LFLAG = 0x1000,
                   CTL_UFLAG = 0x8000, CTL_RFLAG = 0x10000, CTL_BYTES = 0x30000;
__host__ __device__ constexpr unsigned uflag_off(int s, int unit) { return CTL_UFLAG + (unsigned)(s * 32 + unit) * 128u; }
__host__ __device__ constexpr unsigned rflag_off(int s, int pos) { return CTL_RFLAG + (unsigned)(s * 128 + (pos >> 2)) * 128u + (unsigned)(pos & 3) * 4u; }
constexpr unsigned OOB = 0x40000000u;
constexpr unsigned long long DEADLINE_TICKS = 100000000ull;

constexpr int A_SLOT = 18 * 2048, B_SLOT = 16 * 2048;
constexpr int LDS_A = 0, LDS_B = 2 * A_SLOT;
constexpr int LDS_TBL = LDS_B + 2 * B_SLOT;
constexpr int LDS_SNAP = LDS_TBL + 352 * 4;
constexpr int LDS_ORD = LDS_SNAP + 6144;
constexpr int LDS_CTL = LDS_ORD + 128;
constexpr int LDS_BYTES = LDS_CTL + 256;

struct Params {
  const uint8_t* x; const uint8_t* w; uint8_t* ctl;
  unsigned sched01[2];
  uint8_t* peer[14];
  const float* sx; const float* sw; float* out;
  float* outs[NR];
  int rank;
  unsigned pubmask;
  int fine_start;
  unsigned sched[32];
  uint8_t inv[NR][128];
};

#define COUNT_IN(ctl) __hip_atomic_fetch_add((unsigned*)((ctl) + CTL_DONE), 1u, __ATOMIC_RELAXED, __HIP_MEMORY_SCOPE_AGENT)
#define WAIT_V(n) asm volatile("s_waitcnt vmcnt(" #n ")" ::: "memory")
#define WAIT_L(n) asm volatile("s_waitcnt lgkmcnt(" #n ")" ::: "memory")
#define BAR do { asm volatile("" ::: "memory"); __builtin_amdgcn_s_barrier(); asm volatile("" ::: "memory"); } while (0)
#define SCHED __builtin_amdgcn_sched_barrier(0)

__device__ __forceinline__ unsigned tile_src_off(int type, int kt, int g, int wv, int lane) {
  const int row = wv * 8 + (lane >> 3), ch = (lane & 7) * 16;
  return type ? (unsigned)((kt * 128 + (g & 1) * 64 + row) * MN + (g >> 1) * 128 + ch)
              : (unsigned)((g * 64 + row) * KS + kt * 128 + ch);
}
__device__ __forceinline__ unsigned unit_dst_off(int type, int kt, int g, int wv) {
  const int u = type ? ((g >> 1) * 8 + wv) * 2 + (g & 1) : g * 8 + wv;
  return (unsigned)kt * ENT_B + (unsigned)type * OP_B + (unsigned)u * 1024u;
}
__device__ __forceinline__ u32x4 unit_read(int type, const LAS unsigned char* tile, int wv, int lane) {
  if (type) {
    const LAS unsigned* a = (const LAS unsigned*)(tile + (lane >> 4) * 2048 + (wv >> 2) * 64 + (lane & 15) * 4);
    const unsigned n = wv & 3, s1 = n | ((n + 4u) << 8);
    u32x4 r;
#pragma unroll
    for (int i = 0; i < 4; ++i) {
      const unsigned lo = __builtin_amdgcn_perm(a[(4 * i + 1) * 32], a[(4 * i) * 32], s1);
      const unsigned hi = __builtin_amdgcn_perm(a[(4 * i + 3) * 32], a[(4 * i + 2) * 32], s1);
      r[i] = __builtin_amdgcn_perm(hi, lo, 0x05040100u);
    }
    return r;
  }
  return *(const LAS u32x4*)(tile + ((wv >> 1) * 16 + (lane & 15)) * 128 + (wv & 1) * 64 + (lane >> 4) * 16);
}

#define TLOAD "s_mov_b32 m0, %[lm]\n\ts_nop 3\n\tbuffer_load_dwordx4 %[lo], %[rl], 0 offen lds\n\t"
#define TLOAD_IN [lm] "s"(lm), [lo] "v"(lo), [rl] "s"(rl)
#define STR2(x) #x
#define STR(x) STR2(x)
#define PUSH1(j) asm volatile("s_waitcnt vmcnt(" STR(WINM1) ")\n\tbuffer_store_dwordx4 %0, %1, %2, %3 offen sc0 sc1" \
                              :: "v"(cur), "v"(so), "s"(rs[j]), "s"((skip >> j) & 1u ? OOB : 0u) : "memory")
#define DRAW(vnew) asm volatile("s_mov_b64 exec, 1\n\tglobal_atomic_add " vnew ", %0, %1, %2 sc0\n\ts_mov_b64 exec, -1" \
                                :: "v"(ao), "v"(one), "s"(ctl) : "memory", "v254", "v255")
#define GOT(vold) asm volatile("v_mov_b32 %0, " vold : "=v"(aret) :: "memory", "v254", "v255")

__device__ __forceinline__ void comm_role(const Params& p, LAS unsigned char* lds, int wg, int wv, int lane) {
  LAS unsigned* tick = (LAS unsigned*)(lds + LDS_CTL);
  uint8_t* const ctl = p.ctl;
  const unsigned ldsw = (unsigned)(uintptr_t)lds + wv * 1024;
  auto code_of = [&](int pos) -> int { return (int)((p.sched[(pos >> 4) & 31] >> (((pos >> 2) & 3) * 8)) & 127u); };
#define TILE_ARGS(_c, pos, slot) \
    const __amdgpu_buffer_rsrc_t rl = __builtin_amdgcn_make_buffer_rsrc((void*)(((_c) & 16) ? p.w : p.x), 0, MN * KS, 0x00020000); \
    const unsigned lo = tile_src_off(((_c) >> 4) & 1, (_c) >> 5, ((_c) & 15) * 4 + ((pos) & 3), wv, lane), lm = ldsw + ((slot) & 3) * 8192
  int pm1 = 0, p0 = wg, p1 = 16 + wg, p2 = 32 + wg;
  int c0 = (int)((p.sched01[0] >> ((wg >> 2) * 8)) & 127u), c1 = (int)((p.sched01[1] >> ((wg >> 2) * 8)) & 127u);
  { TILE_ARGS(c0, p0, 0); asm volatile(TLOAD :: TLOAD_IN : "memory"); }
  unsigned done0;
  asm volatile("global_load_dword %0, %1, %2 sc1" : "=v"(done0) : "v"(CTL_DONE), "s"(ctl) : "memory");
  { TILE_ARGS(c1, p1, 1); asm volatile(TLOAD :: TLOAD_IN : "memory"); }
  __amdgpu_buffer_rsrc_t rs[7];
  const int rot = (wg + wv) % 7;
#pragma unroll
  for (int j = 0; j < 7; ++j)
    rs[j] = __builtin_amdgcn_make_buffer_rsrc(p.peer[rot + j] + DATA_OFF + (size_t)p.rank * 4 * ENT_B, 0, SLOT_B + 4 * ENT_B, 0x00020000);
  if (wv < 7 && lane == 0) ((LAS unsigned long long*)(tick + 8))[wv] = (unsigned long long)p.peer[wv];
  unsigned open = p.pubmask;
  const int fine_start = p.fine_start, rank = p.rank;
  asm volatile("s_waitcnt vmcnt(0) lgkmcnt(0)" : "+v"(done0) :: "memory");
  BAR;
  const unsigned call = __builtin_amdgcn_readfirstlane(done0) / NWG + 1u;
  const unsigned ao = CTL_TICK + (call & 1u) * 128u, one = 1u;
  const unsigned sbase = (call & 1u) * SLOT_B + lane * 16;
  for (int r = 0;; ++r) {
    u32x4 cur; unsigned so = 0, skip = 0;
    if (p0 < 512) {
      const int c = c0;
      cur = unit_read((c >> 4) & 1, lds + (r & 3) * 8192, wv, lane);
      so = unit_dst_off((c >> 4) & 1, c >> 5, (c & 15) * 4 + (p0 & 3), wv) + sbase;
      if ((c & 31) == 15) {
        const int d = ((p0 & 3) * 2 + (wv >> 2) - rank - 1) & (NR - 1);
        skip = d == 7 ? 0x7fu : 0x7fu & ~(1u << ((d + 7 - rot) % 7));
      }
#pragma unroll
      for (int j = 0; j < WINM1; ++j) PUSH1(j);
      asm volatile("s_waitcnt vmcnt(" STR(WINM1) ")" ::: "memory");
      if (skip) WAIT_V(0);
      if (wv == 0) {
        unsigned aret;
        if (r & 1) GOT("v254"); else GOT("v255");
        if (lane == 0) tick[r & 1] = aret;
        WAIT_L(0);
      }
    } else {
      WAIT_V(0);
    }
    BAR;
    if (r == 0) {
      if (threadIdx.x == 128) COUNT_IN(ctl);
      if (wg == 0 && threadIdx.x == 64)
        __hip_atomic_store((unsigned*)(ctl + CTL_TICK + ((call & 1u) ^ 1u) * 128u), 0u, __ATOMIC_RELAXED, __HIP_MEMORY_SCOPE_AGENT);
    } else {
      GAS uint8_t* const pctl = (GAS uint8_t*)((LAS unsigned long long*)(tick + 8))[lane < 7 ? lane : 0];
      if ((pm1 >> 4) >= fine_start) {
        if (wv == 1 + (r & 3) && lane < 7)
          __hip_atomic_store((GAS unsigned*)(pctl + rflag_off(rank, pm1)), call * 16u, __ATOMIC_RELAXED, __HIP_MEMORY_SCOPE_SYSTEM);
      }
      unsigned done = (p0 >> 4) >= 32 ? open : open & ((1u << (p0 >> 4)) - 1u);
      open &= ~done;
      for (; done; done &= done - 1u)
        if (wv == 1 + (r & 3) && lane < 7)
          __hip_atomic_fetch_add((GAS unsigned*)(pctl + uflag_off(rank, __builtin_ctz(done))), 1u, __ATOMIC_RELAXED, __HIP_MEMORY_SCOPE_SYSTEM);
    }
    if (p0 >= 512) break;
    if (r > 0) p2 = p1 < 512 ? 48 + __builtin_amdgcn_readfirstlane(tick[r & 1]) : 512;
#pragma unroll
    for (int j = WINM1; j < 7; ++j) PUSH1(j);
    const int pn = p2 < 512 ? p2 : 511, c2 = code_of(pn);
    { TILE_ARGS(c2, pn, r + 2); asm volatile(TLOAD :: TLOAD_IN : "memory"); }
    if (wv == 0) { if (r & 1) DRAW("v255"); else DRAW("v254"); }
    pm1 = p0; p0 = p1; p1 = p2; c0 = c1; c1 = c2;
  }
  WAIT_V(0);
  if (wg == 0 && wv == 0) {
    const unsigned* cnt = (const unsigned*)(ctl + (lane & 1 ? CTL_TMO : CTL_SLICE));
    const unsigned long long t0 = __builtin_amdgcn_s_memrealtime();
    for (;;) {
      const unsigned v = __hip_atomic_load(cnt, __ATOMIC_RELAXED, __HIP_MEMORY_SCOPE_SYSTEM);
      if ((int)((unsigned)__builtin_amdgcn_readlane(v, 0) - call * (7u * 32u)) >= 0) break;
      if ((unsigned)__builtin_amdgcn_readlane(v, 1) == call || __builtin_amdgcn_s_memrealtime() - t0 > DEADLINE_TICKS) {
        if (lane == 0) {
          __hip_atomic_store((unsigned*)(ctl + CTL_TMO), call, __ATOMIC_RELAXED, __HIP_MEMORY_SCOPE_AGENT);
          p.out[0] = __builtin_nanf("");
        }
        break;
      }
      __builtin_amdgcn_s_sleep(4);
    }
  }
}

#define DMA2(m0v, soff) asm volatile("s_mov_b32 m0, %0\n\ts_nop 0\n\t" \
    "buffer_load_dwordx4 %1, %2, %3 offen " DMA_POL " lds\n\t" \
    "buffer_load_dwordx4 %1, %2, %3 offen offset:1024 " DMA_POL " lds" \
    :: "s"(m0v), "v"(voff), "s"(rdata), "s"(soff) : "memory")
#define DMA3(m0v, soff, m0h, soffh) asm volatile("s_mov_b32 m0, %0\n\ts_nop 0\n\t" \
    "buffer_load_dwordx4 %1, %2, %3 offen " DMA_POL " lds\n\t" \
    "buffer_load_dwordx4 %1, %2, %3 offen offset:1024 " DMA_POL " lds\n\t" \
    "s_mov_b32 m0, %4\n\ts_mov_b32 exec_hi, 0\n\t" \
    "buffer_load_dwordx4 %1, %2, %5 offen " DMA_POL " lds\n\t" \
    "s_mov_b32 exec_hi, -1" \
    :: "s"(m0v), "v"(voff), "s"(rdata), "s"(soff), "s"(m0h), "s"(soffh) : "memory")

#define DMA1(m0v, soff) asm volatile("s_mov_b32 m0, %0\n\ts_nop 0\n\tbuffer_load_dwordx4 %1, %2, %3 offen " DMA_POL " lds" \
    :: "s"(m0v), "v"(voff), "s"(rdata), "s"(soff) : "memory")
#define DMA1B(m0v, soff) asm volatile("s_mov_b32 m0, %0\n\ts_nop 0\n\tbuffer_load_dwordx4 %1, %2, %3 offen offset:1024 " DMA_POL " lds" \
    :: "s"(m0v), "v"(voff), "s"(rdata), "s"(soff) : "memory")
#define DMA1H(m0v, soff) asm volatile("s_mov_b32 m0, %0\n\ts_mov_b32 exec_hi, 0\n\tbuffer_load_dwordx4 %1, %2, %3 offen " DMA_POL " lds\n\t" \
    "s_mov_b32 exec_hi, -1" :: "s"(m0v), "v"(voff), "s"(rdata), "s"(soff) : "memory")
#define PIN(x) asm volatile("" : "+s"(x))

__device__ __forceinline__ int ready_from(unsigned long long okmask, int base) {
  const int r = base + (int)((unsigned)__builtin_ctzll(~okmask | (1ull << 63)) * 57u >> 9);
  return r < 32 ? r : 32;
}

__device__ __forceinline__ void gemm_role(const Params& p, LAS unsigned char* lds, int g, int wv, int lane) {
  const int lin = (g & 7) * 30 + (g >> 3), band = lin / 80, cb = (lin % 80) / 5;
  const int tcol = (band & 1) ? 15 - cb : cb, trow = band * 5 + lin % 5;
  uint8_t* const ctl = p.ctl;
  if (g < TOUCH_WGS && wv < 7 && ((TOUCH_MASK >> lane) & 1) && lane < 9) {
    const unsigned o = lane == 0 ? CTL_UFLAG : (unsigned)DATA_OFF + (unsigned)((lane - 1) >> 2) * SLOT_B + ((unsigned)p.rank * 4u + ((lane - 1) & 3)) * ENT_B;
    const unsigned v = __hip_atomic_load((const unsigned*)(p.peer[wv] + o), __ATOMIC_RELAXED, __HIP_MEMORY_SCOPE_SYSTEM);
    asm volatile("" :: "v"(v));
  }
  __builtin_amdgcn_s_sleep(GDELAY);
#pragma unroll
  for (int i = 0; i < 3; ++i) {
    const int gid = g + i * NGEMM, type = (gid >> 8) & 1, kt = (gid >> 6) & 3;
    if (gid < 512) {
      const __amdgpu_buffer_rsrc_t rl = __builtin_amdgcn_make_buffer_rsrc((void*)(type ? p.w : p.x), 0, MN * KS, 0x00020000);
      const unsigned lo = tile_src_off(type, kt, gid & 63, wv, lane), lm = (unsigned)(uintptr_t)lds + i * 8192 + wv * 1024;
      asm volatile(TLOAD :: TLOAD_IN : "memory");
    }
  }
  const unsigned call = __builtin_amdgcn_readfirstlane(
      __hip_atomic_load((const unsigned*)(ctl + CTL_DONE), __ATOMIC_RELAXED, __HIP_MEMORY_SCOPE_AGENT)) / NWG + 1u;
  const __amdgpu_buffer_rsrc_t rdata =
      __builtin_amdgcn_make_buffer_rsrc(ctl + DATA_OFF + (size_t)(call & 1u) * SLOT_B, 0, SLOT_B, 0x00020000);
  WAIT_V(0);
  BAR;
#pragma unroll
  for (int i = 0; i < 3; ++i) {
    const int gid = g + i * NGEMM, type = (gid >> 8) & 1, kt = (gid >> 6) & 3;
    if (gid < 512)
      __builtin_amdgcn_raw_buffer_store_b128(unit_read(type, lds + i * 8192, wv, lane), rdata,
                                             (unsigned)p.rank * 4u * ENT_B + unit_dst_off(type, kt, gid & 63, wv) + lane * 16, 0, 17);
  }
  LAS unsigned* ord = (LAS unsigned*)(lds + LDS_ORD);
  LAS unsigned* tbl = (LAS unsigned*)(lds + LDS_TBL);
  LAS unsigned* bc = (LAS unsigned*)(lds + LDS_CTL);
  const int hj = (trow / 5 + 2 - tcol % 3) % 3;
  const bool host = trow % 5 == 0 && hj < 2;
  auto group_of = [&](int i) -> int { return i < 4 ? 4 * trow + i : i < 8 ? 4 * tcol + i - 4 : host ? 60 + (p.rank >> 1) : 4 * trow; };
  auto pub_pos = [&](int s, int kt, int i) -> int {
    const int type = (i >> 2) & 1, gg = group_of(i);
    const int pos = p.inv[s][kt * 32 + type * 16 + (gg >> 2)] * 4 + (gg & 3), r = pos >> 4;
    return r >= p.fine_start ? pos : (r + __builtin_ctz(p.pubmask >> r)) * 16 + 15;
  };
  if (wv == 0) {
    const int e = lane & 31, s = e >> 2, kt = e & 3;
    int key = kt;
    if (s != p.rank) {
      int m = 0;
      for (int i = 0; i < 9; ++i) { const int r = pub_pos(s, kt, i); m = r > m ? r : m; }
      key = 4 + m * 32 + kt * 8 + ((s - p.rank + g) & 7);
    }
    int rk = 0;
    for (int l = 0; l < 32; ++l) rk += __builtin_amdgcn_readlane(key, l) < key;
    if (lane < 32) ord[rk] = e;
  }
  WAIT_V(0);
  __syncthreads();
  if (threadIdx.x == 64) COUNT_IN(ctl);
  if (threadIdx.x == 0)
    __hip_atomic_store((unsigned*)(ctl + CTL_LFLAG) + g, call * 16u, __ATOMIC_RELAXED, __HIP_MEMORY_SCOPE_SYSTEM);
  if (threadIdx.x < 352) {
    int t = threadIdx.x / 9; const int i = threadIdx.x - t * 9; t = t < 32 ? t : 31;
    const int e = ord[t], s = e >> 2, kt = e & 3, type = (i >> 2) & 1, gg = group_of(i);
    unsigned off;
    if (s == p.rank) off = CTL_LFLAG + 4u * (unsigned)((type * 256 + kt * 64 + gg) % NGEMM);
    else { const int pp = pub_pos(s, kt, i);
           off = (pp >> 4) >= p.fine_start ? rflag_off(s, pp) : uflag_off(s, pp >> 4); }
    tbl[threadIdx.x] = off;
  }
  const int ordv = ord[lane & 31];
  ((LAS unsigned long long*)(lds + LDS_SNAP))[threadIdx.x] = 0ull;
  ((LAS unsigned*)(lds + LDS_SNAP + 4096))[threadIdx.x] = 0u;
  if (threadIdx.x == 0) bc[63] = 0u;
  if (threadIdx.x < 7) ((LAS unsigned long long*)(bc + 32))[threadIdx.x] = (unsigned long long)(p.peer[threadIdx.x] + CTL_SLICE);
  if (wv == 7 && lane < 5 + (host ? 8 : 0)) {
    const float* q = lane < 5 ? p.out + (size_t)(trow * 256 + (lane == 4 ? 255 : lane * 64)) * MN + tcol * 256
                              : p.outs[lane - 5] + (size_t)(3840 + 32 * p.rank + 16 * hj) * MN + tcol * 256;
    const unsigned v = __hip_atomic_load((const unsigned*)q, __ATOMIC_RELAXED, __HIP_MEMORY_SCOPE_SYSTEM);
    asm volatile("" :: "v"(v));
  }
  int scale_bits = __builtin_amdgcn_readfirstlane(__builtin_bit_cast(int, p.sx[0] * p.sw[0]));
  asm volatile("" : "+s"(scale_bits));
  WAIT_V(0);
  __syncthreads();

  const int wr = wv >> 2, wc = wv & 3;
  const unsigned voff = lane * 16;
  const unsigned ldsb = (unsigned)(uintptr_t)lds;
  const unsigned a0o = (9 * wr + wc) * 2048, a8o = (9 * wr + 8) * 2048 + wc * 512, b0o = (4 * (wv >> 1) + (wv & 1)) * 2048;
  const bool has9 = host && wr == 0;
  const unsigned ga0 = (unsigned)(16 * trow + 8 * wr + wc) * 2048u, gb0 = OP_B + tcol * 32768u + b0o;
  const unsigned ga8 = has9 ? (unsigned)(240 + 2 * p.rank + hj) * 2048u + wc * 512u : OOB;
  const unsigned la0 = ldsb + LDS_A + a0o, la8 = ldsb + LDS_A + a8o, lb0 = ldsb + LDS_B + b0o;
#define ENT(t) ((unsigned)__builtin_amdgcn_readlane(ordv, (t)) * ENT_B)
#define ST_A03(eb, sl) DMA2(la0 + (sl) * A_SLOT, (eb) + ga0)
#define ST_A48(eb, sl) DMA3(la0 + 8192 + (sl) * A_SLOT, (eb) + ga0 + 8192, la8 + (sl) * A_SLOT, (eb) + ga8)
#define ST_B0(eb, sl) DMA2(lb0 + (sl) * B_SLOT, (eb) + gb0)
#define ST_B1(eb, sl) DMA2(lb0 + 4096 + (sl) * B_SLOT, (eb) + gb0 + 4096)
#define ST_ALL(eb, sl) do { ST_B0(eb, sl); ST_B1(eb, sl); ST_A03(eb, sl); ST_A48(eb, sl); } while (0)
#define LDF(dst, base, off) do { const i32x4 _l = *(const LAS i32x4*)((base) + (off)), _h = *(const LAS i32x4*)((base) + (off) + 1024); \
    dst = __builtin_shufflevector(_l, _h, 0, 1, 2, 3, 4, 5, 6, 7); SCHED; } while (0)
#define MM(m, n, Af) do { acc[m][n] = __builtin_amdgcn_mfma_scale_f32_16x16x128_f8f6f4(Af, Bf[n], acc[m][n], 1, 1, 0, 0, 0, 0); SCHED; } while (0)

  f32x4 acc[9][4];
#pragma unroll
  for (int m = 0; m < 9; ++m)
#pragma unroll
    for (int n = 0; n < 4; ++n) acc[m][n] = (f32x4){0.f, 0.f, 0.f, 0.f};
  i32x8 a0, a1, a2, a3, a4, Bf[4];
  int t = 0, ready = 0, nrestart = 0;
  unsigned dead = 0;
  const unsigned thr = call * 16u;
  const unsigned pollo = tbl[threadIdx.x < 352 ? threadIdx.x : 351];
  const __amdgpu_buffer_rsrc_t rctl = __builtin_amdgcn_make_buffer_rsrc(ctl, 0, CTL_BYTES, 0x00020000);
  const __amdgpu_buffer_rsrc_t rout = __builtin_amdgcn_make_buffer_rsrc(p.out, 0, MN * MN * 4, 0x00020000);
  const LAS unsigned char* const pa0 = lds + LDS_A + wr * (9 * 2048) + lane * 16;
  const LAS unsigned char* const pb0 = lds + LDS_B + wc * (4 * 2048) + lane * 16;
#define POLL(buf) do { const unsigned _po = (unsigned)(ready * 9 + 62 - wv * 64) < 126u && ready < 32 ? 0u : OOB; \
    asm volatile("s_mov_b32 m0, %0\n\ts_nop 0\n\tbuffer_load_dword %1, %2, %3 offen " DMA_POL " lds" \
                 :: "s"(ldsb + LDS_SNAP + (buf) * 2048 + wv * 256), "v"(pollo), "s"(rctl), "s"(_po) : "memory"); } while (0)
#define SNAP_AT(buf) ((const LAS unsigned*)(lds + LDS_SNAP + ((buf) * 512 + ready * 9) * 4) + lane)
#define OKMASK(v) __builtin_amdgcn_ballot_w64((int)((v) - thr) >= 0)
#define STAGE_FROM(u) do { const unsigned _e0 = ENT(u); ST_ALL(_e0, (u) & 1); sn = ready > (u) + 1; \
    if (sn) { BAR;   const unsigned _e1 = ENT((u) + 1); ST_ALL(_e1, ((u) & 1) ^ 1); } \
    POLL(((u) & 1) ^ 1); } while (0)
  int sn = -1;
  for (;;) {
    if (sn < 0) {
      if (ready <= t) {
        if (wv == 0) {
          const unsigned* fp = (const unsigned*)(ctl + (lane == 63 ? CTL_TMO : tbl[t * 9 + lane]));
          unsigned* tmo = (unsigned*)(ctl + CTL_TMO);
          const unsigned long long t0 = __builtin_amdgcn_s_memrealtime();
          int r; unsigned dd = 0;
          for (;;) {
            const unsigned v = __hip_atomic_load(fp, __ATOMIC_RELAXED, __HIP_MEMORY_SCOPE_SYSTEM);
            r = ready_from(OKMASK(v), t);
            if (r > t) break;
            if ((unsigned)__builtin_amdgcn_readlane(v, 63) == call || __builtin_amdgcn_s_memrealtime() - t0 > DEADLINE_TICKS) {
              if (lane == 0) __hip_atomic_store(tmo, call, __ATOMIC_RELAXED, __HIP_MEMORY_SCOPE_AGENT);
              r = 32; dd = 64; break;
            }
            __builtin_amdgcn_s_sleep(2);
          }
          if (lane == 0) bc[nrestart & 31] = (unsigned)r | (dd << 8);
          asm volatile("s_waitcnt vmcnt(0) lgkmcnt(0)" ::: "memory");
        }
        BAR;
        const unsigned v = __builtin_amdgcn_readfirstlane(bc[nrestart & 31]);
        ready = v & 63; dead |= v >> 8; ++nrestart;
      }
      STAGE_FROM(t);
    }
    int s1 = sn;
    sn = -1;
    if (s1) WAIT_V(13); else WAIT_V(4);
    BAR;
    { const LAS unsigned char* pa = pa0 + (t & 1) * A_SLOT; const LAS unsigned char* pb = pb0 + (t & 1) * B_SLOT;
      LDF(Bf[0], pb, 0); LDF(a0, pa, 0); LDF(a1, pa, 2048); LDF(Bf[1], pb, 2048); LDF(Bf[2], pb, 4096); LDF(a2, pa, 4096); LDF(Bf[3], pb, 6144); }
    if (t == 31) break;
    unsigned en2 = ENT((t + 2) & 31);
    int first = 1;
    for (;;) {
      const int sl = t & 1;
      const LAS unsigned char* pa = pa0 + sl * A_SLOT;
      LDF(a3, pa, 3 * 2048);
      MM(0, 0, a0);
      const int rich = (ready > t + 2 || ready == 32 || (first & s1)) ? 1 : 0;
      const LAS unsigned* psn = SNAP_AT(sl ^ rich ^ 1);
      asm volatile("" : "+v"(psn)); SCHED;
      MM(1, 0, a1);
      const LAS unsigned char* pan = pa0 + (sl ^ 1) * A_SLOT; const LAS unsigned char* pbn = pb0 + (sl ^ 1) * B_SLOT;
      asm volatile("" : "+v"(pan), "+v"(pbn)); SCHED;
      MM(0, 1, a0);
      unsigned mB = lb0 + sl * B_SLOT, mA = la0 + sl * A_SLOT;
      PIN(mB); PIN(mA); SCHED;
      MM(1, 1, a1);
      unsigned mA8 = la8 + sl * A_SLOT, mP = ldsb + LDS_SNAP + sl * 2048 + wv * 256;
      PIN(mA8); PIN(mP); SCHED;
      MM(0, 2, a0); MM(1, 2, a1);
      WAIT_V(10); if (!rich) { if (first) WAIT_V(0); else WAIT_V(3); }
      first = 0;
      WAIT_L(0); BAR; SCHED;
      const unsigned snap = *psn;
      SCHED;
      MM(0, 3, a0);
      LDF(a4, pa, 6 * 2048);
      MM(1, 3, a1);
      LDF(a0, pa, 4 * 2048);
      MM(2, 0, a2);
      unsigned long long okm = OKMASK(snap);
      SCHED;
      MM(2, 1, a2);
      LDF(a1, pa, 5 * 2048);
      unsigned fz = (unsigned)__builtin_ctzll(~okm | (1ull << 63)) * 57u;
      PIN(fz); SCHED;
      MM(2, 2, a2);
      ready += (int)(fz >> 9); ready = ready < 32 ? ready : 32;
      if (__builtin_expect(!s1, 0)) {
        if (ready > t + 1) { const unsigned e1 = ENT(t + 1); ST_ALL(e1, sl ^ 1); s1 = 1; }
        POLL(2);
      }
      SCHED;
      MM(2, 3, a2);
      int s2 = ready > t + 2 ? s1 : 0;
      unsigned e2 = s2 ? en2 : OOB;
      PIN(s2); PIN(e2);
      LDF(a2, pa, 8 * 2048);
      MM(3, 0, a3);
      DMA1(mB, e2 + gb0); SCHED;
      MM(3, 1, a3);
      DMA1B(mB, e2 + gb0); SCHED;
      MM(3, 2, a3);
      DMA1(mB + 4096, e2 + gb0 + 4096); SCHED;
      MM(3, 3, a3);
      LDF(a3, pa, 7 * 2048);
      DMA1B(mB + 4096, e2 + gb0 + 4096); SCHED;
      MM(4, 0, a0);
      DMA1(mA, e2 + ga0); SCHED;
      MM(4, 1, a0);
      DMA1B(mA, e2 + ga0); SCHED;
      MM(4, 2, a0);
      en2 = ENT((t + 3) & 31);
      SCHED;
      MM(4, 3, a0);
      unsigned po = (unsigned)(ready * 9 + 62 - wv * 64) < 126u && ready < 32 ? 0u : OOB;
      PIN(po); SCHED;
      MM(5, 0, a1); MM(5, 1, a1);
      WAIT_V(10); if (__builtin_expect(!s1, 0)) WAIT_V(0);
      WAIT_L(0); BAR; SCHED;
      MM(5, 2, a1);
      LDF(a0, pan, 0);
      asm volatile("s_mov_b32 m0, %0\n\ts_nop 0\n\tbuffer_load_dword %1, %2, %3 offen " DMA_POL " lds"
                   :: "s"(mP), "v"(pollo), "s"(rctl), "s"(po) : "memory");
      SCHED;
      MM(5, 3, a1);
      LDF(a1, pan, 2048);
      DMA1(mA + 8192, e2 + ga0 + 8192); SCHED;
      if (has9) { MM(8, 0, a2); MM(8, 1, a2); MM(8, 2, a2); MM(8, 3, a2); }
      LDF(a2, pan, 4096);
      DMA1B(mA + 8192, e2 + ga0 + 8192); SCHED;
      MM(7, 0, a3);
      DMA1H(mA8, e2 + ga8);
      if (__builtin_expect(!s1, 0)) {
        ready = ready_from(OKMASK(*SNAP_AT(2)), ready);
        if (ready > t + 1) STAGE_FROM(t + 1);
      }
      SCHED;
      MM(6, 0, a4);
      LDF(Bf[0], pbn, 0);
      MM(7, 1, a3); MM(6, 1, a4);
      LDF(Bf[1], pbn, 2048);
      MM(7, 2, a3); MM(6, 2, a4);
      LDF(Bf[2], pbn, 4096);
      MM(7, 3, a3); MM(6, 3, a4);
      LDF(Bf[3], pbn, 6144);
      ++t;
      if (!s1 || t == 31) break;
      s1 = s2;
    }
    if (s1) break;
    if (sn < 0) {
      asm volatile("s_waitcnt vmcnt(0) lgkmcnt(0)" ::: "memory"); BAR;
      ready = ready_from(OKMASK(*SNAP_AT((t & 1) ^ 1)), ready);
      WAIT_L(0); BAR;
    }
  }
  {
    const float sc = __builtin_bit_cast(float, dead ? 0x7fc00000 : scale_bits);
    const unsigned ov = (unsigned)((lane >> 4) * 4 * MN + (lane & 15) * 4) * 4u;
    const unsigned os = (unsigned)((trow * 256 + wr * 128) * MN + tcol * 256 + wc * 64) * 4u;
#define OUT(m) do { f32x4 o[4]; \
      asm volatile("" : "+v"(acc[m][0]), "+v"(acc[m][1]), "+v"(acc[m][2]), "+v"(acc[m][3])); \
      _Pragma("unroll") for (int j = 0; j < 4; ++j) { o[j][0] = acc[m][0][j] * sc; o[j][1] = acc[m][1][j] * sc; o[j][2] = acc[m][2][j] * sc; o[j][3] = acc[m][3][j] * sc; } \
      SCHED; \
      _Pragma("unroll") for (int j = 0; j < 4; ++j) \
        __builtin_amdgcn_raw_buffer_store_b128(__builtin_bit_cast(u32x4, o[j]), rout, ov, os + (unsigned)((m) * 16 + j) * (MN * 4u), OUT_AUX); \
      asm volatile("s_nop 1" ::: "memory");                     \
      SCHED; } while (0)
    const LAS unsigned char* pa = pa0 + A_SLOT;
    LDF(a3, pa, 3 * 2048);
    MM(0, 0, a0); MM(1, 0, a1); MM(0, 1, a0); MM(1, 1, a1); MM(0, 2, a0); MM(1, 2, a1);
    WAIT_V(0); BAR; SCHED;
    MM(0, 3, a0);
    LDF(a4, pa, 8 * 2048);
    MM(1, 3, a1);
    LDF(a0, pa, 4 * 2048);
    MM(2, 0, a2); MM(2, 1, a2);
    LDF(a1, pa, 5 * 2048);
    MM(2, 2, a2); MM(2, 3, a2);
    LDF(a2, pa, 6 * 2048);
    f32x4 o8[4];
    const unsigned os8 = (unsigned)((3840 + 32 * p.rank + 16 * hj) * MN + tcol * 256 + wc * 64) * 4u;
#define SHARE(d) do { if (has9) { \
      const __amdgpu_buffer_rsrc_t rd = __builtin_amdgcn_make_buffer_rsrc(p.outs[(p.rank + 1 + ((d) < 7 ? ((d) + tcol + wc) % 7 : 7)) & (NR - 1)], 0, MN * MN * 4, 0x00020000); \
      asm volatile("s_waitcnt vmcnt(" STR(SHARE_WIN) ")" ::: "memory"); \
      _Pragma("unroll") for (int j = 0; j < 4; ++j) \
        __builtin_amdgcn_raw_buffer_store_b128(__builtin_bit_cast(u32x4, o8[j]), rd, ov, os8 + (unsigned)j * (MN * 4u), 17); } \
      SCHED; } while (0)
    if (has9) { MM(8, 0, a4); MM(8, 1, a4); MM(8, 2, a4); MM(8, 3, a4); }
    OUT(0);
#pragma unroll
    for (int j = 0; j < 4; ++j) { o8[j][0] = acc[8][0][j] * sc; o8[j][1] = acc[8][1][j] * sc; o8[j][2] = acc[8][2][j] * sc; o8[j][3] = acc[8][3][j] * sc; }
    SCHED;
    SHARE(0);
    MM(3, 0, a3); MM(3, 1, a3); MM(3, 2, a3); MM(3, 3, a3);
    LDF(a3, pa, 7 * 2048);
    OUT(1); SHARE(1);
    MM(4, 0, a0); MM(4, 1, a0); MM(4, 2, a0); MM(4, 3, a0);
    OUT(2); SHARE(2);
    MM(5, 0, a1); MM(5, 1, a1); MM(5, 2, a1); MM(5, 3, a1);
    OUT(3); SHARE(3);
    MM(6, 0, a2); MM(6, 1, a2); MM(6, 2, a2); MM(6, 3, a2);
    OUT(4); SHARE(4);
    MM(7, 0, a3); MM(7, 1, a3); MM(7, 2, a3); MM(7, 3, a3);
    OUT(5); SHARE(5); OUT(6); SHARE(6); OUT(7); SHARE(7);
    if (has9) {
      WAIT_V(8);
      unsigned old = 0;
      if (lane == 0) old = __hip_atomic_fetch_add(bc + 63, 1u, __ATOMIC_RELAXED, __HIP_MEMORY_SCOPE_WORKGROUP);
      if (__builtin_amdgcn_readfirstlane(old) == 3u && lane < 7)
        __hip_atomic_fetch_add((GAS unsigned*)((LAS unsigned long long*)(bc + 32))[lane], 1u, __ATOMIC_RELAXED, __HIP_MEMORY_SCOPE_SYSTEM);
    }
  }
}

__global__ void __launch_bounds__(512, 2) gemm_ag_kernel(const Params p) {
  __shared__ __attribute__((aligned(16))) unsigned char lds_raw[LDS_BYTES];
  LAS unsigned char* lds = (LAS unsigned char*)lds_raw;
  const int wv = __builtin_amdgcn_readfirstlane(threadIdx.x >> 6), lane = threadIdx.x & 63;
  if (blockIdx.x < NCOMM) comm_role(p, lds, blockIdx.x, wv, lane);
  else gemm_role(p, lds, blockIdx.x - NCOMM, wv, lane);
}

static void make_schedule(int s, uint8_t* sc) {
  static const int nlate[5] = {0, 3, 5, 8, 8};
  static const double fin[5] = {0.0, 3.0, 5.2, 7.6, 7.6};
  int r = 0;
  for (int kt = 0; kt < 3; ++kt)
    for (int q = 0; q < 16; ++q) {
      sc[r++] = (uint8_t)(kt * 32 + 16 + (q + 2 * s) % 16);
      sc[r++] = (uint8_t)(kt * 32 + (q + 2 * s) % 16);
    }
  double due[16]; int late[16], nl = 0;
  sc[r++] = (uint8_t)(96 + 15);
  for (int c = 0; c < 15; ++c) {
    const int lv = c % 5, i = ((s - (lv * 3 + c / 5) * 3) % 8 + 8) % 8;
    if (i < nlate[lv]) { due[nl] = fin[lv] - (nlate[lv] - i) * 1.15 + 0.01 * (c / 5); late[nl++] = c; }
    else sc[r++] = (uint8_t)(96 + c);
  }
  for (int q = 0; q < 16; ++q) sc[r++] = (uint8_t)(96 + 16 + (q + 2 * s) % 16);
  for (int k = 0; k < nl; ++k) {
    int best = -1;
    for (int j = 0; j < nl; ++j) if (late[j] >= 0 && (best < 0 || due[j] < due[best])) best = j;
    sc[r++] = (uint8_t)(96 + late[best]); late[best] = -1;
  }
}

}

extern "C" void kernel_launch(int rank, int nranks, ncclComm_t comm,
                              hipStream_t stream,
                              void* const* d_in, const int* in_sizes, int n_in,
                              void* d_out, int out_size,
                              const int* in_dtypes, int out_dtype,
                              void* const* d_out_all,
                              void* const* d_ws_all, size_t ws_bytes) {
  (void)comm; (void)in_dtypes; (void)out_dtype;
  if (nranks != NR || n_in != 4 || (unsigned)rank >= (unsigned)NR || ws_bytes < WS_NEED || out_size != MN * MN ||
      in_sizes[0] != MN * KS || in_sizes[1] != MN * KS) {
    fprintf(stderr, "kernel_launch: unexpected problem shape\n");
    return;
  }
  Params p;
  p.x = (const uint8_t*)d_in[0]; p.w = (const uint8_t*)d_in[1];
  p.sx = (const float*)d_in[2]; p.sw = (const float*)d_in[3];
  p.out = (float*)d_out;
  for (int i = 0; i < NR; ++i) p.outs[i] = (float*)d_out_all[i];
  p.ctl = (uint8_t*)d_ws_all[rank];
  for (int i = 0; i < 14; ++i) p.peer[i] = (uint8_t*)d_ws_all[(rank + 1 + i % 7) % NR];
  p.rank = rank;
  p.pubmask = 0x00AA8080u;
  p.fine_start = 24;
  for (int s = 0; s < NR; ++s) {
    uint8_t sc[128];
    make_schedule(s, sc);
    for (int r = 0; r < 128; ++r) p.inv[s][sc[r]] = (uint8_t)r;
    if (s == rank)
      for (int i = 0; i < 32; ++i)
        p.sched[i] = sc[4 * i] | (sc[4 * i + 1] << 8) | (sc[4 * i + 2] << 16) | ((unsigned)sc[4 * i + 3] << 24);
  }
  p.sched01[0] = p.sched[0]; p.sched01[1] = p.sched[1];
  hipLaunchKernelGGL(gemm_ag_kernel, dim3(NWG), dim3(512), 0, stream, p);
  HIP_CHECK(hipGetLastError());
}
```

```cpp
#include <hip/hip_runtime.h>
#include <rccl/rccl.h>
#include <cstdio>
#include <cstdint>

#define HIP_CHECK(expr) do { hipError_t _e = (expr); if (_e != hipSuccess) { \
    fprintf(stderr, "HIP error %s at %s:%d\n", hipGetErrorString(_e), __FILE__, __LINE__); \
    return; } } while (0)

#define DMA_POL "sc0 sc1"
#define WINM1 4
#define SHARE_WIN 6
#define OUT_AUX 16

namespace {

#define LAS __attribute__((address_space(3)))
#define GAS __attribute__((address_space(1)))
typedef unsigned u32x4 __attribute__((ext_vector_type(4)));
typedef int i32x4 __attribute__((ext_vector_type(4)));
typedef int i32x8 __attribute__((ext_vector_type(8)));
typedef float f32x4 __attribute__((ext_vector_type(4)));

constexpr int NR = 8;
constexpr int MN = 4096;
constexpr int KS = 512;
constexpr int NCOMM = 16;
constexpr int NGEMM = 240;
constexpr int NWG = NCOMM + NGEMM;
constexpr unsigned ENT_B = 1u << 20;
constexpr unsigned OP_B = 1u << 19;
constexpr unsigned SLOT_B = 32u << 20;
constexpr size_t DATA_OFF = 1u << 20;
constexpr size_t WS_NEED = DATA_OFF + 2ull * SLOT_B;
constexpr unsigned CTL_DONE = 0x0000, CTL_TMO = 0x0080, CTL_TICK = 0x0200, CTL_SLICE = 0x0400, CTL_LFLAG = 0x1000,
                   CTL_UFLAG = 0x8000, CTL_RFLAG = 0x10000, CTL_BYTES = 0x30000;
__host__ __device__ constexpr unsigned uflag_off(int s, int unit) { return CTL_UFLAG + (unsigned)(s * 32 + unit) * 128u; }
__host__ __device__ constexpr unsigned rflag_off(int s, int pos) { return CTL_RFLAG + (unsigned)(s * 128 + (pos >> 2)) * 128u + (unsigned)(pos & 3) * 4u; }
constexpr unsigned OOB = 0x40000000u;
constexpr unsigned long long DEADLINE_TICKS = 100000000ull;

constexpr int A_SLOT = 18 * 2048, B_SLOT = 16 * 2048;
constexpr int LDS_A = 0, LDS_B = 2 * A_SLOT;
constexpr int LDS_TBL = LDS_B + 2 * B_SLOT;
constexpr int LDS_SNAP = LDS_TBL + 352 * 4;
constexpr int LDS_ORD = LDS_SNAP + 6144;
constexpr int LDS_CTL = LDS_ORD + 128;
constexpr int LDS_BYTES = LDS_CTL + 256;

struct Params {
  const uint8_t* x; const uint8_t* w; uint8_t* ctl;
  unsigned sched01[2];
  uint8_t* peer[14];
  const float* sx; const float* sw; float* out;
  float* outs[NR];
  int rank;
  unsigned pubmask;
  int fine_start;
  unsigned sched[32];
  uint8_t inv[NR][128];
};

#define COUNT_IN(ctl) __hip_atomic_fetch_add((unsigned*)((ctl) + CTL_DONE), 1u, __ATOMIC_RELAXED, __HIP_MEMORY_SCOPE_AGENT)
#define WAIT_V(n) asm volatile("s_waitcnt vmcnt(" #n ")" ::: "memory")
#define WAIT_L(n) asm volatile("s_waitcnt lgkmcnt(" #n ")" ::: "memory")
#define BAR do { asm volatile("" ::: "memory"); __builtin_amdgcn_s_barrier(); asm volatile("" ::: "memory"); } while (0)
#define SCHED __builtin_amdgcn_sched_barrier(0)

__device__ __forceinline__ unsigned tile_src_off(int type, int kt, int g, int wv, int lane) {
  const int row = wv * 8 + (lane >> 3), ch = (lane & 7) * 16;
  return type ? (unsigned)((kt * 128 + (g & 1) * 64 + row) * MN + (g >> 1) * 128 + ch)
              : (unsigned)((g * 64 + row) * KS + kt * 128 + ch);
}
__device__ __forceinline__ unsigned unit_dst_off(int type, int kt, int g, int wv) {
  const int u = type ? ((g >> 1) * 8 + wv) * 2 + (g & 1) : g * 8 + wv;
  return (unsigned)kt * ENT_B + (unsigned)type * OP_B + (unsigned)u * 1024u;
}
__device__ __forceinline__ u32x4 unit_read(int type, const LAS unsigned char* tile, int wv, int lane) {
  if (type) {
    const LAS unsigned* a = (const LAS unsigned*)(tile + (lane >> 4) * 2048 + (wv >> 2) * 64 + (lane & 15) * 4);
    const unsigned n = wv & 3, s1 = n | ((n + 4u) << 8);
    u32x4 r;
#pragma unroll
    for (int i = 0; i < 4; ++i) {
      const unsigned lo = __builtin_amdgcn_perm(a[(4 * i + 1) * 32], a[(4 * i) * 32], s1);
      const unsigned hi = __builtin_amdgcn_perm(a[(4 * i + 3) * 32], a[(4 * i + 2) * 32], s1);
      r[i] = __builtin_amdgcn_perm(hi, lo, 0x05040100u);
    }
    return r;
  }
  return *(const LAS u32x4*)(tile + ((wv >> 1) * 16 + (lane & 15)) * 128 + (wv & 1) * 64 + (lane >> 4) * 16);
}

#define TLOAD "s_mov_b32 m0, %[lm]\n\ts_nop 3\n\tbuffer_load_dwordx4 %[lo], %[rl], 0 offen lds\n\t"
#define TLOAD_IN [lm] "s"(lm), [lo] "v"(lo), [rl] "s"(rl)
#define STR2(x) #x
#define STR(x) STR2(x)
#define PUSH1(j) asm volatile("s_waitcnt vmcnt(" STR(WINM1) ")\n\tbuffer_store_dwordx4 %0, %1, %2, %3 offen sc0 sc1" \
                              :: "v"(cur), "v"(so), "s"(rs[j]), "s"((skip >> j) & 1u ? OOB : 0u) : "memory")
#define DRAW(vnew) asm volatile("s_mov_b64 exec, 1\n\tglobal_atomic_add " vnew ", %0, %1, %2 sc0\n\ts_mov_b64 exec, -1" \
                                :: "v"(ao), "v"(one), "s"(ctl) : "memory", "v254", "v255")
#define GOT(vold) asm volatile("v_mov_b32 %0, " vold : "=v"(aret) :: "memory", "v254", "v255")

__device__ __forceinline__ void comm_role(const Params& p, LAS unsigned char* lds, int wg, int wv, int lane) {
  LAS unsigned* tick = (LAS unsigned*)(lds + LDS_CTL);
  uint8_t* const ctl = p.ctl;
  const unsigned ldsw = (unsigned)(uintptr_t)lds + wv * 1024;
  auto code_of = [&](int pos) -> int { return (int)((p.sched[(pos >> 4) & 31] >> (((pos >> 2) & 3) * 8)) & 127u); };
#define TILE_ARGS(_c, pos, slot) \
    const __amdgpu_buffer_rsrc_t rl = __builtin_amdgcn_make_buffer_rsrc((void*)(((_c) & 16) ? p.w : p.x), 0, MN * KS, 0x00020000); \
    const unsigned lo = tile_src_off(((_c) >> 4) & 1, (_c) >> 5, ((_c) & 15) * 4 + ((pos) & 3), wv, lane), lm = ldsw + ((slot) & 3) * 8192
  int pm1 = 0, p0 = wg, p1 = 16 + wg, p2 = 32 + wg;
  int c0 = (int)((p.sched01[0] >> ((wg >> 2) * 8)) & 127u), c1 = (int)((p.sched01[1] >> ((wg >> 2) * 8)) & 127u);
  { TILE_ARGS(c0, p0, 0); asm volatile(TLOAD :: TLOAD_IN : "memory"); }
  unsigned done0;
  asm volatile("global_load_dword %0, %1, %2 sc1" : "=v"(done0) : "v"(CTL_DONE), "s"(ctl) : "memory");
  { TILE_ARGS(c1, p1, 1); asm volatile(TLOAD :: TLOAD_IN : "memory"); }
  __amdgpu_buffer_rsrc_t rs[7];
  const int rot = (wg + wv) % 7;
#pragma unroll
  for (int j = 0; j < 7; ++j)
    rs[j] = __builtin_amdgcn_make_buffer_rsrc(p.peer[rot + j] + DATA_OFF + (size_t)p.rank * 4 * ENT_B, 0, SLOT_B + 4 * ENT_B, 0x00020000);
  if (wv < 7 && lane == 0) ((LAS unsigned long long*)(tick + 8))[wv] = (unsigned long long)p.peer[wv];
  unsigned open = p.pubmask;
  const int fine_start = p.fine_start, rank = p.rank;
  asm volatile("s_waitcnt vmcnt(0) lgkmcnt(0)" : "+v"(done0) :: "memory");
  BAR;
  const unsigned call = __builtin_amdgcn_readfirstlane(done0) / NWG + 1u;
  const unsigned ao = CTL_TICK + (call & 1u) * 128u, one = 1u;
  const unsigned sbase = (call & 1u) * SLOT_B + lane * 16;
  for (int r = 0;; ++r) {
    u32x4 cur; unsigned so = 0, skip = 0;
    if (p0 < 512) {
      const int c = c0;
      cur = unit_read((c >> 4) & 1, lds + (r & 3) * 8192, wv, lane);
      so = unit_dst_off((c >> 4) & 1, c >> 5, (c & 15) * 4 + (p0 & 3), wv) + sbase;
      if ((c & 31) == 15) {
        const int d = ((p0 & 3) * 2 + (wv >> 2) - rank - 1) & (NR - 1);
        skip = d == 7 ? 0x7fu : 0x7fu & ~(1u << ((d + 7 - rot) % 7));
      }
#pragma unroll
      for (int j = 0; j < WINM1; ++j) PUSH1(j);
      asm volatile("s_waitcnt vmcnt(" STR(WINM1) ")" ::: "memory");
      if (skip) WAIT_V(0);
      if (wv == 0) {
        unsigned aret;
        if (r & 1) GOT("v254"); else GOT("v255");
        if (lane == 0) tick[r & 1] = aret;
        WAIT_L(0);
      }
    } else {
      WAIT_V(0);
    }
    BAR;
    if (r == 0) {
      if (threadIdx.x == 128) COUNT_IN(ctl);
      if (wg == 0 && threadIdx.x == 64)
        __hip_atomic_store((unsigned*)(ctl + CTL_TICK + ((call & 1u) ^ 1u) * 128u), 0u, __ATOMIC_RELAXED, __HIP_MEMORY_SCOPE_AGENT);
    } else {
      GAS uint8_t* const pctl = (GAS uint8_t*)((LAS unsigned long long*)(tick + 8))[lane < 7 ? lane : 0];
      if ((pm1 >> 4) >= fine_start) {
        if (wv == 1 + (r & 3) && lane < 7)
          __hip_atomic_store((GAS unsigned*)(pctl + rflag_off(rank, pm1)), call * 16u, __ATOMIC_RELAXED, __HIP_MEMORY_SCOPE_SYSTEM);
      }
      unsigned done = (p0 >> 4) >= 32 ? open : open & ((1u << (p0 >> 4)) - 1u);
      open &= ~done;
      for (; done; done &= done - 1u)
        if (wv == 1 + (r & 3) && lane < 7)
          __hip_atomic_fetch_add((GAS unsigned*)(pctl + uflag_off(rank, __builtin_ctz(done))), 1u, __ATOMIC_RELAXED, __HIP_MEMORY_SCOPE_SYSTEM);
    }
    if (p0 >= 512) break;
    if (r > 0) {
      const unsigned tk = __builtin_amdgcn_readfirstlane(tick[r & 1]);
      p2 = p1 < 512 && tk < 464u ? 48 + (int)tk : 512;
    }
#pragma unroll
    for (int j = WINM1; j < 7; ++j) PUSH1(j);
    const int pn = p2 < 512 ? p2 : 511, c2 = code_of(pn);
    { TILE_ARGS(c2, pn, r + 2); asm volatile(TLOAD :: TLOAD_IN : "memory"); }
    if (wv == 0) { if (r & 1) DRAW("v255"); else DRAW("v254"); }
    pm1 = p0; p0 = p1; p1 = p2; c0 = c1; c1 = c2;
  }
  WAIT_V(0);
  if (wg == 0 && wv == 0) {
    const unsigned* cnt = (const unsigned*)(ctl + (lane & 1 ? CTL_TMO : CTL_SLICE));
    const unsigned long long t0 = __builtin_amdgcn_s_memrealtime();
    for (;;) {
      const unsigned v = __hip_atomic_load(cnt, __ATOMIC_RELAXED, __HIP_MEMORY_SCOPE_SYSTEM);
      if ((int)((unsigned)__builtin_amdgcn_readlane(v, 0) - call * (7u * 32u)) >= 0) break;
      if ((unsigned)__builtin_amdgcn_readlane(v, 1) == call || __builtin_amdgcn_s_memrealtime() - t0 > DEADLINE_TICKS) {
        if (lane == 0) {
          __hip_atomic_store((unsigned*)(ctl + CTL_TMO), call, __ATOMIC_RELAXED, __HIP_MEMORY_SCOPE_AGENT);
          p.out[0] = __builtin_nanf("");
        }
        break;
      }
      __builtin_amdgcn_s_sleep(4);
    }
  }
}

#define DMA2(m0v, soff) asm volatile("s_mov_b32 m0, %0\n\ts_nop 0\n\t" \
    "buffer_load_dwordx4 %1, %2, %3 offen " DMA_POL " lds\n\t" \
    "buffer_load_dwordx4 %1, %2, %3 offen offset:1024 " DMA_POL " lds" \
    :: "s"(m0v), "v"(voff), "s"(rdata), "s"(soff) : "memory")
#define DMA3(m0v, soff, m0h, soffh) asm volatile("s_mov_b32 m0, %0\n\ts_nop 0\n\t" \
    "buffer_load_dwordx4 %1, %2, %3 offen " DMA_POL " lds\n\t" \
    "buffer_load_dwordx4 %1, %2, %3 offen offset:1024 " DMA_POL " lds\n\t" \
    "s_mov_b32 m0, %4\n\ts_mov_b32 exec_hi, 0\n\t" \
    "buffer_load_dwordx4 %1, %2, %5 offen " DMA_POL " lds\n\t" \
    "s_mov_b32 exec_hi, -1" \
    :: "s"(m0v), "v"(voff), "s"(rdata), "s"(soff), "s"(m0h), "s"(soffh) : "memory")

#define DMA1(m0v, soff) asm volatile("s_mov_b32 m0, %0\n\ts_nop 0\n\tbuffer_load_dwordx4 %1, %2, %3 offen " DMA_POL " lds" \
    :: "s"(m0v), "v"(voff), "s"(rdata), "s"(soff) : "memory")
#define DMA1B(m0v, soff) asm volatile("s_mov_b32 m0, %0\n\ts_nop 0\n\tbuffer_load_dwordx4 %1, %2, %3 offen offset:1024 " DMA_POL " lds" \
    :: "s"(m0v), "v"(voff), "s"(rdata), "s"(soff) : "memory")
#define DMA1H(m0v, soff) asm volatile("s_mov_b32 m0, %0\n\ts_mov_b32 exec_hi, 0\n\tbuffer_load_dwordx4 %1, %2, %3 offen " DMA_POL " lds\n\t" \
    "s_mov_b32 exec_hi, -1" :: "s"(m0v), "v"(voff), "s"(rdata), "s"(soff) : "memory")
#define PIN(x) asm volatile("" : "+s"(x))

__device__ __forceinline__ int ready_from(unsigned long long okmask, int base) {
  const int r = base + (int)((unsigned)__builtin_ctzll(~okmask | (1ull << 63)) * 57u >> 9);
  return r < 32 ? r : 32;
}

__device__ __forceinline__ void gemm_role(const Params& p, LAS unsigned char* lds, int g, int wv, int lane) {
  const int lin = (g & 7) * 30 + (g >> 3), band = lin / 80, cb = (lin % 80) / 5;
  const int tcol = (band & 1) ? 15 - cb : cb, trow = band * 5 + lin % 5;
  uint8_t* const ctl = p.ctl;
  if (g < 8 && wv < 7 && lane < 2) {
    const unsigned v = __hip_atomic_load((const unsigned*)(p.peer[wv] + DATA_OFF + (size_t)lane * SLOT_B + (size_t)p.rank * 4 * ENT_B),
                                         __ATOMIC_RELAXED, __HIP_MEMORY_SCOPE_SYSTEM);
    asm volatile("" :: "v"(v));
  }
  __builtin_amdgcn_s_sleep(127);
#pragma unroll
  for (int i = 0; i < 3; ++i) {
    const int gid = g + i * NGEMM, type = (gid >> 8) & 1, kt = (gid >> 6) & 3;
    if (gid < 512) {
      const __amdgpu_buffer_rsrc_t rl = __builtin_amdgcn_make_buffer_rsrc((void*)(type ? p.w : p.x), 0, MN * KS, 0x00020000);
      const unsigned lo = tile_src_off(type, kt, gid & 63, wv, lane), lm = (unsigned)(uintptr_t)lds + i * 8192 + wv * 1024;
      asm volatile(TLOAD :: TLOAD_IN : "memory");
    }
  }
  const unsigned call = __builtin_amdgcn_readfirstlane(
      __hip_atomic_load((const unsigned*)(ctl + CTL_DONE), __ATOMIC_RELAXED, __HIP_MEMORY_SCOPE_AGENT)) / NWG + 1u;
  const __amdgpu_buffer_rsrc_t rdata =
      __builtin_amdgcn_make_buffer_rsrc(ctl + DATA_OFF + (size_t)(call & 1u) * SLOT_B, 0, SLOT_B, 0x00020000);
  WAIT_V(0);
  BAR;
#pragma unroll
  for (int i = 0; i < 3; ++i) {
    const int gid = g + i * NGEMM, type = (gid >> 8) & 1, kt = (gid >> 6) & 3;
    if (gid < 512)
      __builtin_amdgcn_raw_buffer_store_b128(unit_read(type, lds + i * 8192, wv, lane), rdata,
                                             (unsigned)p.rank * 4u * ENT_B + unit_dst_off(type, kt, gid & 63, wv) + lane * 16, 0, 17);
  }
  LAS unsigned* ord = (LAS unsigned*)(lds + LDS_ORD);
  LAS unsigned* tbl = (LAS unsigned*)(lds + LDS_TBL);
  LAS unsigned* bc = (LAS unsigned*)(lds + LDS_CTL);
  const int hj = (trow / 5 + 2 - tcol % 3) % 3;
  const bool host = trow % 5 == 0 && hj < 2;
  auto group_of = [&](int i) -> int { return i < 4 ? 4 * trow + i : i < 8 ? 4 * tcol + i - 4 : host ? 60 + (p.rank >> 1) : 4 * trow; };
  auto pub_pos = [&](int s, int kt, int i) -> int {
    const int type = (i >> 2) & 1, gg = group_of(i);
    const int pos = p.inv[s][kt * 32 + type * 16 + (gg >> 2)] * 4 + (gg & 3), r = pos >> 4;
    return r >= p.fine_start ? pos : (r + __builtin_ctz(p.pubmask >> r)) * 16 + 15;
  };
  if (wv == 0) {
    const int e = lane & 31, s = e >> 2, kt = e & 3;
    int key = kt;
    if (s != p.rank) {
      int m = 0;
      for (int i = 0; i < 9; ++i) { const int r = pub_pos(s, kt, i); m = r > m ? r : m; }
      key = 4 + m * 32 + kt * 8 + ((s - p.rank + g) & 7);
    }
    int rk = 0;
    for (int l = 0; l < 32; ++l) rk += __builtin_amdgcn_readlane(key, l) < key;
    if (lane < 32) ord[rk] = e;
  }
  WAIT_V(0);
  __syncthreads();
  if (threadIdx.x == 64) COUNT_IN(ctl);
  if (threadIdx.x == 0)
    __hip_atomic_store((unsigned*)(ctl + CTL_LFLAG) + g, call * 16u, __ATOMIC_RELAXED, __HIP_MEMORY_SCOPE_SYSTEM);
  if (threadIdx.x < 352) {
    int t = threadIdx.x / 9; const int i = threadIdx.x - t * 9; t = t < 32 ? t : 31;
    const int e = ord[t], s = e >> 2, kt = e & 3, type = (i >> 2) & 1, gg = group_of(i);
    unsigned off;
    if (s == p.rank) off = CTL_LFLAG + 4u * (unsigned)((type * 256 + kt * 64 + gg) % NGEMM);
    else { const int pp = pub_pos(s, kt, i);
           off = (pp >> 4) >= p.fine_start ? rflag_off(s, pp) : uflag_off(s, pp >> 4); }
    tbl[threadIdx.x] = off;
  }
  const int ordv = ord[lane & 31];
  ((LAS unsigned long long*)(lds + LDS_SNAP))[threadIdx.x] = 0ull;
  ((LAS unsigned*)(lds + LDS_SNAP + 4096))[threadIdx.x] = 0u;
  if (threadIdx.x == 0) bc[63] = 0u;
  if (threadIdx.x < 7) ((LAS unsigned long long*)(bc + 32))[threadIdx.x] = (unsigned long long)(p.peer[threadIdx.x] + CTL_SLICE);
  if (wv == 7 && lane < 5 + (host ? 8 : 0)) {
    const float* q = lane < 5 ? p.out + (size_t)(trow * 256 + (lane == 4 ? 255 : lane * 64)) * MN + tcol * 256
                              : p.outs[lane - 5] + (size_t)(3840 + 32 * p.rank + 16 * hj) * MN + tcol * 256;
    const unsigned v = __hip_atomic_load((const unsigned*)q, __ATOMIC_RELAXED, __HIP_MEMORY_SCOPE_SYSTEM);
    asm volatile("" :: "v"(v));
  }
  int scale_bits = __builtin_amdgcn_readfirstlane(__builtin_bit_cast(int, p.sx[0] * p.sw[0]));
  asm volatile("" : "+s"(scale_bits));
  WAIT_V(0);
  __syncthreads();

  const int wr = wv >> 2, wc = wv & 3;
  const unsigned voff = lane * 16;
  const unsigned ldsb = (unsigned)(uintptr_t)lds;
  const unsigned a0o = (9 * wr + wc) * 2048, a8o = (9 * wr + 8) * 2048 + wc * 512, b0o = (4 * (wv >> 1) + (wv & 1)) * 2048;
  const bool has9 = host && wr == 0;
  const unsigned ga0 = (unsigned)(16 * trow + 8 * wr + wc) * 2048u, gb0 = OP_B + tcol * 32768u + b0o;
  const unsigned ga8 = has9 ? (unsigned)(240 + 2 * p.rank + hj) * 2048u + wc * 512u : OOB;
  const unsigned la0 = ldsb + LDS_A + a0o, la8 = ldsb + LDS_A + a8o, lb0 = ldsb + LDS_B + b0o;
#define ENT(t) ((unsigned)__builtin_amdgcn_readlane(ordv, (t)) * ENT_B)
#define ST_A03(eb, sl) DMA2(la0 + (sl) * A_SLOT, (eb) + ga0)
#define ST_A48(eb, sl) DMA3(la0 + 8192 + (sl) * A_SLOT, (eb) + ga0 + 8192, la8 + (sl) * A_SLOT, (eb) + ga8)
#define ST_B0(eb, sl) DMA2(lb0 + (sl) * B_SLOT, (eb) + gb0)
#define ST_B1(eb, sl) DMA2(lb0 + 4096 + (sl) * B_SLOT, (eb) + gb0 + 4096)
#define ST_ALL(eb, sl) do { ST_B0(eb, sl); ST_B1(eb, sl); ST_A03(eb, sl); ST_A48(eb, sl); } while (0)
#define LDF(dst, base, off) do { const i32x4 _l = *(const LAS i32x4*)((base) + (off)), _h = *(const LAS i32x4*)((base) + (off) + 1024); \
    dst = __builtin_shufflevector(_l, _h, 0, 1, 2, 3, 4, 5, 6, 7); SCHED; } while (0)
#define MM(m, n, Af) do { acc[m][n] = __builtin_amdgcn_mfma_scale_f32_16x16x128_f8f6f4(Af, Bf[n], acc[m][n], 1, 1, 0, 0, 0, 0); SCHED; } while (0)

  f32x4 acc[9][4];
#pragma unroll
  for (int m = 0; m < 9; ++m)
#pragma unroll
    for (int n = 0; n < 4; ++n) acc[m][n] = (f32x4){0.f, 0.f, 0.f, 0.f};
  i32x8 a0, a1, a2, a3, a4, Bf[4];
  int t = 0, ready = 0, nrestart = 0;
  unsigned dead = 0;
  const unsigned thr = call * 16u;
  const unsigned pollo = tbl[threadIdx.x < 352 ? threadIdx.x : 351];
  const __amdgpu_buffer_rsrc_t rctl = __builtin_amdgcn_make_buffer_rsrc(ctl, 0, CTL_BYTES, 0x00020000);
  const __amdgpu_buffer_rsrc_t rout = __builtin_amdgcn_make_buffer_rsrc(p.out, 0, MN * MN * 4, 0x00020000);
  const LAS unsigned char* const pa0 = lds + LDS_A + wr * (9 * 2048) + lane * 16;
  const LAS unsigned char* const pb0 = lds + LDS_B + wc * (4 * 2048) + lane * 16;
#define POLL(buf) do { const unsigned _po = (unsigned)(ready * 9 + 62 - wv * 64) < 126u && ready < 32 ? 0u : OOB; \
    asm volatile("s_mov_b32 m0, %0\n\ts_nop 0\n\tbuffer_load_dword %1, %2, %3 offen " DMA_POL " lds" \
                 :: "s"(ldsb + LDS_SNAP + (buf) * 2048 + wv * 256), "v"(pollo), "s"(rctl), "s"(_po) : "memory"); } while (0)
#define SNAP_AT(buf) ((const LAS unsigned*)(lds + LDS_SNAP + ((buf) * 512 + ready * 9) * 4) + lane)
#define OKMASK(v) __builtin_amdgcn_ballot_w64((int)((v) - thr) >= 0)
#define STAGE_FROM(u) do { const unsigned _e0 = ENT(u); ST_ALL(_e0, (u) & 1); sn = ready > (u) + 1; \
    if (sn) { BAR;   const unsigned _e1 = ENT((u) + 1); ST_ALL(_e1, ((u) & 1) ^ 1); } \
    POLL(((u) & 1) ^ 1); } while (0)
  int sn = -1;
  for (;;) {
    if (sn < 0) {
      if (ready <= t) {
        if (wv == 0) {
          const unsigned* fp = (const unsigned*)(ctl + (lane == 63 ? CTL_TMO : tbl[t * 9 + lane]));
          unsigned* tmo = (unsigned*)(ctl + CTL_TMO);
          const unsigned long long t0 = __builtin_amdgcn_s_memrealtime();
          int r; unsigned dd = 0;
          for (;;) {
            const unsigned v = __hip_atomic_load(fp, __ATOMIC_RELAXED, __HIP_MEMORY_SCOPE_SYSTEM);
            r = ready_from(OKMASK(v), t);
            if (r > t) break;
            if ((unsigned)__builtin_amdgcn_readlane(v, 63) == call || __builtin_amdgcn_s_memrealtime() - t0 > DEADLINE_TICKS) {
              if (lane == 0) __hip_atomic_store(tmo, call, __ATOMIC_RELAXED, __HIP_MEMORY_SCOPE_AGENT);
              r = 32; dd = 64; break;
            }
            __builtin_amdgcn_s_sleep(2);
          }
          if (lane == 0) bc[nrestart & 31] = (unsigned)r | (dd << 8);
          asm volatile("s_waitcnt vmcnt(0) lgkmcnt(0)" ::: "memory");
        }
        BAR;
        const unsigned v = __builtin_amdgcn_readfirstlane(bc[nrestart & 31]);
        ready = v & 63; dead |= v >> 8; ++nrestart;
      }
      STAGE_FROM(t);
    }
    int s1 = sn;
    sn = -1;
    if (s1) WAIT_V(13); else WAIT_V(4);
    BAR;
    { const LAS unsigned char* pa = pa0 + (t & 1) * A_SLOT; const LAS unsigned char* pb = pb0 + (t & 1) * B_SLOT;
      LDF(Bf[0], pb, 0); LDF(a0, pa, 0); LDF(a1, pa, 2048); LDF(Bf[1], pb, 2048); LDF(Bf[2], pb, 4096); LDF(a2, pa, 4096); LDF(Bf[3], pb, 6144); }
    if (t == 31) break;
    unsigned en2 = ENT((t + 2) & 31);
    int first = 1;
    for (;;) {
      const int sl = t & 1;
      const LAS unsigned char* pa = pa0 + sl * A_SLOT;
      LDF(a3, pa, 3 * 2048);
      MM(0, 0, a0);
      const int rich = (ready > t + 2 || ready == 32 || (first & s1)) ? 1 : 0;
      const LAS unsigned* psn = SNAP_AT(sl ^ rich ^ 1);
      asm volatile("" : "+v"(psn)); SCHED;
      MM(1, 0, a1);
      const LAS unsigned char* pan = pa0 + (sl ^ 1) * A_SLOT; const LAS unsigned char* pbn = pb0 + (sl ^ 1) * B_SLOT;
      asm volatile("" : "+v"(pan), "+v"(pbn)); SCHED;
      MM(0, 1, a0);
      unsigned mB = lb0 + sl * B_SLOT, mA = la0 + sl * A_SLOT;
      PIN(mB); PIN(mA); SCHED;
      MM(1, 1, a1);
      unsigned mA8 = la8 + sl * A_SLOT, mP = ldsb + LDS_SNAP + sl * 2048 + wv * 256;
      PIN(mA8); PIN(mP); SCHED;
      MM(0, 2, a0); MM(1, 2, a1);
      WAIT_V(10); if (!rich) { if (first) WAIT_V(0); else WAIT_V(3); }
      first = 0;
      WAIT_L(0); BAR; SCHED;
      const unsigned snap = *psn;
      SCHED;
      MM(0, 3, a0);
      LDF(a4, pa, 6 * 2048);
      MM(1, 3, a1);
      LDF(a0, pa, 4 * 2048);
      MM(2, 0, a2);
      unsigned long long okm = OKMASK(snap);
      SCHED;
      MM(2, 1, a2);
      LDF(a1, pa, 5 * 2048);
      unsigned fz = (unsigned)__builtin_ctzll(~okm | (1ull << 63)) * 57u;
      PIN(fz); SCHED;
      MM(2, 2, a2);
      ready += (int)(fz >> 9); ready = ready < 32 ? ready : 32;
      if (__builtin_expect(!s1, 0)) {
        if (ready > t + 1) { const unsigned e1 = ENT(t + 1); ST_ALL(e1, sl ^ 1); s1 = 1; }
        POLL(2);
      }
      SCHED;
      MM(2, 3, a2);
      int s2 = ready > t + 2 ? s1 : 0;
      unsigned e2 = s2 ? en2 : OOB;
      PIN(s2); PIN(e2);
      LDF(a2, pa, 8 * 2048);
      MM(3, 0, a3);
      DMA1(mB, e2 + gb0); SCHED;
      MM(3, 1, a3);
      DMA1B(mB, e2 + gb0); SCHED;
      MM(3, 2, a3);
      DMA1(mB + 4096, e2 + gb0 + 4096); SCHED;
      MM(3, 3, a3);
      LDF(a3, pa, 7 * 2048);
      DMA1B(mB + 4096, e2 + gb0 + 4096); SCHED;
      MM(4, 0, a0);
      DMA1(mA, e2 + ga0); SCHED;
      MM(4, 1, a0);
      DMA1B(mA, e2 + ga0); SCHED;
      MM(4, 2, a0);
      en2 = ENT((t + 3) & 31);
      SCHED;
      MM(4, 3, a0);
      unsigned po = (unsigned)(ready * 9 + 62 - wv * 64) < 126u && ready < 32 ? 0u : OOB;
      PIN(po); SCHED;
      MM(5, 0, a1); MM(5, 1, a1);
      WAIT_V(10); if (__builtin_expect(!s1, 0)) WAIT_V(0);
      WAIT_L(0); BAR; SCHED;
      MM(5, 2, a1);
      LDF(a0, pan, 0);
      asm volatile("s_mov_b32 m0, %0\n\ts_nop 0\n\tbuffer_load_dword %1, %2, %3 offen " DMA_POL " lds"
                   :: "s"(mP), "v"(pollo), "s"(rctl), "s"(po) : "memory");
      SCHED;
      MM(5, 3, a1);
      LDF(a1, pan, 2048);
      DMA1(mA + 8192, e2 + ga0 + 8192); SCHED;
      if (has9) { MM(8, 0, a2); MM(8, 1, a2); MM(8, 2, a2); MM(8, 3, a2); }
      LDF(a2, pan, 4096);
      DMA1B(mA + 8192, e2 + ga0 + 8192); SCHED;
      MM(7, 0, a3);
      DMA1H(mA8, e2 + ga8);
      if (__builtin_expect(!s1, 0)) {
        ready = ready_from(OKMASK(*SNAP_AT(2)), ready);
        if (ready > t + 1) STAGE_FROM(t + 1);
      }
      SCHED;
      MM(6, 0, a4);
      LDF(Bf[0], pbn, 0);
      MM(7, 1, a3); MM(6, 1, a4);
      LDF(Bf[1], pbn, 2048);
      MM(7, 2, a3); MM(6, 2, a4);
      LDF(Bf[2], pbn, 4096);
      MM(7, 3, a3); MM(6, 3, a4);
      LDF(Bf[3], pbn, 6144);
      ++t;
      if (!s1 || t == 31) break;
      s1 = s2;
    }
    if (s1) break;
    if (sn < 0) {
      asm volatile("s_waitcnt vmcnt(0) lgkmcnt(0)" ::: "memory"); BAR;
      ready = ready_from(OKMASK(*SNAP_AT((t & 1) ^ 1)), ready);
      WAIT_L(0); BAR;
    }
  }
  {
    const float sc = __builtin_bit_cast(float, dead ? 0x7fc00000 : scale_bits);
    const unsigned ov = (unsigned)((lane >> 4) * 4 * MN + (lane & 15) * 4) * 4u;
    const unsigned os = (unsigned)((trow * 256 + wr * 128) * MN + tcol * 256 + wc * 64) * 4u;
#define OUT(m) do { f32x4 o[4]; \
      asm volatile("" : "+v"(acc[m][0]), "+v"(acc[m][1]), "+v"(acc[m][2]), "+v"(acc[m][3])); \
      _Pragma("unroll") for (int j = 0; j < 4; ++j) { o[j][0] = acc[m][0][j] * sc; o[j][1] = acc[m][1][j] * sc; o[j][2] = acc[m][2][j] * sc; o[j][3] = acc[m][3][j] * sc; } \
      SCHED; \
      _Pragma("unroll") for (int j = 0; j < 4; ++j) \
        __builtin_amdgcn_raw_buffer_store_b128(__builtin_bit_cast(u32x4, o[j]), rout, ov, os + (unsigned)((m) * 16 + j) * (MN * 4u), OUT_AUX); \
      asm volatile("s_nop 1" ::: "memory");                     \
      SCHED; } while (0)
    const LAS unsigned char* pa = pa0 + A_SLOT;
    LDF(a3, pa, 3 * 2048);
    MM(0, 0, a0); MM(1, 0, a1); MM(0, 1, a0); MM(1, 1, a1); MM(0, 2, a0); MM(1, 2, a1);
    WAIT_V(0); BAR; SCHED;
    MM(0, 3, a0);
    LDF(a4, pa, 8 * 2048);
    MM(1, 3, a1);
    LDF(a0, pa, 4 * 2048);
    MM(2, 0, a2); MM(2, 1, a2);
    LDF(a1, pa, 5 * 2048);
    MM(2, 2, a2); MM(2, 3, a2);
    LDF(a2, pa, 6 * 2048);
    f32x4 o8[4];
    const unsigned os8 = (unsigned)((3840 + 32 * p.rank + 16 * hj) * MN + tcol * 256 + wc * 64) * 4u;
#define SHARE(d) do { if (has9) { \
      const __amdgpu_buffer_rsrc_t rd = __builtin_amdgcn_make_buffer_rsrc(p.outs[(p.rank + 1 + ((d) < 7 ? ((d) + tcol + wc) % 7 : 7)) & (NR - 1)], 0, MN * MN * 4, 0x00020000); \
      asm volatile("s_waitcnt vmcnt(" STR(SHARE_WIN) ")" ::: "memory"); \
      _Pragma("unroll") for (int j = 0; j < 4; ++j) \
        __builtin_amdgcn_raw_buffer_store_b128(__builtin_bit_cast(u32x4, o8[j]), rd, ov, os8 + (unsigned)j * (MN * 4u), 17); } \
      SCHED; } while (0)
    if (has9) { MM(8, 0, a4); MM(8, 1, a4); MM(8, 2, a4); MM(8, 3, a4); }
    OUT(0);
#pragma unroll
    for (int j = 0; j < 4; ++j) { o8[j][0] = acc[8][0][j] * sc; o8[j][1] = acc[8][1][j] * sc; o8[j][2] = acc[8][2][j] * sc; o8[j][3] = acc[8][3][j] * sc; }
    SCHED;
    SHARE(0);
    MM(3, 0, a3); MM(3, 1, a3); MM(3, 2, a3); MM(3, 3, a3);
    LDF(a3, pa, 7 * 2048);
    OUT(1); SHARE(1);
    MM(4, 0, a0); MM(4, 1, a0); MM(4, 2, a0); MM(4, 3, a0);
    OUT(2); SHARE(2);
    MM(5, 0, a1); MM(5, 1, a1); MM(5, 2, a1); MM(5, 3, a1);
    OUT(3); SHARE(3);
    MM(6, 0, a2); MM(6, 1, a2); MM(6, 2, a2); MM(6, 3, a2);
    OUT(4); SHARE(4);
    MM(7, 0, a3); MM(7, 1, a3); MM(7, 2, a3); MM(7, 3, a3);
    OUT(5); SHARE(5); OUT(6); SHARE(6); OUT(7); SHARE(7);
    if (has9) {
      WAIT_V(8);
      unsigned old = 0;
      if (lane == 0) old = __hip_atomic_fetch_add(bc + 63, 1u, __ATOMIC_RELAXED, __HIP_MEMORY_SCOPE_WORKGROUP);
      if (__builtin_amdgcn_readfirstlane(old) == 3u && lane < 7)
        __hip_atomic_fetch_add((GAS unsigned*)((LAS unsigned long long*)(bc + 32))[lane], 1u, __ATOMIC_RELAXED, __HIP_MEMORY_SCOPE_SYSTEM);
    }
  }
}

__global__ void __launch_bounds__(512, 2) gemm_ag_kernel(const Params p) {
  __shared__ __attribute__((aligned(16))) unsigned char lds_raw[LDS_BYTES];
  LAS unsigned char* lds = (LAS unsigned char*)lds_raw;
  const int wv = __builtin_amdgcn_readfirstlane(threadIdx.x >> 6), lane = threadIdx.x & 63;
  if (blockIdx.x < NCOMM) comm_role(p, lds, blockIdx.x, wv, lane);
  else gemm_role(p, lds, blockIdx.x - NCOMM, wv, lane);
}

static void make_schedule(int s, uint8_t* sc) {
  static const int nlate[5] = {0, 3, 5, 8, 8};
  static const double fin[5] = {0.0, 3.0, 5.2, 7.6, 7.6};
  int r = 0;
  for (int kt = 0; kt < 3; ++kt)
    for (int q = 0; q < 16; ++q) {
      sc[r++] = (uint8_t)(kt * 32 + 16 + (q + 2 * s) % 16);
      sc[r++] = (uint8_t)(kt * 32 + (q + 2 * s) % 16);
    }
  double due[16]; int late[16], nl = 0;
  sc[r++] = (uint8_t)(96 + 15);
  for (int c = 0; c < 15; ++c) {
    const int lv = c % 5, i = ((s - (lv * 3 + c / 5) * 3) % 8 + 8) % 8;
    if (i < nlate[lv]) { due[nl] = fin[lv] - (nlate[lv] - i) * 1.15 + 0.01 * (c / 5); late[nl++] = c; }
    else sc[r++] = (uint8_t)(96 + c);
  }
  for (int q = 0; q < 16; ++q) sc[r++] = (uint8_t)(96 + 16 + (q + 2 * s) % 16);
  for (int k = 0; k < nl; ++k) {
    int best = -1;
    for (int j = 0; j < nl; ++j) if (late[j] >= 0 && (best < 0 || due[j] < due[best])) best = j;
    sc[r++] = (uint8_t)(96 + late[best]); late[best] = -1;
  }
}

}

extern "C" void kernel_launch(int rank, int nranks, ncclComm_t comm,
                              hipStream_t stream,
                              void* const* d_in, const int* in_sizes, int n_in,
                              void* d_out, int out_size,
                              const int* in_dtypes, int out_dtype,
                              void* const* d_out_all,
                              void* const* d_ws_all, size_t ws_bytes) {
  (void)comm; (void)in_dtypes; (void)out_dtype;
  if (nranks != NR || n_in != 4 || (unsigned)rank >= (unsigned)NR || ws_bytes < WS_NEED || out_size != MN * MN ||
      in_sizes[0] != MN * KS || in_sizes[1] != MN * KS) {
    fprintf(stderr, "kernel_launch: unexpected problem shape\n");
    return;
  }
  Params p;
  p.x = (const uint8_t*)d_in[0]; p.w = (const uint8_t*)d_in[1];
  p.sx = (const float*)d_in[2]; p.sw = (const float*)d_in[3];
  p.out = (float*)d_out;
  for (int i = 0; i < NR; ++i) p.outs[i] = (float*)d_out_all[i];
  p.ctl = (uint8_t*)d_ws_all[rank];
  for (int i = 0; i < 14; ++i) p.peer[i] = (uint8_t*)d_ws_all[(rank + 1 + i % 7) % NR];
  p.rank = rank;
  p.pubmask = 0x00AA8080u;
  p.fine_start = 24;
  for (int s = 0; s < NR; ++s) {
    uint8_t sc[128];
    make_schedule(s, sc);
    for (int r = 0; r < 128; ++r) p.inv[s][sc[r]] = (uint8_t)r;
    if (s == rank)
      for (int i = 0; i < 32; ++i)
        p.sched[i] = sc[4 * i] | (sc[4 * i + 1] << 8) | (sc[4 * i + 2] << 16) | ((unsigned)sc[4 * i + 3] << 24);
  }
  p.sched01[0] = p.sched[0]; p.sched01[1] = p.sched[1];
  hipLaunchKernelGGL(gemm_ag_kernel, dim3(NWG), dim3(512), 0, stream, p);
  HIP_CHECK(hipGetLastError());
}
```
